```python
import numpy as np

kernel_name = "gpu_gemm_ar_m4096_k4096_n16384_w8a8_dqrelu"


ABSMAX_RTOL = 0.001

COMPUTE_DTYPE = "int8"
EPILOGUE = "dequant_relu"
IN_QUANT = "int8"
OUT_QUANT = None
IN_DTYPES = {"x": "int8", "w_mat": "int8",
             "scale_x": "float32", "scale_w": "float32"}
WORLD_SIZE = 8

def setup_inputs():

    x = np.random.randint(-127, 128, size=(4096, 4096)).astype(np.int8)
    w_mat = np.random.randint(-127, 128, size=(4096, 16384)).astype(np.int8)
    scale_x = (0.02 * (1.0 + np.random.rand(1))).astype(np.float32)
    scale_w = (0.02 * (1.0 + np.random.rand(1))).astype(np.float32)
    return {"x": x, "w_mat": w_mat, "scale_x": scale_x, "scale_w": scale_w}

def reference(x, w_mat, scale_x, scale_w):
    acc = x.astype(np.float64) @ w_mat.astype(np.float64)
    y = (acc.astype(np.float32) * (scale_x * scale_w)).astype(np.float32)
    return np.maximum(y, 0.0).astype(np.float32)
```

```cpp
#include <hip/hip_runtime.h>
#include <rccl/rccl.h>
#include <cstdio>

#define HIP_CHECK(expr) do { hipError_t _e = (expr); if (_e != hipSuccess) { \
    fprintf(stderr, "HIP error %s at %s:%d\n", hipGetErrorString(_e), __FILE__, __LINE__); \
    return; } } while (0)

namespace {

typedef int i32x4 __attribute__((ext_vector_type(4)));
typedef unsigned u32x4 __attribute__((ext_vector_type(4)));
typedef float f32x4 __attribute__((ext_vector_type(4)));
#define LAS __attribute__((address_space(3)))
#define GAS __attribute__((address_space(1)))

constexpr int M = 4096, N = 16384, KSH = 512;
constexpr int NWG = 256;
constexpr unsigned OFF_CNT = 0;
constexpr unsigned OFF_TMO = 32768;
constexpr unsigned OFF_FLAGS = 65536;
constexpr unsigned OFF_DBG = 1u << 20;
constexpr unsigned OFF_STAGE = 4u << 20;
constexpr unsigned SHARD_BYTES = 10u << 20;
constexpr unsigned REGION_BYTES = 2u * SHARD_BYTES;
constexpr unsigned WS_NEED = OFF_STAGE + 8u * REGION_BYTES;
constexpr int LDS_BYTES = 131072 + 64;
constexpr unsigned DEADLINE_TICKS = 30000000u;

#define STAMP 0
#define SLOT_P1 1900
#define SLOT_P2 2000
#define LEAD_SLOTS -50
#define SKIP_MUL 650
#define EXP_NOWAIT 0
#define EXP_PUSHMASK 0
#define EXP_GMASK 127
#define EXP_PARITY 1u
#define DMA_AUX 17
#define POLL_AUX 17
#define POLL_SLEEP 4
#define POLL_WIN 64
#define AUX_SYS 17
#define AUX_NT 2

struct Params {
    const signed char* x;
    const signed char* w;
    const float* sx;
    const float* sw;
    float* out;
    char* ws[8];
    int rank;
};

#define WAIT_V(n) asm volatile("s_waitcnt vmcnt(%0)" ::"n"(n) : "memory")
#define WAIT_L0() __builtin_amdgcn_s_waitcnt(0xC07F)
#define BAR() __builtin_amdgcn_s_barrier()
#define SCHED() __builtin_amdgcn_sched_barrier(0)

#define WT1(n) "s_waitcnt vmcnt(" #n ")\n\ts_branch .Lwv_end%=\n\t"
#define WT8(a, b, c, d, e, f, g, h) WT1(a) WT1(b) WT1(c) WT1(d) WT1(e) WT1(f) WT1(g) WT1(h)
__device__ __forceinline__ void wait_vm_dyn(int e) {
    e = e > 55 ? 55 : e;
    const int jump = e * 8 + 12;
    asm volatile(
        "s_getpc_b64 vcc\n\t"
        "s_add_u32 vcc_lo, vcc_lo, %0\n\t"
        "s_addc_u32 vcc_hi, vcc_hi, 0\n\t"
        "s_setpc_b64 vcc\n\t"
        WT8(8, 9, 10, 11, 12, 13, 14, 15) WT8(16, 17, 18, 19, 20, 21, 22, 23) WT8(24, 25, 26, 27, 28, 29, 30, 31)
        WT8(32, 33, 34, 35, 36, 37, 38, 39) WT8(40, 41, 42, 43, 44, 45, 46, 47) WT8(48, 49, 50, 51, 52, 53, 54, 55)
        WT8(56, 57, 58, 59, 60, 61, 62, 63)
        ".Lwv_end%=:"
        ::"s"(jump) : "vcc", "scc", "memory");
}

template <int AUX>
__device__ __forceinline__ void store16(u32x4 v, __amdgpu_buffer_rsrc_t r, int vo, unsigned so) {
    if constexpr (AUX == AUX_NT) asm volatile("buffer_store_dwordx4 %0, %1, %2, %3 offen nt\n\ts_nop 1" ::"v"(v), "v"(vo), "s"(r), "s"(so) : "memory");
    else asm volatile("buffer_store_dwordx4 %0, %1, %2, %3 offen sc0 sc1\n\ts_nop 1" ::"v"(v), "v"(vo), "s"(r), "s"(so) : "memory");
}
__device__ __forceinline__ void store4k(const i32x4 (&v)[4], __amdgpu_buffer_rsrc_t r, int vo, unsigned so) {
    asm volatile("buffer_store_dwordx4 %0, %4, %5, %6 offen sc0 sc1\n\t"
                 "buffer_store_dwordx4 %1, %4, %5, %6 offen offset:1024 sc0 sc1\n\t"
                 "buffer_store_dwordx4 %2, %4, %5, %6 offen offset:2048 sc0 sc1\n\t"
                 "buffer_store_dwordx4 %3, %4, %5, %6 offen offset:3072 sc0 sc1\n\t"
                 "s_nop 1"
                 ::"v"(v[0]), "v"(v[1]), "v"(v[2]), "v"(v[3]), "v"(vo), "s"(r), "s"(so) : "memory");
}
__device__ __forceinline__ void store1k(i32x4 v, __amdgpu_buffer_rsrc_t r, int vo, unsigned so) {
    asm volatile("s_nop 4\n\tbuffer_store_dwordx4 %0, %1, %2, %3 offen sc0 sc1\n\ts_nop 1" ::"v"(v), "v"(vo), "s"(r), "s"(so) : "memory");
}

__device__ __forceinline__ unsigned now_ticks() { return (unsigned)__builtin_amdgcn_s_memrealtime(); }

__device__ __forceinline__ int due_of(int n) {
    const int a = n < 1024 ? n : 1024, b = n - a;
    return (a * SLOT_P1 + b * SLOT_P2 - LEAD_SLOTS * SLOT_P1) >> 8;
}

__global__ __launch_bounds__(512, 2) void fused_kernel(Params P) {
    extern __shared__ __attribute__((aligned(16))) char smem_[];
    LAS char* lds = (LAS char*)smem_;
    volatile LAS unsigned* lds_word = (volatile LAS unsigned*)(lds + 131072);
    const unsigned t0 = now_ticks();
    const int tid = threadIdx.x, lane = tid & 63;
    const int wid = __builtin_amdgcn_readfirstlane(tid >> 6);
    const int wr = wid >> 2, wc = wid & 3;
    const int b = blockIdx.x;
    const int xcd = b & 7, idx = b >> 3;
    const int rb = (xcd >> 1) * 4 + (idx & 3);
    const int cq = (idx >> 2) * 2 + (xcd & 1);
#define STMP() do {} while (0)
    STMP();

    char* const my_ws = P.ws[0];
    const __amdgpu_buffer_rsrc_t rW = __builtin_amdgcn_make_buffer_rsrc(my_ws, 0, WS_NEED, 0x00020000);
    const unsigned call_v = __builtin_amdgcn_raw_buffer_load_b32(rW, 0, OFF_CNT + b * 128, AUX_SYS);
    const int rot = (b + wid) % 7;
    unsigned long long peer_ws;
    {
        int ql = (lane & 7) + rot; ql = ql >= 7 ? ql - 7 : ql; ql += 1;
        peer_ws = (unsigned long long)P.ws[ql & 7] + ((lane & 8) ? OFF_STAGE + (unsigned)(8 - ql) * REGION_BYTES : OFF_FLAGS - (unsigned)ql * 32u);
    }
    if (tid == 0) { lds_word[0] = 0u; lds_word[1] = 0u; }

    const int wp = (b & ~15) | ((b & 7) << 1) | ((b >> 3) & 1);
    const int nB = wid * 256 + wp;
    const int jB = nB >> 9, cB = (nB >> 7) & 3, cqB = (nB >> 3) & 15, kkB = (nB >> 2) & 1, quB = nB & 3;
    const int slotB = jB == 0 ? cB * 256 + 128 + (nB & 127) : 512 + nB;
    const bool hasA = wid >= 6;
    const int nA = (wid & 1) * 256 + wp;
    const int cA = (nA >> 7) & 3, rbA = (nA >> 3) & 15, kkA = (nA >> 2) & 1, quA = nA & 3;
    const int slotA = cA * 256 + (nA & 127);

    i32x4 pa[4], pb[4];
    {
        unsigned d[16];
        const signed char* wsrc = P.w + (size_t)((cB * 2 + kkB) * 64 + (lane >> 4) * 16) * N + (jB * 16 + cqB) * 256 + quB * 64 + (lane & 15) * 4;
#pragma unroll
        for (int i = 0; i < 16; ++i) d[i] = *(const unsigned*)(wsrc + (size_t)i * N);
        {
            const signed char* xsrc = P.x + (size_t)(rbA * 256 + quA * 64 + (lane & 15)) * KSH + (cA * 2 + kkA) * 64 + (lane >> 4) * 16;
#pragma unroll
            for (int f = 0; f < 4; ++f) pa[f] = hasA ? *(const i32x4*)(xsrc + (size_t)f * 16 * KSH) : (i32x4){0, 0, 0, 0};
        }
#pragma unroll
        for (int dd = 0; dd < 4; ++dd) {
            const unsigned lo01 = __builtin_amdgcn_perm(d[4 * dd + 1], d[4 * dd], 0x05010400u), hi01 = __builtin_amdgcn_perm(d[4 * dd + 1], d[4 * dd], 0x07030602u);
            const unsigned lo23 = __builtin_amdgcn_perm(d[4 * dd + 3], d[4 * dd + 2], 0x05010400u), hi23 = __builtin_amdgcn_perm(d[4 * dd + 3], d[4 * dd + 2], 0x07030602u);
            pb[0][dd] = (int)__builtin_amdgcn_perm(lo23, lo01, 0x05040100u);
            pb[1][dd] = (int)__builtin_amdgcn_perm(lo23, lo01, 0x07060302u);
            pb[2][dd] = (int)__builtin_amdgcn_perm(hi23, hi01, 0x05040100u);
            pb[3][dd] = (int)__builtin_amdgcn_perm(hi23, hi01, 0x07060302u);
        }
    }
    const unsigned call = __builtin_amdgcn_readfirstlane(call_v) + 1u;
    const unsigned par_off = (call & EXP_PARITY) * SHARD_BYTES;
    const int voff = lane * 16;
    const unsigned flagB = (unsigned)(16 + jB * 16 + cqB) * 1024 + kkB * 16 + quB * 4;
    const unsigned flagA = (unsigned)rbA * 1024 + kkA * 16 + quA * 4;

    int jstate = 0;
    int jleft = hasA ? 2 : 1;
    int jdue = due_of(hasA ? slotA : slotB);
    const unsigned dataA = (unsigned)nA * 4096u, dataB = (2u << 20) + (unsigned)nB * 4096u;
    unsigned jdata = hasA ? dataA : dataB;
    unsigned jflag = hasA ? flagA : flagB;
    const int jc = cB;
    int jage = 0;
    int c1 = 0, c2 = 0, c3 = 0;
    bool failed = false;

    const int dueB = due_of(slotB);
    auto send_frag = [&]() {
        i32x4 v;
        const int sel = jstate + (jleft > 1 ? 4 : 0);
        switch (sel) {
            case 0: v = pb[0]; break; case 1: v = pb[1]; break; case 2: v = pb[2]; break; case 3: v = pb[3]; break;
            case 4: v = pa[0]; break; case 5: v = pa[1]; break; case 6: v = pa[2]; break; default: v = pa[3]; break;
        }
        const unsigned so = par_off + jdata + (unsigned)jstate * 1024u;
#pragma unroll
        for (int i = 0; i < 7; ++i) {
            const unsigned lo = __builtin_amdgcn_readlane((unsigned)peer_ws, 8 + i), hi = __builtin_amdgcn_readlane((unsigned)(peer_ws >> 32), 8 + i);
            const __amdgpu_buffer_rsrc_t r = __builtin_amdgcn_make_buffer_rsrc((char*)(((unsigned long long)hi << 32) | lo), 0, EXP_PUSHMASK == 1 ? 0u : EXP_PUSHMASK == 2 ? ((P.rank < 4 && ((P.rank + (i + rot) % 7 + 1) & 7) >= 4) ? REGION_BYTES : 0u) : REGION_BYTES, 0x00020000);
            store1k(v, r, voff, so);
        }
        ++jstate;
    };
    auto send_flags = [&]() {
        if (voff < 7 * 16) {
            GAS unsigned* fp = (GAS unsigned*)(peer_ws + (jflag + (unsigned)(11 + 7 * jc) * 32u));
            __hip_atomic_store(fp, call, __ATOMIC_RELAXED, __HIP_MEMORY_SCOPE_SYSTEM);
        }
        jstate = jleft > 1 ? 0 : 5;
        jleft -= 1;
        jdue = dueB; jdata = dataB; jflag = flagB;
    };
    int attn = 0;
    auto seg_slow = [&](int seg) {
        c3 = c2; c2 = c1; c1 = 0;
        int skip = 0x3fffffff;
        if (jstate < 4) {
            int left = 0;
            if (jstate == 0) left = jdue - (int)(now_ticks() - t0);
            if (left <= 0) {
                send_frag(); c1 += 7;
                jage = 0;
            }
            skip = left > 0 ? (left * SKIP_MUL) >> 16 : 0;
        } else if (jstate == 4) {
            if (++jage > 3) { send_flags(); c1 += 1; }
            skip = 0;
        }
        wait_vm_dyn(c1 + c2 + c3);
        attn = seg + 1 + (c1 + c2 != 0 ? 0 : skip);
        c3 = 0;
    };
    auto service_blocking = [&]() {
        if (jstate < 4 && (int)(now_ticks() - t0) >= jdue) {
            while (jstate < 4) send_frag();
        }
        if (jstate == 4) {
            WAIT_V(0);
            c1 = 0; c2 = 0; c3 = 0;
            send_flags(); c1 += 1;
        }
        attn = 0;
    };

    if ((int)(now_ticks() - t0) >= jdue) { while (jstate < 4) send_frag(); }
    {
        if (hasA) store4k(pa, rW, voff, OFF_STAGE + par_off + dataA);
        store4k(pb, rW, voff, OFF_STAGE + par_off + dataB);
        WAIT_V(0);
        if (lane == 0) {
            __builtin_amdgcn_raw_buffer_store_b32(call, rW, 0, OFF_FLAGS + flagB + cB * 32u, AUX_SYS);
            if (hasA) __builtin_amdgcn_raw_buffer_store_b32(call, rW, 0, OFF_FLAGS + flagA + cA * 32u, AUX_SYS);
        }
        if (jstate == 4) send_flags();
    }
    const float scale = P.sx[0] * P.sw[0];
    WAIT_L0();
    BAR();
    STMP();

    int ready = EXP_NOWAIT ? 1 << 20 : 0;
    auto ensure_slow = [&](int need) {
        const unsigned w0 = now_ticks();
        unsigned spins = 0;
        for (;;) {
            if (wid == 0) {
                const int round = need >> 5;
                const int p0 = ((need & 31) * 2) & ~7;
                const int vo = (voff >= p0 * 16 && voff < (p0 + POLL_WIN) * 16) ? voff : 0x7ffffff0;
                const u32x4 vb = __builtin_amdgcn_raw_buffer_load_b128(rW, vo, OFF_FLAGS + (16 + round * 16 + cq) * 1024, POLL_AUX);
                const u32x4 va = __builtin_amdgcn_raw_buffer_load_b128(rW, round == 0 ? vo : 0x7ffffff0, OFF_FLAGS + rb * 1024, POLL_AUX);
                WAIT_V(0);
                c1 = 0; c2 = 0; c3 = 0;
                const unsigned mb = min(min(vb[0], vb[1]), min(vb[2], vb[3])), ma = round == 0 ? min(min(va[0], va[1]), min(va[2], va[3])) : call;
                const unsigned long long bad = __ballot(min(ma, mb) < call && voff >= p0 * 16);
                const int cnt = bad ? __builtin_ctzll(bad) : 64;
                const int r = round * 32 + (cnt >> 1);
                if (r > ready) lds_word[0] = (unsigned)r;
                ready = r > ready ? r : ready;
            } else {
                ready = __builtin_amdgcn_readfirstlane(lds_word[0]);
            }
            if (ready > need) break;
            service_blocking();
            if ((++spins & 63u) == 0u) {
                const unsigned waited = now_ticks() - w0;
                bool dead = __builtin_amdgcn_readfirstlane(lds_word[1]) != 0u || waited > 2u * DEADLINE_TICKS;
                if (wid == 0) {
                    const unsigned tv = __builtin_amdgcn_readfirstlane(__builtin_amdgcn_raw_buffer_load_b32(rW, 0, OFF_TMO, AUX_SYS));
                    dead = dead || waited > DEADLINE_TICKS || tv == call;
                    if (dead) { __builtin_amdgcn_raw_buffer_store_b32(call, rW, 0, OFF_TMO, AUX_SYS); lds_word[1] = 1u; }
                }
                failed = failed || dead;
                ready = dead ? (1 << 20) : ready;
                if (dead) break;
            }
            __builtin_amdgcn_s_sleep(POLL_SLEEP);
        }
    };
#define ENSURE(need) do { if (__builtin_expect(ready <= (need), 0)) ensure_slow(need); } while (0)

    const __amdgpu_buffer_rsrc_t rS = __builtin_amdgcn_make_buffer_rsrc(my_ws + OFF_STAGE + par_off, 0, 8u * REGION_BYTES - SHARD_BYTES, 0x00020000);
    const __amdgpu_buffer_rsrc_t rC = __builtin_amdgcn_make_buffer_rsrc((void*)P.out, 0, (unsigned)((size_t)M * N * 4), 0x00020000);
    const int a_base = rb * 32768 + ((wid >> 1) & 1) * 16384 + (wid >> 2) * 8192 + (wid & 1) * 2048;
    const int b_base = (2 << 20) + cq * 32768 + wid * 2048;
    int src_tab;
    {
        const int s = lane & 31, u = s - 4, c = s < 4 ? s : (u * 37) >> 8;
        src_tab = (s < 4 ? 0 : (u - c * 7 + 1) * (int)REGION_BYTES) + (c << 19);
    }
    auto src_off = [&](int g) { return __builtin_amdgcn_readlane(src_tab, g & 31 & EXP_GMASK); };
#define DMA2(D, SO) do { __builtin_amdgcn_raw_ptr_buffer_load_lds(rS, D, 16, voff, SO, 0, DMA_AUX); __builtin_amdgcn_raw_ptr_buffer_load_lds(rS, D, 16, voff, SO, 1024, DMA_AUX); } while (0)
    auto issueB = [&](int so, int buf) { LAS char* d = lds + buf * 65536 + wid * 2048; DMA2(d, so); DMA2(d + 16384, so + 16384); };
    auto issueAlo = [&](int so, int buf) { LAS char* d = lds + buf * 65536 + 32768 + wid * 2048; DMA2(d, so); };
    auto issueAhi = [&](int so, int buf) { LAS char* d = lds + buf * 65536 + 49152 + wid * 2048; DMA2(d, so + 4096); };
    i32x4 acc[8][4];
#pragma unroll
    for (int m = 0; m < 8; ++m)
#pragma unroll
        for (int t = 0; t < 4; ++t) acc[m][t] = (i32x4){0, 0, 0, 0};
    i32x4 af[4][2], bf[4][2];
    int rdA = 32768 + wr * 8192 + lane * 16;
    int rdB = wc * 4096 + lane * 16;
    constexpr int TOT = 128;

    ENSURE(1);
    STMP();
    int oA = a_base + (1 << 19);
    issueB(b_base, 0); issueAlo(a_base, 0); issueAhi(a_base, 0); issueB(b_base + (1 << 19), 1); issueAlo(oA, 1);
    WAIT_V(8);
    c1 = 0; c2 = 0; c3 = 0;
    BAR();
    if (wr == 1) BAR();

#define MMA(MB) do { __builtin_amdgcn_s_setprio(1); \
    _Pragma("unroll") for (int m = 0; m < 4; ++m) _Pragma("unroll") for (int t_ = 0; t_ < 4; ++t_) { const int t = (m & 1) ? 3 - t_ : t_; \
      _Pragma("unroll") for (int k_ = 0; k_ < 2; ++k_) { const int kk = (t_ & 1) ? 1 - k_ : k_; \
        acc[MB + m][t] = __builtin_amdgcn_mfma_i32_16x16x64_i8(af[m][kk], bf[t][kk], acc[MB + m][t], 0, 0, 0); SCHED(); } } \
    __builtin_amdgcn_s_setprio(0); } while (0)
#define SEG_END(seg) do { const int attn_ = attn; if (__builtin_expect((seg) >= attn_, 0)) seg_slow(seg); \
        asm volatile("s_cmp_ge_i32 %0, %1\n\ts_cbranch_scc1 .Lsk%=\n\ts_waitcnt vmcnt(8)\n.Lsk%=:" ::"s"(seg), "s"(attn_) : "scc", "memory"); \
        WAIT_L0(); SCHED(); BAR(); SCHED(); } while (0)

    int buf = 0;
    const int hi1 = wr == 1 ? 0 : 1 << 24;
    for (int round = 0; round < 4; ++round) {
        for (int s = 0; s < 32; ++s) {
            const int g = round * 32 + s;
            const int g2 = g + 2 < TOT ? g + 2 : TOT - 1;
#pragma unroll
            for (int t = 0; t < 4; ++t)
#pragma unroll
                for (int kk = 0; kk < 2; ++kk) bf[t][kk] = *(LAS i32x4*)(lds + rdB + kk * 16384 + t * 1024);
#pragma unroll
            for (int m = 0; m < 4; ++m)
#pragma unroll
                for (int kk = 0; kk < 2; ++kk) af[m][kk] = *(LAS i32x4*)(lds + rdA + kk * 4096 + m * 1024);
            issueAhi(oA, buf ^ 1);
            const int so2 = src_off(g2);
            SEG_END(2 * g);
            MMA(0);
            SCHED();
            if (__builtin_expect(g2 >= (ready | hi1), 0)) ensure_slow(g2);
            BAR();
            SCHED();
            ENSURE(g2);
#pragma unroll
            for (int m = 0; m < 4; ++m)
#pragma unroll
                for (int kk = 0; kk < 2; ++kk) af[m][kk] = *(LAS i32x4*)(lds + rdA + 16384 + kk * 4096 + m * 1024);
            oA = a_base + so2;
            issueB(b_base + so2 + (((g2 & EXP_GMASK) >> 5) << 21), buf); issueAlo(oA, buf);
            SEG_END(2 * g + 1);
            MMA(4);
            SCHED();
            BAR();
            SCHED();
            buf ^= 1; rdA ^= 65536; rdB ^= 65536;
        }
        STMP();
        {
            const int cb = round * 16 + cq;
            int vz = voff; asm volatile("" : "+v"(vz));
            const unsigned vo = (unsigned)((vz >> 8) * 4 * N * 4 + (vz & 255));
            const unsigned so0 = (unsigned)((rb * 256 + wr * 128) * N + cb * 256 + wc * 64) * 4u;
            const float sc = failed ? __builtin_nanf("") : scale;
#pragma unroll
            for (int m = 0; m < 8; ++m)
#pragma unroll
                for (int r = 0; r < 4; ++r) {
                    f32x4 o;
#pragma unroll
                    for (int t = 0; t < 4; ++t) o[t] = __builtin_elementwise_maximum((float)acc[m][t][r] * sc, 0.f);
                    store16<AUX_NT>(__builtin_bit_cast(u32x4, o), rC, (int)vo, so0 + (unsigned)((m * 16 + r) * N * 4));
                }
#pragma unroll
            for (int m = 0; m < 8; ++m)
#pragma unroll
                for (int t = 0; t < 4; ++t) acc[m][t] = (i32x4){0, 0, 0, 0};
            c1 += 32;
            attn = 0;
        }
        STMP();
    }
    while (jstate != 5) { service_blocking(); __builtin_amdgcn_s_sleep(4); }
    WAIT_V(0);
    if (wr == 0) BAR();
    if (wid == 0) __builtin_amdgcn_raw_buffer_store_b32(call, rW, 0, OFF_CNT + b * 128, AUX_SYS);
}

}

extern "C" void kernel_launch(int rank, int nranks, ncclComm_t comm,
                              hipStream_t stream,
                              void* const* d_in, const int* in_sizes, int n_in,
                              void* d_out, int out_size,
                              const int* in_dtypes, int out_dtype,
                              void* const* d_out_all,
                              void* const* d_ws_all, size_t ws_bytes) {
    (void)comm; (void)in_dtypes; (void)out_dtype; (void)d_out_all;
    if (nranks != 8 || n_in != 4 || ws_bytes < WS_NEED || out_size != M * N || in_sizes[0] != M * KSH || in_sizes[1] != KSH * N) {
        fprintf(stderr, "unexpected problem shape\n");
        return;
    }
    static bool attr_set = false;
    if (!attr_set) {
        HIP_CHECK(hipFuncSetAttribute((const void*)fused_kernel, hipFuncAttributeMaxDynamicSharedMemorySize, LDS_BYTES));
        attr_set = true;
    }
    Params p;
    p.x = (const signed char*)d_in[0];
    p.w = (const signed char*)d_in[1];
    p.sx = (const float*)d_in[2];
    p.sw = (const float*)d_in[3];
    p.out = (float*)d_out;
    p.rank = rank;
    for (int q = 0; q < 8; ++q) p.ws[q] = (char*)d_ws_all[(rank + q) & 7];
    hipLaunchKernelGGL(fused_kernel, dim3(NWG), dim3(512), LDS_BYTES, stream, p);
    HIP_CHECK(hipGetLastError());
}
```

```cpp
#include <hip/hip_runtime.h>
#include <rccl/rccl.h>
#include <cstdio>

#define HIP_CHECK(expr) do { hipError_t _e = (expr); if (_e != hipSuccess) { \
    fprintf(stderr, "HIP error %s at %s:%d\n", hipGetErrorString(_e), __FILE__, __LINE__); \
    return; } } while (0)

namespace {

typedef int i32x4 __attribute__((ext_vector_type(4)));
typedef unsigned u32x4 __attribute__((ext_vector_type(4)));
typedef float f32x4 __attribute__((ext_vector_type(4)));
typedef __amdgpu_buffer_rsrc_t rsrc_t;
#define LAS __attribute__((address_space(3)))
#define GAS __attribute__((address_space(1)))

constexpr int M = 4096, N = 16384, KSH = 512;
constexpr int NWG = 256;
constexpr int TOT = 128;
constexpr unsigned OFF_CNT = 0;
constexpr unsigned OFF_TMO = 32768;
constexpr unsigned OFF_FLAG = 65536;
constexpr unsigned FLAG_A = 32768;
constexpr unsigned FLAG_BYTES = 32768 + 2048;
constexpr unsigned OFF_DBG = 1u << 20;
constexpr unsigned OFF_STAGE = 4u << 20;
constexpr unsigned SHARD_BYTES = 10u << 20;
constexpr unsigned REGION_BYTES = 2u * SHARD_BYTES;
constexpr unsigned WS_NEED = OFF_STAGE + 8u * REGION_BYTES;
constexpr int LBUF = 69632;
constexpr int LDS_BYTES = 2 * LBUF + 64;
constexpr unsigned DEADLINE_TICKS = 30000000u;
constexpr int FAR = 0x40000000;
constexpr int OOB = 0x7ffffff0;

#define STAMP 0
#define EXP_NOWAIT 0
#define EXP_NOPUSH 0
#define EXP_NOGEMM 0
#define POLL_SLEEP 4
#define AUX_SYS 17

struct Params {
    const signed char* x;
    const signed char* w;
    const float* sx;
    const float* sw;
    float* out;
    char* ws[8];
};

#define WAIT_V(n) asm volatile("s_waitcnt vmcnt(%0)" ::"n"(n) : "memory")
#define WAIT_L0() __builtin_amdgcn_s_waitcnt(0xC07F)
#define BAR() __builtin_amdgcn_s_barrier()
#define SCHED() __builtin_amdgcn_sched_barrier(0)

__device__ __forceinline__ rsrc_t mk_rsrc(const void* p, unsigned bytes) { return __builtin_amdgcn_make_buffer_rsrc((void*)p, 0, bytes, 0x00020000); }
__device__ __forceinline__ unsigned now_ticks() { return (unsigned)__builtin_amdgcn_s_memrealtime(); }

__device__ __forceinline__ void store16_nt(f32x4 v, rsrc_t r, int vo, unsigned so) {
    asm volatile("buffer_store_dwordx4 %0, %1, %2, %3 offen nt\n\ts_nop 1" ::"v"(v), "v"(vo), "s"(r), "s"(so) : "memory");
}
__device__ __forceinline__ void store4k(i32x4 a, i32x4 b, i32x4 c, i32x4 d, rsrc_t r, int vo, unsigned so) {
    asm volatile("buffer_store_dwordx4 %0, %4, %5, %6 offen sc0 sc1\n\t"
                 "buffer_store_dwordx4 %1, %4, %5, %6 offen offset:1024 sc0 sc1\n\t"
                 "buffer_store_dwordx4 %2, %4, %5, %6 offen offset:2048 sc0 sc1\n\t"
                 "buffer_store_dwordx4 %3, %4, %5, %6 offen offset:3072 sc0 sc1\n\t"
                 "s_nop 1"
                 ::"v"(a), "v"(b), "v"(c), "v"(d), "v"(vo), "s"(r), "s"(so) : "memory");
}

__device__ __forceinline__ bool wait_expired(rsrc_t rW, unsigned call, unsigned w0) {
    const unsigned tv = __builtin_amdgcn_readfirstlane(__builtin_amdgcn_raw_buffer_load_b32(rW, 0, OFF_TMO, AUX_SYS));
    const bool dead = tv == call || now_ticks() - w0 > DEADLINE_TICKS;
    if (dead) __builtin_amdgcn_raw_buffer_store_b32(call, rW, 0, OFF_TMO, AUX_SYS);
    return dead;
}

struct Unit { bool isA; unsigned src, dst; int flag, fstride; };

__device__ __forceinline__ void comm_role(const Params& P, rsrc_t rW, int ci, int wid, int lane, unsigned call, unsigned t0) {
    const unsigned par_off = (call & 1u) * SHARD_BYTES;
    rsrc_t rD[7];
    const int rot = (ci + wid) % 7;
#pragma unroll
    for (int k = 0; k < 7; ++k) {
        int q = k + rot; q = (q >= 7 ? q - 7 : q) + 1;
        rD[k] = mk_rsrc(P.ws[q] + OFF_STAGE + (unsigned)(8 - q) * REGION_BYTES + par_off, EXP_NOPUSH ? 0u : SHARD_BYTES);
    }
    const int ql = (lane & 7) == 7 ? 1 : (lane & 7) + 1;
    char* const flag_base = P.ws[ql] + OFF_FLAG;
    const rsrc_t rL = mk_rsrc(P.ws[0] + OFF_STAGE + par_off, SHARD_BYTES);
    const rsrc_t rX = mk_rsrc(P.x, M * KSH);
    const int voff = lane * 16 + wid * 1024;
    const int vx = ((lane >> 3) + wid * 8) * KSH + (((lane & 7) ^ (lane >> 4) ^ ((wid & 1) << 2)) << 4);
    constexpr int NU = 80;

    auto desc = [&](int i) {
        Unit u;
        const int R = i >> 2, k = i & 3;
        const int un = k * 16 + ci;
        const bool p1 = R < 8;
        const int c = p1 ? R >> 1 : R & 3, j = p1 ? 0 : (R >> 2) - 1;
        u.isA = p1 && !(R & 1);
        u.dst = (u.isA ? 0u : (unsigned)(j + 1) << 21) + ((unsigned)c << 19) + ((unsigned)un << 13);
        u.src = u.isA ? (unsigned)un * (64 * KSH) + c * 128 : u.dst;
        u.fstride = u.isA ? 64 : 16;
        u.flag = (u.isA ? (int)FLAG_A + ci * 4 : (j * 16 + (un >> 2)) * 512 + (un & 3) * 4) + (11 + 7 * c) * u.fstride;
        if (u.isA && k != 3) u.flag = -1;
        return u;
    };
    auto send_flags = [&](const Unit& u) {
        if (lane < 7)
            __hip_atomic_store((GAS unsigned*)(flag_base + (u.flag - ql * u.fstride)), call, __ATOMIC_RELAXED, __HIP_MEMORY_SCOPE_SYSTEM);
    };
#define LOADU(D, U) do { const rsrc_t rs = (U).isA ? rX : rL; const int vs = (U).isA ? vx : voff; \
        asm volatile("buffer_load_dwordx4 %0, %1, %2, %3 offen sc1" : "=v"(D) : "v"(vs), "s"(rs), "s"((U).src) : "memory"); } while (0)
#define ST1(D, k, SD) asm volatile("buffer_store_dwordx4 %0, %1, %2, %3 offen sc0 sc1" ::"v"(D), "v"(voff), "s"(rD[k]), "s"(SD) : "memory")

    bool dead = false, sent = false;
    auto wait_packed = [&]() {
        const unsigned w0 = now_ticks();
        for (unsigned spins = 1;; ++spins) {
            unsigned mn = 0xffffffffu;
#pragma unroll
            for (int s = 0; s < 4; ++s) {
                const u32x4 f = __builtin_amdgcn_raw_buffer_load_b128(rW, lane * 512 + s * 16, OFF_FLAG, AUX_SYS);
                mn = min(mn, min(min(f[0], f[1]), min(f[2], f[3])));
            }
            if (__ballot(mn < call) == 0ull) break;
            if ((spins & 63u) == 0u && wait_expired(rW, call, w0)) { WAIT_V(0); break; }
            __builtin_amdgcn_s_sleep(8);
        }
    };
#define CST(i) do {} while (0)

    i32x4 X, Y;
    Unit prev = desc(0), cur = prev, nx1 = desc(1);
    prev.flag = -1;
    LOADU(X, cur); LOADU(Y, nx1);
    asm volatile("s_waitcnt vmcnt(0)" : "+v"(X), "+v"(Y) :: "memory");
#define STEP(D, O, i) do { \
        Unit nx2 = desc((i) + 2 < NU ? (i) + 2 : (i)); \
        ST1(D, 0, cur.dst); ST1(D, 1, cur.dst); ST1(D, 2, cur.dst); ST1(D, 3, cur.dst); ST1(D, 4, cur.dst); ST1(D, 5, cur.dst); ST1(D, 6, cur.dst); \
        asm volatile("s_nop 1" ::"v"(D)); \
        if ((i) == 2) wait_packed(); \
        LOADU(D, nx2); \
        if (!sent) WAIT_V(8); \
        asm volatile("s_waitcnt vmcnt(9)" : "+v"(O) :: "memory"); \
        BAR(); \
        if ((i) == 2) dead = __builtin_amdgcn_readfirstlane(__builtin_amdgcn_raw_buffer_load_b32(rW, 0, OFF_TMO, AUX_SYS)) == call; \
        sent = wid == ((i) & 7) && !dead && prev.flag >= 0; \
        if (sent) send_flags(prev); \
        CST(i); \
        prev = cur; cur = nx1; nx1 = nx2; \
    } while (0)
    for (int i = 0; i < NU; i += 2) {
        STEP(X, Y, i);
        STEP(Y, X, i + 1);
    }
    asm volatile("s_waitcnt vmcnt(0)" : "+v"(X), "+v"(Y) :: "memory");
    BAR();
    if (wid == 0 && !dead) send_flags(prev);
    (void)t0;
}

template <int GRP>
__device__ __forceinline__ void gemm_role(const Params& P, rsrc_t rW, LAS char* lds, int wid, int lane, unsigned call,
                                          int ri, int cq, int b, unsigned t0) {
    volatile LAS unsigned* lds_fail = (volatile LAS unsigned*)(lds + 2 * LBUF);
    const unsigned par_off = (call & 1u) * SHARD_BYTES;
    const int wc = wid & 3;
    const int voff = lane * 16;
#define STMP() do {} while (0)
    STMP();
    const float scale = P.sx[0] * P.sw[0];

    int shift_m = 0, spec_m = 0;
#pragma unroll
    for (int j = 0; j < 4; ++j) { const int is = (4 * cq + j) % 15; shift_m |= (ri > is ? 1 : 0) << j; spec_m |= (ri == is ? 1 : 0) << j; }

    int ready = (wid == 4 && !EXP_NOWAIT) ? 0 : 1 << 20;
    auto ensure_slow = [&](int need) {
        const rsrc_t rF = mk_rsrc(P.ws[0] + OFF_FLAG, FLAG_BYTES);
        const unsigned w0 = now_ticks();
        const int round = need >> 5, s0 = need & 31;
        int vz = voff; asm volatile("" : "+v"(vz));
        for (unsigned spins = 1;; ++spins) {
            const u32x4 fb = __builtin_amdgcn_raw_buffer_load_b128(rF, (vz < 512 && vz >= s0 * 16) ? vz : OOB, (round * 16 + cq) * 512, AUX_SYS);
            const u32x4 fa = __builtin_amdgcn_raw_buffer_load_b128(rF, round == 0 ? vz : OOB, FLAG_A + s0 * 64, AUX_SYS);
            const unsigned mb = min(min(fb[0], fb[1]), min(fb[2], fb[3])), ma = min(min(fa[0], fa[1]), min(fa[2], fa[3]));
            const unsigned long long badb = __ballot(mb < call && vz >= s0 * 16);
            const unsigned long long bada = __ballot(ma < call);
            const int rb_ = __builtin_ctzll(badb);
            const int ra_ = round == 0 ? s0 + (bada ? __builtin_ctzll(bada) >> 2 : 16) : 32;
            ready = round * 32 + (rb_ < ra_ ? rb_ : ra_);
            ready = ready == TOT ? 1 << 20 : ready;
            if (ready > need) break;
            if ((spins & 63u) == 0u && wait_expired(rW, call, w0)) { lds_fail[0] = 1u; ready = 1 << 20; break; }
            __builtin_amdgcn_s_sleep(POLL_SLEEP);
        }
    };
#define ENSURE(need) do { if (GRP == 1 && __builtin_expect(ready <= (need), 0)) ensure_slow(need); } while (0)

    const rsrc_t rS = mk_rsrc(P.ws[0] + OFF_STAGE + par_off, 8u * REGION_BYTES - SHARD_BYTES);
    const rsrc_t rC = mk_rsrc(P.out, (unsigned)((size_t)M * N * 4));
    const int a_base = (17 * ri + GRP * 8 + wc) * 2048;
    const int x_delta = (16 - wc) * 2048 + wc * 1024;
    const int b_base = (2 << 20) + cq * 32768 + wid * 4096;
    int src_tab;
    {
        const int s = lane & 31, u = s - 4, c = s < 4 ? s : (u * 37) >> 8;
        src_tab = (s < 4 ? 0 : (u - c * 7 + 1) * (int)REGION_BYTES) + (c << 19);
    }
    if (lane >= 32) {
        const int j = lane & 7, kind = (lane >> 3) & 3;
        const int aj = a_base + (((shift_m >> j) & 1) << 11);
        const bool xok = wc < 2 || ((spec_m >> j) & 1);
        src_tab = j >= 4 ? FAR : kind == 0 ? aj : kind == 1 ? b_base + (j << 21) : xok ? aj + x_delta : FAR;
    }
#define DMA(D, SO, IMM) __builtin_amdgcn_raw_ptr_buffer_load_lds(rS, D, 16, voff, SO, IMM, AUX_SYS)
#define ISSUE_B(so, buf) do { LAS char* d_ = lds + (buf) * LBUF + wid * 4096; DMA(d_, so, 0); DMA(d_, so, 1024); DMA(d_, so, 2048); DMA(d_, so, 3072); } while (0)
#define ISSUE_ALO(so, buf) do { LAS char* d_ = lds + (buf) * LBUF + 32768 + wid * 2048; DMA(d_, so, 0); DMA(d_, so, 1024); } while (0)
#define ISSUE_AHI(so, sx, buf) do { LAS char* d_ = lds + (buf) * LBUF + 49152 + wid * 2048; DMA(d_, so, 0); DMA(d_, so, 1024); \
        if (GRP == 0) { LAS char* e_ = lds + (buf) * LBUF + 65536 + wid * 1024; DMA(e_, sx, 0); } } while (0)
    constexpr int NV = GRP == 0 ? 9 : 8;
    constexpr int NVR = GRP == 0 ? 45 : 40;

    i32x4 acc[9][4];
    i32x4 af[4][2], ax[2], bf[4][2];
    const int rdA0 = 32768 + GRP * 8192 + (lane & 15) * 128 + (((lane >> 4) ^ ((lane >> 1) & 7)) << 4);
    const int rdA[2] = {rdA0, rdA0 ^ 64};
    const int rdB = wc * 4096 + lane * 16;

    ENSURE(1);
    BAR();
    STMP();
    int aj = __builtin_amdgcn_readlane(src_tab, 32), bj = b_base, xj = __builtin_amdgcn_readlane(src_tab, 48);
    ISSUE_B(bj, 0); ISSUE_ALO(aj, 0); ISSUE_AHI(aj + 8192, xj, 0);
    ISSUE_B(bj + (1 << 19), 1); ISSUE_ALO(aj + (1 << 19), 1);
    int oA1 = aj + (1 << 19) + 8192, oX1 = xj + (1 << 19);
    WAIT_V(NV);
    BAR();
    if (GRP == 1) BAR();

#define MP(ROW, A_, t, k0, Z) do { \
        if (Z) asm volatile("v_mfma_i32_16x16x64_i8 %0, %1, %2, 0\n\tv_mfma_i32_16x16x64_i8 %0, %3, %4, %0" : "=&v"(acc[ROW][t]) : "v"(A_[k0]), "v"(bf[t][k0]), "v"(A_[1 - (k0)]), "v"(bf[t][1 - (k0)])); \
        else asm volatile("v_mfma_i32_16x16x64_i8 %0, %1, %2, %0\n\tv_mfma_i32_16x16x64_i8 %0, %3, %4, %0" : "+v"(acc[ROW][t]) : "v"(A_[k0]), "v"(bf[t][k0]), "v"(A_[1 - (k0)]), "v"(bf[t][1 - (k0)])); \
        SCHED(); } while (0)
#define MROW_UP(ROW, A_, Z) do { MP(ROW, A_, 0, 0, Z); MP(ROW, A_, 1, 1, Z); MP(ROW, A_, 2, 0, Z); MP(ROW, A_, 3, 1, Z); } while (0)
#define MROW_DN(ROW, A_, Z) do { MP(ROW, A_, 3, 0, Z); MP(ROW, A_, 2, 1, Z); MP(ROW, A_, 1, 0, Z); MP(ROW, A_, 0, 1, Z); } while (0)
#define PIN(x) do { asm volatile("" : "+s"(x)); SCHED(); } while (0)
#define SEG_WAIT(FIRST) do { if (FIRST) { if (round == 0) WAIT_V(NV); WAIT_V(NVR); } else WAIT_V(NV); WAIT_L0(); SCHED(); BAR(); SCHED(); } while (0)
#define STAGE(buf, FIRST) do { \
        LAS char* sb = lds + (buf) * LBUF; \
        _Pragma("unroll") for (int t = 0; t < 4; ++t) _Pragma("unroll") for (int kk = 0; kk < 2; ++kk) bf[t][kk] = *(LAS i32x4*)(sb + rdB + kk * 16384 + t * 1024); \
        _Pragma("unroll") for (int m = 0; m < 4; ++m) _Pragma("unroll") for (int kk = 0; kk < 2; ++kk) af[m][kk] = *(LAS i32x4*)(sb + rdA[kk] + m * 2048); \
        ISSUE_AHI(oA1, oX1, (buf) ^ 1); \
        ENSURE(g2); \
        SEG_WAIT(FIRST); \
        __builtin_amdgcn_s_setprio(1); \
        MP(0, af[0], 0, 0, FIRST); PIN(g2); int s2 = g2 & 31; PIN(s2); \
        MP(0, af[0], 1, 1, FIRST); int so2 = __builtin_amdgcn_readlane(src_tab, s2); PIN(so2); \
        MP(0, af[0], 2, 0, FIRST); int oA2 = aj + so2; PIN(oA2); int oB2 = bj + so2; PIN(oB2); \
        MP(0, af[0], 3, 1, FIRST); oA1 = oA2 + 8192; PIN(oA1); if (GRP == 0) { oX1 = xj + so2; PIN(oX1); } \
        MP(1, af[1], 3, 0, FIRST); g2 += 1; PIN(g2); \
        MP(1, af[1], 2, 1, FIRST); MP(1, af[1], 1, 0, FIRST); MP(1, af[1], 0, 1, FIRST); \
        MROW_UP(2, af[2], FIRST); MROW_DN(3, af[3], FIRST); \
        __builtin_amdgcn_s_setprio(0); \
        SCHED(); BAR(); SCHED(); \
        _Pragma("unroll") for (int m = 0; m < 4; ++m) _Pragma("unroll") for (int kk = 0; kk < 2; ++kk) af[m][kk] = *(LAS i32x4*)(sb + rdA[kk] + 16384 + m * 2048); \
        if (GRP == 0 || __builtin_expect(special, 0)) { _Pragma("unroll") for (int kk = 0; kk < 2; ++kk) ax[kk] = *(LAS i32x4*)(sb + rdA[kk] + 32768 - GRP * 6144); } \
        ISSUE_B(oB2, buf); ISSUE_ALO(oA2, buf); \
        SEG_WAIT(FIRST); \
        __builtin_amdgcn_s_setprio(1); \
        if ((buf) == 1) { \
            MP(4, af[0], 0, 0, FIRST); PIN(g2); int j2 = (g2 >> 5) + 32; PIN(j2); \
            MP(4, af[0], 1, 1, FIRST); aj = __builtin_amdgcn_readlane(src_tab, j2); PIN(aj); j2 += 8; PIN(j2); \
            MP(4, af[0], 2, 0, FIRST); bj = __builtin_amdgcn_readlane(src_tab, j2); PIN(bj); j2 += 8; PIN(j2); \
            MP(4, af[0], 3, 1, FIRST); if (GRP == 0) { xj = __builtin_amdgcn_readlane(src_tab, j2); PIN(xj); } \
        } else { MROW_UP(4, af[0], FIRST); } \
        MROW_DN(5, af[1], FIRST); MROW_UP(6, af[2], FIRST); \
        if (GRP == 0) { MROW_DN(8, ax, FIRST); MROW_UP(7, af[3], FIRST); } \
        else { if (__builtin_expect(special, 0)) MROW_DN(8, ax, FIRST); MROW_DN(7, af[3], FIRST); } \
        __builtin_amdgcn_s_setprio(0); \
        SCHED(); BAR(); SCHED(); \
    } while (0)

    int g2 = 2;
    if (GRP == 1) {
#pragma unroll
        for (int t = 0; t < 4; ++t) acc[8][t] = (i32x4){0, 0, 0, 0};
        ax[0] = ax[1] = (i32x4){0, 0, 0, 0};
    }
    for (int round = 0; round < 4; ++round) {
        const bool special = (spec_m >> round) & 1;
        STAGE(0, true);
        STAGE(1, false);
        for (int s = 2; s < 32; s += 2) {
            STAGE(0, false);
            STAGE(1, false);
        }
        STMP();
        {
            int vz = voff; asm volatile("s_nop 7\n\ts_nop 7" : "+v"(vz)); SCHED();
            const int vo = (vz >> 8) * 4 * N * 4 + (vz & 255);
            const int r0 = 17 * ri + ((shift_m >> round) & 1);
            const unsigned so0 = (unsigned)(r0 * 16 * N + (round * 16 + cq) * 256 + wc * 64) * 4u;
            const float sc = __builtin_amdgcn_readfirstlane(lds_fail[0]) ? __builtin_nanf("") : scale;
#define EPI_ROW(m, rg) do { _Pragma("unroll") for (int r = 0; r < 4; ++r) { f32x4 o; \
                _Pragma("unroll") for (int t = 0; t < 4; ++t) o[t] = __builtin_elementwise_maximum((float)acc[m][t][r] * sc, 0.f);     \
                store16_nt(o, rC, vo, so0 + (unsigned)(((rg) * 16 + r) * N * 4)); } } while (0)
#pragma unroll
            for (int m = 0; m < 8; ++m) EPI_ROW(m, GRP * 8 + m);
            if (GRP == 0 || special) EPI_ROW(8, 16 + GRP);
        }
        STMP();
    }
    WAIT_V(0);
    if (GRP == 0) BAR();
    (void)b; (void)t0;
}

__global__ __launch_bounds__(512, 2) void fused_kernel(Params P) {
    extern __shared__ __attribute__((aligned(16))) char smem_[];
    LAS char* lds = (LAS char*)smem_;
    const unsigned t0 = STAMP ? now_ticks() : 0u;
    const int tid = threadIdx.x, lane = tid & 63;
    const int wid = __builtin_amdgcn_readfirstlane(tid >> 6);
    const int b = blockIdx.x;
    const int xcd = b & 7, idx = b >> 3;
    const rsrc_t rW = mk_rsrc(P.ws[0], WS_NEED);
    const unsigned call_v = __builtin_amdgcn_raw_buffer_load_b32(rW, 0, OFF_CNT + b * 128, AUX_SYS);

    if (idx < 2) {
        const unsigned call = __builtin_amdgcn_readfirstlane(call_v) + 1u;
        comm_role(P, rW, b, wid, lane, call, t0);
        WAIT_V(0);
        BAR();
        if (wid == 0) __builtin_amdgcn_raw_buffer_store_b32(call, rW, 0, OFF_CNT + b * 128, AUX_SYS);
        return;
    }

    const int n = wid * 240 + (b - 16);
    unsigned call;
    if (n < 1024) {
        const int j = n >> 8, c = (n >> 6) & 3, cqp = (n >> 2) & 15, part = n & 3;
        const rsrc_t rWm = mk_rsrc(P.w, KSH * N);
        const int vw = (lane >> 4) * 16 * N + (lane & 15) * 4;
        const unsigned sw0 = (unsigned)((c * 128 + (part >> 1) * 64) * N + (j * 16 + cqp) * 256 + (part & 1) * 128);
        unsigned d[2][16];
#pragma unroll
        for (int i = 0; i < 16; ++i) {
            d[0][i] = __builtin_amdgcn_raw_buffer_load_b32(rWm, vw, sw0 + i * N, 0);
            d[1][i] = __builtin_amdgcn_raw_buffer_load_b32(rWm, vw + 64, sw0 + i * N, 0);
        }
        call = __builtin_amdgcn_readfirstlane(call_v) + 1u;
        const unsigned so = OFF_STAGE + (call & 1u) * SHARD_BYTES + (2u << 20) + (unsigned)n * 8192u;
#pragma unroll
        for (int h = 0; h < 2; ++h) {
            i32x4 pb[4];
#pragma unroll
            for (int dd = 0; dd < 4; ++dd) {
                const unsigned d0 = d[h][4 * dd], d1 = d[h][4 * dd + 1], d2 = d[h][4 * dd + 2], d3 = d[h][4 * dd + 3];
                const unsigned lo01 = __builtin_amdgcn_perm(d1, d0, 0x05010400u), hi01 = __builtin_amdgcn_perm(d1, d0, 0x07030602u);
                const unsigned lo23 = __builtin_amdgcn_perm(d3, d2, 0x05010400u), hi23 = __builtin_amdgcn_perm(d3, d2, 0x07030602u);
                pb[0][dd] = (int)__builtin_amdgcn_perm(lo23, lo01, 0x05040100u);
                pb[1][dd] = (int)__builtin_amdgcn_perm(lo23, lo01, 0x07060302u);
                pb[2][dd] = (int)__builtin_amdgcn_perm(hi23, hi01, 0x05040100u);
                pb[3][dd] = (int)__builtin_amdgcn_perm(hi23, hi01, 0x07060302u);
            }
            store4k(pb[0], pb[1], pb[2], pb[3], rW, lane * 16, so + h * 4096u);
        }
        WAIT_V(0);
        __builtin_amdgcn_raw_buffer_store_b32(call, rW, lane == 0 ? 0 : OOB, OFF_FLAG + (j * 16 + cqp) * 512 + c * 16 + part * 4, AUX_SYS);
    } else if (n >= 1200 && n < 1264) {
        const int c = (n - 1200) >> 4, blk = (n - 1200) & 15;
        const rsrc_t rX = mk_rsrc(P.x, M * KSH);
        const int vx0 = (lane >> 3) * KSH + (((lane & 7) ^ (lane >> 4)) << 4), vx1 = (vx0 ^ 64) + 8 * KSH;
        u32x4 d[32];
#pragma unroll
        for (int i = 0; i < 32; ++i) d[i] = __builtin_amdgcn_raw_buffer_load_b128(rX, (i & 1) ? vx1 : vx0, (blk * 256 + (i >> 1) * 16) * KSH + c * 128, 0);
        call = __builtin_amdgcn_readfirstlane(call_v) + 1u;
        const unsigned so = OFF_STAGE + (call & 1u) * SHARD_BYTES + ((unsigned)c << 19) + (unsigned)blk * 32768u;
#pragma unroll
        for (int i = 0; i < 32; i += 4)
            store4k(__builtin_bit_cast(i32x4, d[i]), __builtin_bit_cast(i32x4, d[i + 1]), __builtin_bit_cast(i32x4, d[i + 2]), __builtin_bit_cast(i32x4, d[i + 3]), rW, lane * 16, so + i * 1024u);
        WAIT_V(0);
        __builtin_amdgcn_raw_buffer_store_b32(call, rW, lane == 0 ? 0 : OOB, OFF_FLAG + FLAG_A + c * 64 + blk * 4, AUX_SYS);
    } else {
        call = __builtin_amdgcn_readfirstlane(call_v) + 1u;
    }
    if (wid == 4) *(volatile LAS unsigned*)(lds + 2 * LBUF) = 0u;

    const int a = xcd >> 1, i30 = idx - 2, half = i30 >= 15 ? 1 : 0, i15 = i30 - half * 15;
    const int t16 = i15 < 4 * a + 3 ? i15 : i15 + 1;
    const int u4 = t16 >> 2;
    const int ri = 4 * a - (a > u4 ? 1 : 0) + (t16 & 3);
    const int cq = (half * 4 + u4) * 2 + (xcd & 1);
    if (!EXP_NOGEMM) {
        if (wid < 4) gemm_role<0>(P, rW, lds, wid, lane, call, ri, cq, b, t0);
        else gemm_role<1>(P, rW, lds, wid, lane, call, ri, cq, b, t0);
    } else {
        BAR();
    }
    if (wid == 0) __builtin_amdgcn_raw_buffer_store_b32(call, rW, 0, OFF_CNT + b * 128, AUX_SYS);
}

}

extern "C" void kernel_launch(int rank, int nranks, ncclComm_t comm,
                              hipStream_t stream,
                              void* const* d_in, const int* in_sizes, int n_in,
                              void* d_out, int out_size,
                              const int* in_dtypes, int out_dtype,
                              void* const* d_out_all,
                              void* const* d_ws_all, size_t ws_bytes) {
    (void)comm; (void)in_dtypes; (void)out_dtype; (void)d_out_all;
    if (nranks != 8 || (unsigned)rank >= 8u || n_in != 4 || ws_bytes < WS_NEED || out_size != M * N || in_sizes[0] != M * KSH || in_sizes[1] != KSH * N) {
        fprintf(stderr, "unexpected problem shape\n");
        return;
    }
    static bool attr_set = false;
    if (!attr_set) {
        HIP_CHECK(hipFuncSetAttribute((const void*)fused_kernel, hipFuncAttributeMaxDynamicSharedMemorySize, LDS_BYTES));
        attr_set = true;
    }
    Params p;
    p.x = (const signed char*)d_in[0];
    p.w = (const signed char*)d_in[1];
    p.sx = (const float*)d_in[2];
    p.sw = (const float*)d_in[3];
    p.out = (float*)d_out;
    for (int q = 0; q < 8; ++q) p.ws[q] = (char*)d_ws_all[(rank + q) & 7];
    hipLaunchKernelGGL(fused_kernel, dim3(NWG), dim3(512), LDS_BYTES, stream, p);
    HIP_CHECK(hipGetLastError());
}
```

```cpp
#include <hip/hip_runtime.h>
#include <rccl/rccl.h>
#include <cstdio>

#define HIP_CHECK(expr) do { hipError_t _e = (expr); if (_e != hipSuccess) { \
    fprintf(stderr, "HIP error %s at %s:%d\n", hipGetErrorString(_e), __FILE__, __LINE__); \
    return; } } while (0)

namespace {

typedef int i32x4 __attribute__((ext_vector_type(4)));
typedef unsigned u32x4 __attribute__((ext_vector_type(4)));
typedef unsigned u32x2 __attribute__((ext_vector_type(2)));
typedef float f32x4 __attribute__((ext_vector_type(4)));
typedef __amdgpu_buffer_rsrc_t rsrc_t;
#define LAS __attribute__((address_space(3)))
#define GAS __attribute__((address_space(1)))

constexpr int M = 4096, N = 16384, KSH = 512;
constexpr unsigned OUT_BYTES = (unsigned)((size_t)M * N * 4);
constexpr int NWG = 256;
constexpr int TOT = 128;
constexpr unsigned OFF_CNT = 0;
constexpr unsigned OFF_TMO = 32768;
constexpr unsigned OFF_TICKET = 32768 + 128;
constexpr unsigned OFF_TILE = 32768 + 1024;
constexpr unsigned OFF_FLAG = 65536;
constexpr unsigned FLAG_DONE = 80 * 512;
constexpr unsigned FLAG_BYTES = FLAG_DONE + 512;
constexpr unsigned OFF_DBG = 1u << 20;
constexpr unsigned OFF_STAGE = 4u << 20;
constexpr unsigned SHARD_BYTES = 10u << 20;
constexpr unsigned REGION_BYTES = 2u * SHARD_BYTES;
constexpr unsigned WS_NEED = OFF_STAGE + 8u * REGION_BYTES;
constexpr int LBUF = 65536;
constexpr int LDS_BYTES = 2 * LBUF + 64;
constexpr unsigned DEADLINE_TICKS = 30000000u;
constexpr int FAR = 0x40000000;
constexpr int OOB = 0x7ffffff0;

#define STAMP 0
#define EXP_NOWAIT 0
#define EXP_NOPUSH 0
#define NSEND 16
#define POLL_SLEEP 4
#define AUX_SYS 17

constexpr unsigned long long SHARED_PACK = 15ull | 30ull << 8 | 31ull << 16 | 47ull << 24 | 63ull << 32;
constexpr unsigned long long SHARED_NEED = 0xffull | 0x03ull << 8 | 0x0eull << 16 | 0x18ull << 24 | 0x70ull << 32 | 0xc0ull << 40;
constexpr int NIN = 1280, NOUT = 320, NUNITS = NIN + NOUT;
#define FIRST_ROWS {0, 0}, {0, 1}, {1, 0}, {1, 1}, {0, 2}, {1, 2}, {0, 3}, {1, 3}
constexpr int row_info(int row) {
    constexpr int first[8][2] = {FIRST_ROWS};
    int n = 0;
    for (int r = 0;; ++r) {
        const int g = r < 8 ? first[r][0] : r / 4, c = r < 8 ? first[r][1] : r % 4;
        if (r == row) return n | g << 16 | c << 20;
        n += g == 0 ? 64 : g == 2 ? 76 : 60;
    }
}
static_assert((row_info(20) & 0xffff) == NIN, "send rows");

struct Params {
    const signed char* x;
    const signed char* w;
    const float* sx;
    const float* sw;
    char* ws0;
    char* out0;
    char* peer[24];
    int rank;
};

#define WAIT_V(n) asm volatile("s_waitcnt vmcnt(%0)" ::"n"(n) : "memory")
#define WAIT_L0() __builtin_amdgcn_s_waitcnt(0xC07F)
#define BAR() __builtin_amdgcn_s_barrier()
#define SCHED() __builtin_amdgcn_sched_barrier(0)

__device__ __forceinline__ rsrc_t mk_rsrc(const void* p, unsigned bytes) { return __builtin_amdgcn_make_buffer_rsrc((void*)p, 0, bytes, 0x00020000); }
__device__ __forceinline__ unsigned now_ticks() { return (unsigned)__builtin_amdgcn_s_memrealtime(); }
__device__ __forceinline__ char* lane_ptr(char* p, int l) {
    const unsigned lo = __builtin_amdgcn_readlane((unsigned)(unsigned long long)p, l), hi = __builtin_amdgcn_readlane((unsigned)((unsigned long long)p >> 32), l);
    return (char*)(((unsigned long long)hi << 32) | lo);
}

template <bool THRU>
__device__ __forceinline__ void store_out(f32x4 v, rsrc_t r, int vo, unsigned so) {
    if (THRU) asm volatile("buffer_store_dwordx4 %0, %1, %2, %3 offen sc0 sc1\n\ts_nop 1" ::"v"(v), "v"(vo), "s"(r), "s"(so) : "memory");
    else asm volatile("buffer_store_dwordx4 %0, %1, %2, %3 offen nt\n\ts_nop 1" ::"v"(v), "v"(vo), "s"(r), "s"(so) : "memory");
}
__device__ __forceinline__ void store4k(i32x4 a, i32x4 b, i32x4 c, i32x4 d, rsrc_t r, int vo, unsigned so) {
    asm volatile("buffer_store_dwordx4 %0, %4, %5, %6 offen sc0 sc1\n\t"
                 "buffer_store_dwordx4 %1, %4, %5, %6 offen offset:1024 sc0 sc1\n\t"
                 "buffer_store_dwordx4 %2, %4, %5, %6 offen offset:2048 sc0 sc1\n\t"
                 "buffer_store_dwordx4 %3, %4, %5, %6 offen offset:3072 sc0 sc1\n\t"
                 "s_nop 1"
                 ::"v"(a), "v"(b), "v"(c), "v"(d), "v"(vo), "s"(r), "s"(so) : "memory");
}

__device__ __forceinline__ bool wait_expired(rsrc_t rW, unsigned call, unsigned w0) {
    const unsigned tv = __builtin_amdgcn_readfirstlane(__builtin_amdgcn_raw_buffer_load_b32(rW, 0, OFF_TMO, AUX_SYS));
    const bool dead = tv == call || now_ticks() - w0 > DEADLINE_TICKS;
    if (dead) __builtin_amdgcn_raw_buffer_store_b32(call, rW, 0, OFF_TMO, AUX_SYS);
    return dead;
}
template <int CNT>
__device__ __forceinline__ bool wait_words(rsrc_t rW, unsigned call, int vo, bool use, unsigned so, int step) {
    const unsigned w0 = now_ticks();
    for (unsigned spins = 1;; ++spins) {
        unsigned mn = 0xffffffffu;
#pragma unroll
        for (int i = 0; i < CNT; ++i) {
            const u32x4 f = __builtin_amdgcn_raw_buffer_load_b128(rW, use ? vo : OOB, so + i * step, AUX_SYS);
            mn = min(mn, min(min(f[0], f[1]), min(f[2], f[3])));
        }
        if (__ballot(use && mn < call) == 0ull) return true;
        if ((spins & 63u) == 0u && wait_expired(rW, call, w0)) return false;
        __builtin_amdgcn_s_sleep(8);
    }
}

__device__ __forceinline__ unsigned stage_off(int e) { return (unsigned)((e >> 8) & 7) << 21 | (unsigned)((e >> 2) & 3) << 19 | (unsigned)((e >> 4) & 15) << 15 | (unsigned)(e & 3) << 13; }

__device__ __forceinline__ void comm_role(const Params& P, rsrc_t rW, LAS char* lds, int ci, int b, int wid, int lane, unsigned call_v, unsigned t0) {
    volatile LAS unsigned* lds_w = (volatile LAS unsigned*)lds;
    const rsrc_t rX = mk_rsrc(P.x, M * KSH);
    const int voff = lane * 16 + wid * 1024, vout = lane * 16 + wid * (N * 4);
    const int vx = ((lane >> 3) + wid * 8) * KSH + (((lane & 7) ^ (lane >> 4) ^ ((wid & 1) << 2)) << 4);
    i32x4 X, Y;
    asm volatile("buffer_load_dwordx4 %0, %2, %3, %4 offen\n\tbuffer_load_dwordx4 %1, %2, %3, %5 offen"
                 : "=&v"(X), "=&v"(Y) : "v"(vx), "s"(rX), "s"(ci * 64 * KSH), "s"((NSEND + ci) * 64 * KSH) : "memory");
    char* const peer = P.peer[lane < 24 ? lane : 0];
    const int rank = P.rank & 7;
    const int ql = (lane & 7) + 1;
    if (wid == 2) lds_w[2] = 0u;
    unsigned long long kneed = 0;
    const int rot = (ci + wid) % 7;
#pragma unroll
    for (int k = 0; k < 7; ++k) {
        int q = k + rot; q = (q >= 7 ? q - 7 : q) + 1;
        kneed |= ((SHARED_NEED >> ((rank + q) & 7)) & 0x010101010101ull) << k;
    }
    const rsrc_t rO = mk_rsrc(P.out0, OUT_BYTES);
    const int vtick = lane == 0 ? 0 : OOB;

    int row_tab = 0xffff;
#pragma unroll
    for (int r = 0; r < 20; ++r) row_tab = lane == r ? row_info(r) : row_tab;
    auto code_of = [&](int n) {
        const int ri_ = __builtin_amdgcn_readlane(row_tab, __builtin_popcountll(__ballot((row_tab & 0xffff) <= n)) - 1);
        int i = n - (ri_ & 0xffff);
        const int g = (ri_ >> 16) & 7, c = ri_ >> 20;
        const bool is_sh = g == 2 && ((i & 3) == 3 || i >= 74);
        const int j = i >= 74 ? i - 56 : i >> 2, p = (j * 205) >> 10, sx = j - 5 * p;
        const int sh = p | c << 2 | (16 + ((int)(SHARED_PACK >> (sx * 8 & 63)) & 255)) << 4 | (sx + 1) << 11;
        if (g == 2) i -= i >> 2;
        return n >= NIN ? 0x4000 + n - NIN : is_sh ? sh : (i & 3) | c << 2 | (g * 16 + (i >> 2)) << 4;
    };
    auto out_off = [&](int e) {
        const int i = e - 0x4000, t = rank * 10 + (i >> 5);
        return (unsigned)(((t & 15) * 256 + (i & 31) * 8) * (N * 4) + (int)((SHARED_PACK >> ((t >> 4) * 8 & 63)) & 255) * 1024);
    };
    asm volatile("" : "+v"(row_tab), "+s"(kneed));
    SCHED();
    rsrc_t rD[7];
#pragma unroll
    for (int k = 0; k < 7; ++k) {
        int q = k + rot; q = q >= 7 ? q - 7 : q;
        rD[k] = mk_rsrc(lane_ptr(peer, 16 + q), EXP_NOPUSH ? 0u : REGION_BYTES);
    }
    const unsigned call = __builtin_amdgcn_readfirstlane(call_v) + 1u;
    const unsigned par_off = (call & 1u) * SHARD_BYTES;
    const unsigned tick_base = (call - 1u) * (unsigned)NUNITS - 4u * NSEND;
    auto send_flags = [&](int e) {
        if (lane < 7 && ((SHARED_NEED >> ((e >> 11) * 8 + ((rank + ql) & 7))) & 1))
            __hip_atomic_store((GAS unsigned*)(peer + (((e >> 4) & 127) * 512 + (11 + 7 * ((e >> 2) & 3) - ql) * 16 + (e & 3) * 4)), call, __ATOMIC_RELAXED, __HIP_MEMORY_SCOPE_SYSTEM);
    };
    auto wait_packed = [&]() { if (!wait_words<4>(rW, call, lane * 512, true, OFF_FLAG + 16 * 512, 16)) lds_w[2] = 1u; };
    auto wait_tiles = [&]() { if (!wait_words<1>(rW, call, lane * 16, lane < 20, OFF_TILE, 0)) lds_w[2] = 1u; };
#define CST() do {} while (0)
#define CSHIFT(nn) do {} while (0)
#define ST1(D, k) asm volatile("buffer_store_dwordx4 %0, %1, %2, %3 offen sc0 sc1" ::"v"(D), "v"(vst), "s"(rD[k]), "s"((km >> (k)) & 1 ? dst : (unsigned)FAR) : "memory")

    int ep = 0x4000, e0 = (ci & 3) | (ci >> 2) << 4, e1 = NSEND == 16 ? e0 + 64 : code_of(NSEND + ci), e2 = NSEND == 16 ? e0 + 128 : code_of(2 * NSEND + ci);
    asm volatile("s_waitcnt vmcnt(0)\n\ts_nop 4" : "+v"(X), "+v"(Y) :: "memory");
    int step = 0, vst = voff;
    unsigned tkv = 0;
    bool sent = false, outs = false, tiles_ok = false;
    constexpr int CHECK_STEP = NSEND == 16 ? 4 : 2;
    const int vdraw = wid == 0 ? vtick : OOB;
#define STEP(D, O) do { \
        if (e0 >= 0x4000 && !outs) { \
            outs = true; vst = vout; \
            _Pragma("unroll") for (int k = 0; k < 7; ++k) { int q = k + rot; q = q >= 7 ? q - 7 : q; rD[k] = mk_rsrc(lane_ptr(peer, 8 + q), EXP_NOPUSH ? 0u : OUT_BYTES); } \
            asm volatile("s_nop 4"); \
        } \
        const unsigned dst = outs ? out_off(e0) : stage_off(e0) + par_off; \
        const unsigned km = outs ? 127u : (unsigned)(kneed >> ((e0 >> 11) * 8)); \
        ST1(D, 0); ST1(D, 1); ST1(D, 2); ST1(D, 3); ST1(D, 4); ST1(D, 5); ST1(D, 6); \
        asm volatile("s_nop 1" ::"v"(D)); \
        { const bool isO = e2 >= 0x4000, isA = e2 < 256; \
          if (step <= CHECK_STEP && !isA) wait_packed(); \
          if (isO && !tiles_ok) { wait_tiles(); tiles_ok = true; } \
          const rsrc_t rs = isO ? rO : isA ? rX : rW; const int vs = isO ? vout : isA ? vx : voff; \
          const unsigned src = e2 >= 0x4000 + NOUT ? (unsigned)FAR : isO ? out_off(e2) : isA ? (unsigned)(((e2 >> 4) * 4 + (e2 & 3)) * (64 * KSH) + ((e2 >> 2) & 3) * 128) : stage_off(e2) + par_off + OFF_STAGE; \
          asm volatile("buffer_load_dwordx4 %0, %1, %2, %3 offen sc1" : "=v"(D) : "v"(vs), "s"(rs), "s"(src) : "memory"); } \
        if (!sent) WAIT_V(8); \
        unsigned tks; \
        asm volatile("s_waitcnt vmcnt(9)\n\tv_readfirstlane_b32 %0, %1\n\tv_mov_b32 %1, 1\n\tbuffer_atomic_add %1, %3, %4, %5 offen sc0 sc1" \
                     : "=&s"(tks), "+v"(tkv), "+v"(O) : "v"(vdraw), "s"(rW), "s"(OFF_TICKET) : "memory"); \
        if (wid == 0) lds_w[step & 1] = tks; \
        if (wid == 2 && step == CHECK_STEP) wait_packed(); \
        WAIT_L0(); \
        BAR(); \
        const unsigned tk = __builtin_amdgcn_readfirstlane(lds_w[step & 1]); \
        const bool dead = __builtin_amdgcn_readfirstlane(lds_w[2]) != 0u; \
        sent = wid == 1 && ep < 0x800 && !dead; \
        if (wid == 1 && ep < 0x4000 && !dead) send_flags(ep); \
        CST(); \
        const int nn = step > 0 ? (int)(tk - tick_base) : 3 * NSEND + ci; \
        CSHIFT(nn); \
        ep = e0; e0 = e1; e1 = e2; e2 = code_of(nn); \
        ++step; \
    } while (0)
    for (;;) {
        STEP(X, Y);
        if (e0 >= 0x4000 + NOUT) break;
        STEP(Y, X);
        if (e0 >= 0x4000 + NOUT) break;
    }
    asm volatile("s_waitcnt vmcnt(0)" : "+v"(X), "+v"(Y), "+v"(tkv) :: "memory");
    BAR();
    CST();
    if (wid == 1 && __builtin_amdgcn_readfirstlane(lds_w[2]) == 0u) {
        if (ep < 0x4000) send_flags(ep);
        if (lane < 7 && ci < 16) __hip_atomic_store((GAS unsigned*)(peer + (FLAG_DONE + (8 - ql) * 64 + ci * 4)), call, __ATOMIC_RELAXED, __HIP_MEMORY_SCOPE_SYSTEM);
    }
    if (wid == 0) {
        if (ci == 0 && !EXP_NOWAIT && !wait_words<1>(rW, call, lane * 16, lane >= 4 && lane < 32, OFF_FLAG + FLAG_DONE, 0))
            __builtin_amdgcn_raw_buffer_store_b32(0x7fc00000u, rO, vtick, 0, AUX_SYS);
        __builtin_amdgcn_raw_buffer_store_b32(call, rW, 0, OFF_CNT + b * 128, AUX_SYS);
    }
    (void)t0;
}

template <int GRP>
__device__ __forceinline__ void gemm_role(const Params& P, rsrc_t rW, LAS char* lds, int wid, int lane, unsigned call,
                                          int ri, int kc, unsigned t0) {
    volatile LAS unsigned* lds_fail = (volatile LAS unsigned*)(lds + 2 * LBUF);
    const unsigned par_off = (call & 1u) * SHARD_BYTES;
    const int wc = wid & 3;
    const int voff = lane * 16;
#define STMP() do {} while (0)
    STMP();
    const float scale = P.sx[0] * P.sw[0];

    unsigned rb_p = (unsigned)ri * 0x01010101u, cb_p = (unsigned)kc * 0x01010101u + 0x30201000u;
    const bool shared = kc == 14;
    if (shared) {
        const int t = (P.rank & 7) * 10 + ri;
        rb_p = (rb_p & 0xffff00ffu) | (unsigned)(t & 15) << 8;
        cb_p = (cb_p & 0xffff00ffu) | (ri < 10 ? (unsigned)(SHARED_PACK >> ((t >> 4) * 8 & 63)) & 255u : 255u) << 8;
    }

    int ready = (wid == 4 && !EXP_NOWAIT) ? 0 : 1 << 20;
    auto ensure_slow = [&](int need) {
        const unsigned w0 = now_ticks();
        const int round = need >> 5, s0 = ready & 31;
        const int rb = (rb_p >> (8 * round)) & 255, cb = (cb_p >> (8 * round)) & 255;
        int vz = voff; asm volatile("" : "+v"(vz));
        const bool use = (vz & 511) >= s0 * 16 && cb != 255;
        const int vf = use ? (vz & 511) + (vz < 512 ? 16 + cb : rb) * 512 : OOB;
        for (unsigned spins = 1;; ++spins) {
            const u32x4 f = __builtin_amdgcn_raw_buffer_load_b128(rW, vf, OFF_FLAG, AUX_SYS);
            const unsigned long long bad = __ballot(use && min(min(f[0], f[1]), min(f[2], f[3])) < call);
            const unsigned bad32 = (unsigned)bad | (unsigned)(bad >> 32);
            ready = round * 32 + (bad32 ? __builtin_ctz(bad32) : 32);
            ready = ready == TOT ? 1 << 20 : ready;
            if (ready > need) break;
            if ((spins & 63u) == 0u && wait_expired(rW, call, w0)) { lds_fail[0] = 1u; ready = 1 << 20; break; }
            __builtin_amdgcn_s_sleep(POLL_SLEEP);
        }
    };
#define ENSURE(need) do { if (GRP == 1 && __builtin_expect(ready <= (need), 0)) ensure_slow(need); } while (0)

    const rsrc_t rS = mk_rsrc(P.ws0 + OFF_STAGE + par_off, 8u * REGION_BYTES - SHARD_BYTES);
    const rsrc_t rC = mk_rsrc(P.out0, OUT_BYTES);
    int src_tab;
    {
        const int s = lane & 31, u = s - 4, c = s < 4 ? s : (u * 37) >> 8;
        src_tab = (s < 4 ? 0 : (u - c * 7 + 1) * (int)REGION_BYTES) + (c << 19);
        if (lane >= 32) {
            const int j = lane & 7, rb = (rb_p >> (8 * (j & 3))) & 255, cb = (cb_p >> (8 * (j & 3))) & 255;
            src_tab = (j >= 4 || cb == 255) ? FAR : (lane & 8) ? (((cb >> 4) + 1) << 21) + (cb & 15) * 32768 + wid * 4096 : rb * 32768 + (GRP * 8 + wc) * 2048;
        }
    }
#define DMA(D, SO, IMM) __builtin_amdgcn_raw_ptr_buffer_load_lds(rS, D, 16, voff, SO, IMM, AUX_SYS)
#define ISSUE_B(so, buf) do { LAS char* d_ = lds + (buf) * LBUF + wid * 4096; DMA(d_, so, 0); DMA(d_, so, 1024); DMA(d_, so, 2048); DMA(d_, so, 3072); } while (0)
#define ISSUE_ALO(so, buf) do { LAS char* d_ = lds + (buf) * LBUF + 32768 + wid * 2048; DMA(d_, so, 0); DMA(d_, so, 1024); } while (0)
#define ISSUE_AHI(so, buf) do { LAS char* d_ = lds + (buf) * LBUF + 49152 + wid * 2048; DMA(d_, so, 0); DMA(d_, so, 1024); } while (0)

    i32x4 acc[8][4];
    i32x4 af[4][2], bf[4][2];
    const int rdA0 = 32768 + GRP * 8192 + (lane & 15) * 128 + (((lane >> 4) ^ ((lane >> 1) & 7)) << 4);
    const int rdA[2] = {rdA0, rdA0 ^ 64};
    const int rdB = wc * 4096 + lane * 16;

    ENSURE(1);
    BAR();
    STMP();
    int aj = __builtin_amdgcn_readlane(src_tab, 32), bj = __builtin_amdgcn_readlane(src_tab, 40);
    ISSUE_B(bj, 0); ISSUE_ALO(aj, 0); ISSUE_AHI(aj + 8192, 0);
    ISSUE_B(bj + (1 << 19), 1); ISSUE_ALO(aj + (1 << 19), 1);
    int oA1 = aj + (1 << 19) + 8192;
    WAIT_V(8);
    BAR();
    if (GRP == 1) BAR();

#define MP(ROW, A_, t, k0, Z) do { \
        if (Z) asm volatile("v_mfma_i32_16x16x64_i8 %0, %1, %2, 0\n\tv_mfma_i32_16x16x64_i8 %0, %3, %4, %0" : "=&v"(acc[ROW][t]) : "v"(A_[k0]), "v"(bf[t][k0]), "v"(A_[1 - (k0)]), "v"(bf[t][1 - (k0)])); \
        else asm volatile("v_mfma_i32_16x16x64_i8 %0, %1, %2, %0\n\tv_mfma_i32_16x16x64_i8 %0, %3, %4, %0" : "+v"(acc[ROW][t]) : "v"(A_[k0]), "v"(bf[t][k0]), "v"(A_[1 - (k0)]), "v"(bf[t][1 - (k0)])); \
        SCHED(); } while (0)
#define MROW_UP(ROW, A_, Z) do { MP(ROW, A_, 0, 0, Z); MP(ROW, A_, 1, 1, Z); MP(ROW, A_, 2, 0, Z); MP(ROW, A_, 3, 1, Z); } while (0)
#define MROW_DN(ROW, A_, Z) do { MP(ROW, A_, 3, 0, Z); MP(ROW, A_, 2, 1, Z); MP(ROW, A_, 1, 0, Z); MP(ROW, A_, 0, 1, Z); } while (0)
#define PIN(x) do { asm volatile("" : "+s"(x)); SCHED(); } while (0)
#define SEG_WAIT(FIRST) do { if (FIRST) { if (round == 0) WAIT_V(8); WAIT_V(40); } else WAIT_V(8); WAIT_L0(); SCHED(); BAR(); SCHED(); } while (0)
#define STAGE(buf, FIRST) do { \
        LAS char* sb = lds + (buf) * LBUF; \
        _Pragma("unroll") for (int t = 0; t < 4; ++t) _Pragma("unroll") for (int kk = 0; kk < 2; ++kk) bf[t][kk] = *(LAS i32x4*)(sb + rdB + kk * 16384 + t * 1024); \
        _Pragma("unroll") for (int m = 0; m < 4; ++m) _Pragma("unroll") for (int kk = 0; kk < 2; ++kk) af[m][kk] = *(LAS i32x4*)(sb + rdA[kk] + m * 2048); \
        ISSUE_AHI(oA1, (buf) ^ 1); \
        ENSURE(g2); \
        SEG_WAIT(FIRST); \
        __builtin_amdgcn_s_setprio(1); \
        MP(0, af[0], 0, 0, FIRST); PIN(g2); int s2 = g2 & 31; PIN(s2); \
        MP(0, af[0], 1, 1, FIRST); int so2 = __builtin_amdgcn_readlane(src_tab, s2); PIN(so2); \
        MP(0, af[0], 2, 0, FIRST); int oA2 = aj + so2; PIN(oA2); int oB2 = bj + so2; PIN(oB2); \
        MP(0, af[0], 3, 1, FIRST); oA1 = oA2 + 8192; PIN(oA1); g2 += 1; PIN(g2); \
        MROW_DN(1, af[1], FIRST); MROW_UP(2, af[2], FIRST); MROW_DN(3, af[3], FIRST); \
        __builtin_amdgcn_s_setprio(0); \
        SCHED(); BAR(); SCHED(); \
        _Pragma("unroll") for (int m = 0; m < 4; ++m) _Pragma("unroll") for (int kk = 0; kk < 2; ++kk) af[m][kk] = *(LAS i32x4*)(sb + rdA[kk] + 16384 + m * 2048); \
        ISSUE_B(oB2, buf); ISSUE_ALO(oA2, buf); \
        SEG_WAIT(FIRST); \
        __builtin_amdgcn_s_setprio(1); \
        if ((buf) == 1) { \
            MP(4, af[0], 0, 0, FIRST); PIN(g2); int j2 = (g2 >> 5) + 32; PIN(j2); \
            MP(4, af[0], 1, 1, FIRST); aj = __builtin_amdgcn_readlane(src_tab, j2); PIN(aj); j2 += 8; PIN(j2); \
            MP(4, af[0], 2, 0, FIRST); bj = __builtin_amdgcn_readlane(src_tab, j2); PIN(bj); \
            MP(4, af[0], 3, 1, FIRST); \
        } else { MROW_UP(4, af[0], FIRST); } \
        MROW_DN(5, af[1], FIRST); MROW_UP(6, af[2], FIRST); MROW_DN(7, af[3], FIRST); \
        __builtin_amdgcn_s_setprio(0); \
        SCHED(); BAR(); SCHED(); \
    } while (0)

    int g2 = 2;
    for (int round = 0; round < 4; ++round) {
        STAGE(0, true);
        STAGE(1, false);
        if (shared && round == 2 && ri < 10)
            __builtin_amdgcn_raw_buffer_store_b32(call, rW, voff == 0 ? 0 : OOB, OFF_TILE + (ri * 8 + wid) * 4, AUX_SYS);
        for (int s = 2; s < 32; s += 2) {
            STAGE(0, false);
            STAGE(1, false);
        }
        STMP();
        {
            int vz = voff; asm volatile("s_nop 7\n\ts_nop 7" : "+v"(vz)); SCHED();
            const int vo = (vz >> 8) * 4 * N * 4 + (vz & 255);
            const int rb = (rb_p >> (8 * round)) & 255, cb = (cb_p >> (8 * round)) & 255;
            const unsigned so0 = cb == 255 ? (unsigned)FAR : (unsigned)((rb * 256 + GRP * 128) * N + cb * 256 + wc * 64) * 4u;
            const float sc = __builtin_amdgcn_readfirstlane(lds_fail[0]) ? __builtin_nanf("") : scale;
#define EPILOGUE(THRU) do { _Pragma("unroll") for (int m = 0; m < 8; ++m) _Pragma("unroll") for (int r = 0; r < 4; ++r) { f32x4 o; \
                _Pragma("unroll") for (int t = 0; t < 4; ++t) o[t] = __builtin_elementwise_maximum((float)acc[m][t][r] * sc, 0.f);     \
                store_out<THRU>(o, rC, vo, so0 + (unsigned)((m * 16 + r) * N * 4)); SCHED(); } } while (0)
            if (__builtin_expect(shared && round == 1, 0)) { asm volatile("" : "+v"(acc[0][0])); EPILOGUE(true); }
            else EPILOGUE(false);
        }
        STMP();
    }
    WAIT_V(0);
    if (GRP == 0) BAR();
    (void)t0;
}

__global__ __launch_bounds__(512, 2) void fused_kernel(Params P) {
    extern __shared__ __attribute__((aligned(16))) char smem_[];
    LAS char* lds = (LAS char*)smem_;
    const unsigned t0 = STAMP ? now_ticks() : 0u;
    const int tid = threadIdx.x, lane = tid & 63;
    const int wid = __builtin_amdgcn_readfirstlane(tid >> 6);
    const int b = blockIdx.x;
    const rsrc_t rW = mk_rsrc(P.ws0, WS_NEED);
    const unsigned call_v = __builtin_amdgcn_raw_buffer_load_b32(rW, 0, OFF_CNT + b * 128, AUX_SYS);

    if (b < 16) {
        comm_role(P, rW, lds, b, b, wid, lane, call_v, t0);
        return;
    }

    const int n = wid * 240 + (b - 16);
    unsigned call;
    if (n < 1024) {
        const int j = n >> 8, c = (n >> 6) & 3, cqp = (n >> 2) & 15, part = n & 3;
        const rsrc_t rWm = mk_rsrc(P.w, KSH * N);
        const int vw = (lane >> 4) * 16 * N + (lane & 15) * 4;
        const unsigned sw0 = (unsigned)((c * 128 + (part >> 1) * 64) * N + (j * 16 + cqp) * 256 + (part & 1) * 128);
        unsigned d[2][16];
#pragma unroll
        for (int i = 0; i < 16; ++i) {
            d[0][i] = __builtin_amdgcn_raw_buffer_load_b32(rWm, vw, sw0 + i * N, 0);
            d[1][i] = __builtin_amdgcn_raw_buffer_load_b32(rWm, vw + 64, sw0 + i * N, 0);
        }
        call = __builtin_amdgcn_readfirstlane(call_v) + 1u;
        const unsigned so = OFF_STAGE + (call & 1u) * SHARD_BYTES + (2u << 20) + (unsigned)n * 8192u;
#pragma unroll
        for (int h = 0; h < 2; ++h) {
            i32x4 pb[4];
#pragma unroll
            for (int dd = 0; dd < 4; ++dd) {
                const unsigned d0 = d[h][4 * dd], d1 = d[h][4 * dd + 1], d2 = d[h][4 * dd + 2], d3 = d[h][4 * dd + 3];
                const unsigned lo01 = __builtin_amdgcn_perm(d1, d0, 0x05010400u), hi01 = __builtin_amdgcn_perm(d1, d0, 0x07030602u);
                const unsigned lo23 = __builtin_amdgcn_perm(d3, d2, 0x05010400u), hi23 = __builtin_amdgcn_perm(d3, d2, 0x07030602u);
                pb[0][dd] = (int)__builtin_amdgcn_perm(lo23, lo01, 0x05040100u);
                pb[1][dd] = (int)__builtin_amdgcn_perm(lo23, lo01, 0x07060302u);
                pb[2][dd] = (int)__builtin_amdgcn_perm(hi23, hi01, 0x05040100u);
                pb[3][dd] = (int)__builtin_amdgcn_perm(hi23, hi01, 0x07060302u);
            }
            store4k(pb[0], pb[1], pb[2], pb[3], rW, lane * 16, so + h * 4096u);
        }
        WAIT_V(0);
        __builtin_amdgcn_raw_buffer_store_b32(call, rW, lane == 0 ? 0 : OOB, OFF_FLAG + (16 + j * 16 + cqp) * 512 + c * 16 + part * 4, AUX_SYS);
    } else if (n >= 1200 && n < 1456) {
        const int c = (n - 1200) >> 6, un = (n - 1200) & 63;
        const rsrc_t rX = mk_rsrc(P.x, M * KSH);
        const int vx0 = (lane >> 3) * KSH + (((lane & 7) ^ (lane >> 4)) << 4), vx1 = (vx0 ^ 64) + 8 * KSH;
        u32x4 d[8];
#pragma unroll
        for (int i = 0; i < 8; ++i) d[i] = __builtin_amdgcn_raw_buffer_load_b128(rX, (i & 1) ? vx1 : vx0, (un * 64 + (i >> 1) * 16) * KSH + c * 128, 0);
        call = __builtin_amdgcn_readfirstlane(call_v) + 1u;
        const unsigned so = OFF_STAGE + (call & 1u) * SHARD_BYTES + ((unsigned)c << 19) + (unsigned)un * 8192u;
#pragma unroll
        for (int i = 0; i < 8; i += 4)
            store4k(__builtin_bit_cast(i32x4, d[i]), __builtin_bit_cast(i32x4, d[i + 1]), __builtin_bit_cast(i32x4, d[i + 2]), __builtin_bit_cast(i32x4, d[i + 3]), rW, lane * 16, so + i * 1024u);
        WAIT_V(0);
        __builtin_amdgcn_raw_buffer_store_b32(call, rW, lane == 0 ? 0 : OOB, OFF_FLAG + (un >> 2) * 512 + c * 16 + (un & 3) * 4, AUX_SYS);
    } else {
        call = __builtin_amdgcn_readfirstlane(call_v) + 1u;
    }
    if (wid == 4) *(volatile LAS unsigned*)(lds + 2 * LBUF) = 0u;

    const int xcd = b & 7, i30 = (b >> 3) - 2;
    const int kc = i30 < 28 ? (i30 >> 2) * 2 + (xcd & 1) : 14;
    const int ri = (xcd >> 1) * 4 + (i30 < 28 ? i30 & 3 : (xcd & 1) * 2 + (i30 - 28));
    if (NSEND > 16 && kc == 14 && ri >= 10) { BAR(); comm_role(P, rW, lds, 6 + ri, b, wid, lane, call_v, t0); return; }
    if (wid < 4) gemm_role<0>(P, rW, lds, wid, lane, call, ri, kc, t0);
    else gemm_role<1>(P, rW, lds, wid, lane, call, ri, kc, t0);
    if (wid == 0) __builtin_amdgcn_raw_buffer_store_b32(call, rW, 0, OFF_CNT + b * 128, AUX_SYS);
}

}

extern "C" void kernel_launch(int rank, int nranks, ncclComm_t comm,
                              hipStream_t stream,
                              void* const* d_in, const int* in_sizes, int n_in,
                              void* d_out, int out_size,
                              const int* in_dtypes, int out_dtype,
                              void* const* d_out_all,
                              void* const* d_ws_all, size_t ws_bytes) {
    (void)comm; (void)in_dtypes; (void)out_dtype;
    if (nranks != 8 || (unsigned)rank >= 8u || n_in != 4 || ws_bytes < WS_NEED || out_size != M * N || in_sizes[0] != M * KSH || in_sizes[1] != KSH * N) {
        fprintf(stderr, "unexpected problem shape\n");
        return;
    }
    static bool attr_set = false;
    if (!attr_set) {
        HIP_CHECK(hipFuncSetAttribute((const void*)fused_kernel, hipFuncAttributeMaxDynamicSharedMemorySize, LDS_BYTES));
        attr_set = true;
    }
    Params p;
    p.x = (const signed char*)d_in[0];
    p.w = (const signed char*)d_in[1];
    p.sx = (const float*)d_in[2];
    p.sw = (const float*)d_in[3];
    p.ws0 = (char*)d_ws_all[rank];
    p.out0 = (char*)d_out;
    for (int i = 0; i < 8; ++i) {
        const int q = i < 7 ? i + 1 : 1, r = (rank + q) & 7;
        p.peer[i] = (char*)d_ws_all[r] + OFF_FLAG;
        p.peer[8 + i] = (char*)d_out_all[r];
        p.peer[16 + i] = (char*)d_ws_all[r] + OFF_STAGE + (size_t)(8 - q) * REGION_BYTES;
    }
    p.rank = rank;
    hipLaunchKernelGGL(fused_kernel, dim3(NWG), dim3(512), LDS_BYTES, stream, p);
    HIP_CHECK(hipGetLastError());
}
```

```cpp
#include <hip/hip_runtime.h>
#include <rccl/rccl.h>
#include <cstdio>

#define HIP_CHECK(expr) do { hipError_t _e = (expr); if (_e != hipSuccess) { \
    fprintf(stderr, "HIP error %s at %s:%d\n", hipGetErrorString(_e), __FILE__, __LINE__); \
    return; } } while (0)

namespace {

typedef int i32x4 __attribute__((ext_vector_type(4)));
typedef unsigned u32x4 __attribute__((ext_vector_type(4)));
typedef unsigned u32x2 __attribute__((ext_vector_type(2)));
typedef float f32x4 __attribute__((ext_vector_type(4)));
typedef float f32x2 __attribute__((ext_vector_type(2)));
typedef __amdgpu_buffer_rsrc_t rsrc_t;
#define LAS __attribute__((address_space(3)))
#define GAS __attribute__((address_space(1)))

constexpr int M = 4096, N = 16384, KSH = 512;
constexpr unsigned OUT_BYTES = (unsigned)((size_t)M * N * 4);
constexpr int NWG = 256, NSEND = 16;
constexpr int TOT = 128;
constexpr unsigned OFF_CNT = 0;
constexpr unsigned OFF_TMO = 32768;
constexpr unsigned OFF_TICKET = 32768 + 128;
constexpr unsigned OFF_TILE = 32768 + 1024;
constexpr unsigned OFF_FLAG = 65536;
constexpr unsigned FLAG_DONE = 80 * 512;
constexpr unsigned FLAG_BYTES = FLAG_DONE + 512;
constexpr unsigned OFF_DBG = 1u << 20;
constexpr unsigned OFF_STAGE = 4u << 20;
constexpr unsigned SHARD_BYTES = 10u << 20;
constexpr unsigned REGION_BYTES = 2u * SHARD_BYTES;
constexpr unsigned WS_NEED = OFF_STAGE + 8u * REGION_BYTES;
constexpr int LBUF = 65536;
constexpr int LDS_BYTES = 2 * LBUF + 64 + 1024;
constexpr unsigned DEADLINE_TICKS = 30000000u;
constexpr int FAR = 0x40000000;
constexpr int OOB = 0x7ffffff0;

#define STAMP 0
#define EXP_NOWAIT 0
#define EXP_NOPUSH 0
#define HEAD 5
#define EPI_A 3
#define EPI_B 6
#define OUT_POL "sc0 sc1"
#define PACK_DELAY 3
#define PACK_STEP 2
#define POLL_LOOK 3
#define POLL_SLEEP 4
#define AUX_SYS 17

constexpr unsigned long long SHARED_NEED = 0xffull | 0x03ull << 8 | 0x0cull << 16 | 0x30ull << 24 | 0xc0ull << 32;
constexpr int NIN = 1280, NOUT = 320, NUNITS = NIN + NOUT;
#define FIRST_ROWS {0, 0}, {0, 1}, {1, 0}, {1, 1}, {0, 2}, {1, 2}, {0, 3}, {1, 3}
constexpr int row_info(int row) {
    constexpr int first[8][2] = {FIRST_ROWS};
    int n = 0;
    for (int r = 0;; ++r) {
        const int g = r < 8 ? first[r][0] : r / 4, c = r < 8 ? first[r][1] : r % 4;
        if (r == row) return n | g << 16 | c << 20;
        n += g == 0 ? 64 : g == 2 ? 76 : 60;
    }
}
static_assert((row_info(20) & 0xffff) == NIN, "send rows");

struct Params {
    const signed char* x;
    const signed char* w;
    const float* sx;
    const float* sw;
    char* ws0;
    char* out0;
    char* peer[24];
    int rank;
};

#define WAIT_V(n) asm volatile("s_waitcnt vmcnt(%0)" ::"n"(n) : "memory")
#define WAIT_L0() __builtin_amdgcn_s_waitcnt(0xC07F)
#define BAR() __builtin_amdgcn_s_barrier()
#define SCHED() __builtin_amdgcn_sched_barrier(0)

__device__ __forceinline__ rsrc_t mk_rsrc(const void* p, unsigned bytes) { return __builtin_amdgcn_make_buffer_rsrc((void*)p, 0, bytes, 0x00020000); }
__device__ __forceinline__ unsigned now_ticks() { return (unsigned)__builtin_amdgcn_s_memrealtime(); }
__device__ __forceinline__ char* lane_ptr(char* p, int l) {
    const unsigned lo = __builtin_amdgcn_readlane((unsigned)(unsigned long long)p, l), hi = __builtin_amdgcn_readlane((unsigned)((unsigned long long)p >> 32), l);
    return (char*)(((unsigned long long)hi << 32) | lo);
}

template <bool THRU>
__device__ __forceinline__ void store_out4(const f32x4 (&v)[4], rsrc_t r, int vo, unsigned so) {
#define ST4(POL) asm volatile("buffer_store_dwordx4 %0, %4, %5, %6 offen " POL "\n\tbuffer_store_dwordx4 %1, %4, %5, %7 offen " POL "\n\t" \
                              "buffer_store_dwordx4 %2, %4, %5, %8 offen " POL "\n\tbuffer_store_dwordx4 %3, %4, %5, %9 offen " POL "\n\ts_nop 1" \
                              ::"v"(v[0]), "v"(v[1]), "v"(v[2]), "v"(v[3]), "v"(vo), "s"(r), "s"(so), "s"(so + N * 4), "s"(so + 2 * N * 4), "s"(so + 3 * N * 4) : "memory")
    if (THRU) ST4(OUT_POL); else ST4("nt");
}
__device__ __forceinline__ void store4k(i32x4 a, i32x4 b, i32x4 c, i32x4 d, rsrc_t r, int vo, unsigned so) {
    asm volatile("buffer_store_dwordx4 %0, %4, %5, %6 offen sc0 sc1\n\t"
                 "buffer_store_dwordx4 %1, %4, %5, %6 offen offset:1024 sc0 sc1\n\t"
                 "buffer_store_dwordx4 %2, %4, %5, %6 offen offset:2048 sc0 sc1\n\t"
                 "buffer_store_dwordx4 %3, %4, %5, %6 offen offset:3072 sc0 sc1\n\t"
                 "s_nop 1"
                 ::"v"(a), "v"(b), "v"(c), "v"(d), "v"(vo), "s"(r), "s"(so) : "memory");
}

__device__ __forceinline__ bool wait_expired(rsrc_t rW, unsigned call, unsigned w0) {
    const unsigned tv = __builtin_amdgcn_readfirstlane(__builtin_amdgcn_raw_buffer_load_b32(rW, 0, OFF_TMO, AUX_SYS));
    const bool dead = tv == call || now_ticks() - w0 > DEADLINE_TICKS;
    if (dead) __builtin_amdgcn_raw_buffer_store_b32(call, rW, 0, OFF_TMO, AUX_SYS);
    return dead;
}
template <int CNT>
__device__ __forceinline__ bool wait_words(rsrc_t rW, unsigned call, int vo, bool use, unsigned so, int step) {
    const unsigned w0 = now_ticks();
    for (unsigned spins = 1;; ++spins) {
        unsigned mn = 0xffffffffu;
#pragma unroll
        for (int i = 0; i < CNT; ++i) {
            const u32x4 f = __builtin_amdgcn_raw_buffer_load_b128(rW, use ? vo : OOB, so + i * step, AUX_SYS);
            mn = min(mn, min(min(f[0], f[1]), min(f[2], f[3])));
        }
        if (__ballot(use && mn < call) == 0ull) return true;
        if ((spins & 63u) == 0u && wait_expired(rW, call, w0)) return false;
        __builtin_amdgcn_s_sleep(8);
    }
}

__device__ __forceinline__ unsigned stage_off(int e) { return (unsigned)((e >> 8) & 7) << 21 | (unsigned)((e >> 2) & 3) << 19 | (unsigned)((e >> 4) & 15) << 15 | (unsigned)(e & 3) << 13; }

__device__ __forceinline__ void comm_role(const Params& P, rsrc_t rW, LAS char* lds, int ci, int b, int wid, int lane, unsigned call_v, unsigned t0) {
    volatile LAS unsigned* lds_w = (volatile LAS unsigned*)lds;
    const rsrc_t rX = mk_rsrc(P.x, M * KSH);
    const int voff = lane * 16 + wid * 1024, vout = lane * 16 + wid * (N * 4);
    const int vx = ((lane >> 3) + wid * 8) * KSH + (((lane & 7) ^ (lane >> 4) ^ ((wid & 1) << 2)) << 4);
    i32x4 X, Y;
    asm volatile("buffer_load_dwordx4 %0, %2, %3, %4 offen\n\tbuffer_load_dwordx4 %1, %2, %3, %5 offen"
                 : "=&v"(X), "=&v"(Y) : "v"(vx), "s"(rX), "s"(ci * 64 * KSH), "s"((NSEND + ci) * 64 * KSH) : "memory");
    char* const peer = P.peer[lane < 24 ? lane : 0];
    const int rank = P.rank & 7;
    const int ql = (lane & 7) + 1;
    if (wid == 2) lds_w[2] = 0u;
    unsigned long long kneed = 0;
    const int rot = (ci + wid) % 7;
#pragma unroll
    for (int k = 0; k < 7; ++k) {
        int q = k + rot; q = (q >= 7 ? q - 7 : q) + 1;
        kneed |= ((SHARED_NEED >> ((rank + q) & 7)) & 0x0101010101ull) << k;
    }
    const rsrc_t rO = mk_rsrc(P.out0, OUT_BYTES);
    const int vtick = lane == 0 ? 0 : OOB;

    int row_tab = 0xffff;
#pragma unroll
    for (int r = 0; r < 20; ++r) row_tab = lane == r ? row_info(r) : row_tab;
    auto code_of = [&](int n) {
        const int ri_ = __builtin_amdgcn_readlane(row_tab, __builtin_popcountll(__ballot((row_tab & 0xffff) <= n)) - 1);
        int i = n - (ri_ & 0xffff);
        const int g = (ri_ >> 16) & 7, c = ri_ >> 20;
        const int i5 = (i * 205) >> 10;
        const bool is_sh = g == 2 && (i - 5 * i5 == 4 || i == 75);
        const int sh = (i5 >> 2) | c << 2 | (31 + (i5 & 3) * 16) << 4 | ((i5 & 3) + 1) << 11;
        if (g == 2) i -= i5;
        return n >= NIN ? 0x4000 + n - NIN : is_sh ? sh : (i & 3) | c << 2 | (g * 16 + (i >> 2)) << 4;
    };
    auto out_dist = [&](int e) { const int i = e - 0x4000, j = (i * 205) >> 11; return i - 10 * j == 9 ? 7 - (j - ((j * 37) >> 8) * 7) : 0; };
    auto out_off = [&](int e) {
        const int i = e - 0x4000, j = (i * 205) >> 11, u = i - j, q = out_dist(e);
        const int row8 = q ? (8 + ((rank + q) & 7)) * 32 + j : u < 256 ? (rank & 1) * 256 + u : rank * 32 + u - 256;
        return (unsigned)(row8 * (8 * N * 4) + (q == 0 && u < 256 ? (rank >> 1) * 16 + 15 : 30) * 1024);
    };
    asm volatile("" : "+v"(row_tab), "+s"(kneed));
    SCHED();
    rsrc_t rD[7];
#pragma unroll
    for (int k = 0; k < 7; ++k) {
        int q = k + rot; q = q >= 7 ? q - 7 : q;
        rD[k] = mk_rsrc(lane_ptr(peer, 16 + q), EXP_NOPUSH ? 0u : REGION_BYTES);
    }
    const unsigned call = __builtin_amdgcn_readfirstlane(call_v) + 1u;
    const unsigned par_off = (call & 1u) * SHARD_BYTES;
    const unsigned tick_base = (call - 1u) * (unsigned)NUNITS - 3u * NSEND;
    auto flag_ptr = [&](int e) { return (GAS unsigned*)(peer + (((e >> 4) & 127) * 512 + (11 + 7 * ((e >> 2) & 3) - ql) * 16 + (e & 3) * 4)); };
    auto flag_on = [&](int e) { return lane < 7 && e < 0x4000 && ((SHARED_NEED >> (((e >> 11) & 7) * 8 + ((rank + ql) & 7))) & 1); };
    auto wait_packed = [&]() { if (!wait_words<4>(rW, call, lane * 512, true, OFF_FLAG + 16 * 512, 16)) lds_w[2] = 1u; };
    auto wait_tiles = [&]() { if (!wait_words<1>(rW, call, lane * 16, lane < 32, OFF_TILE, 0)) lds_w[2] = 1u; };
#define CST() do {} while (0)
#define CSHIFT(nn) do {} while (0)
#define ST1(D, k) asm volatile("buffer_store_dwordx4 %0, %1, %2, %3 offen sc0 sc1" ::"v"(D), "v"(vst), "s"(rD[k]), "s"(so[k]) : "memory")

    int e0 = (ci & 3) | (ci >> 2) << 4, e1 = e0 + 64;
    unsigned so[7];
#pragma unroll
    for (int k = 0; k < 7; ++k) so[k] = stage_off(e0) + par_off;
    GAS unsigned* fp = flag_ptr(e0);
    bool fon = false;
    asm volatile("s_waitcnt vmcnt(0)\n\ts_nop 4" : "+v"(X), "+v"(Y) :: "memory");
    int step = 0, vst = voff;
    unsigned tkv = 0;
    bool outs = false, tiles_ok = false;
    constexpr int CHECK_STEP = 5;
    const int vdraw = wid == 0 ? vtick : OOB;
#define STEP(D, O) do { \
        _Pragma("unroll") for (int k = 0; k < HEAD; ++k) ST1(D, k); \
        unsigned tks; \
        asm volatile("s_waitcnt vmcnt(%6)\n\tv_readfirstlane_b32 %0, %1\n\tv_mov_b32 %1, 1\n\tbuffer_atomic_add %1, %3, %4, %5 offen sc0 sc1" \
                     : "=&s"(tks), "+v"(tkv), "+v"(O) : "v"(vdraw), "s"(rW), "s"(OFF_TICKET), "n"(HEAD) : "memory"); \
        if (wid == 0) lds_w[step & 1] = tks; \
        if (wid == 2 && step == CHECK_STEP) wait_packed(); \
        WAIT_L0(); \
        BAR(); \
        if (fon) __hip_atomic_store(fp, call, __ATOMIC_RELAXED, __HIP_MEMORY_SCOPE_SYSTEM); \
        CST(); \
        _Pragma("unroll") for (int k = HEAD; k < 7; ++k) ST1(D, k); \
        asm volatile("s_nop 1" ::"v"(D)); \
        SCHED(); \
        const unsigned tk = __builtin_amdgcn_readfirstlane(lds_w[step & 1]); \
        const bool dead = __builtin_amdgcn_readfirstlane(lds_w[2]) != 0u; \
        const int nn = step > 0 ? (int)(tk - tick_base) : 2 * NSEND + ci; \
        const int e2 = code_of(nn); \
        { const bool isO = e2 >= 0x4000, isA = e2 < 256; \
          if (step < CHECK_STEP && !isA) wait_packed(); \
          if (isO && !tiles_ok) { wait_tiles(); tiles_ok = true; } \
          const rsrc_t rs = isO ? rO : isA ? rX : rW; const int vs = isO ? vout : isA ? vx : voff; \
          const unsigned src = e2 >= 0x4000 + NOUT ? (unsigned)FAR : isO ? out_off(e2) : isA ? (unsigned)(((e2 >> 4) * 4 + (e2 & 3)) * (64 * KSH) + ((e2 >> 2) & 3) * 128) : stage_off(e2) + par_off + OFF_STAGE; \
          asm volatile("buffer_load_dwordx4 %0, %1, %2, %3 offen sc1" : "=v"(D) : "v"(vs), "s"(rs), "s"(src) : "memory"); } \
        if (e1 >= 0x4000 && !outs) { \
            outs = true; vst = vout; \
            _Pragma("unroll") for (int k = 0; k < 7; ++k) { int q = k + rot; q = q >= 7 ? q - 7 : q; rD[k] = mk_rsrc(lane_ptr(peer, 8 + q), EXP_NOPUSH ? 0u : OUT_BYTES); } \
            asm volatile("s_nop 4"); \
        } \
        { const unsigned dst = outs ? out_off(e1) : stage_off(e1) + par_off; \
          int ko = out_dist(e1) - 1 - rot; ko = ko < 0 ? ko + 7 : ko; \
          const unsigned km = outs ? (out_dist(e1) ? 1u << ko : 127u) : (unsigned)(kneed >> ((e1 >> 11) * 8)); \
          _Pragma("unroll") for (int k = 0; k < 7; ++k) so[k] = (km >> k) & 1 ? dst : (unsigned)FAR; } \
        fp = flag_ptr(e0); \
        fon = wid == 1 && !dead && flag_on(e0); \
        CSHIFT(nn); \
        e0 = e1; e1 = e2; \
        ++step; \
    } while (0)
    for (;;) {
        STEP(X, Y);
        if (e0 >= 0x4000 + NOUT) break;
        STEP(Y, X);
        if (e0 >= 0x4000 + NOUT) break;
    }
    asm volatile("s_waitcnt vmcnt(0) lgkmcnt(0)" : "+v"(X), "+v"(Y), "+v"(tkv) :: "memory");
    BAR();
    CST();
    if (wid == 1 && __builtin_amdgcn_readfirstlane(lds_w[2]) == 0u) {
        if (fon) __hip_atomic_store(fp, call, __ATOMIC_RELAXED, __HIP_MEMORY_SCOPE_SYSTEM);
        if (lane < 7) __hip_atomic_store((GAS unsigned*)(peer + (FLAG_DONE + (8 - ql) * 64 + ci * 4)), call, __ATOMIC_RELAXED, __HIP_MEMORY_SCOPE_SYSTEM);
    }
    if (wid == 0) {
        if (ci == 0 && !EXP_NOWAIT && !wait_words<1>(rW, call, lane * 16, lane >= 4 && lane < 32, OFF_FLAG + FLAG_DONE, 0))
            __builtin_amdgcn_raw_buffer_store_b32(0x7fc00000u, rO, vtick, 0, AUX_SYS);
        __builtin_amdgcn_raw_buffer_store_b32(call, rW, 0, OFF_CNT + b * 128, AUX_SYS);
    }
    (void)t0;
}

template <int GRP>
__device__ __forceinline__ void gemm_role(const Params& P, rsrc_t rW, LAS char* lds, int wid, int lane, unsigned call,
                                          int ri, int kc, unsigned t0) {
    volatile LAS unsigned* lds_fail = (volatile LAS unsigned*)(lds + 2 * LBUF);
    const unsigned par_off = (call & 1u) * SHARD_BYTES;
    const int wc = wid & 3;
    const int voff = lane * 16;
#define STMP() do {} while (0)
    STMP();
    const float scale = P.sx[0] * P.sw[0];

    unsigned rb_p = (unsigned)ri * 0x01010101u, cb_p = (unsigned)kc * 0x01010101u + 0x30201000u;
    const bool shared = kc == 14;
    if (shared) {
        const int rank = P.rank & 7;
        rb_p = (rb_p & 0xffff00ffu) | (unsigned)(ri < 8 ? (rank & 1) * 8 + ri : ri == 8 + rank ? rank : ri) << 8;
        if (ri < 8) cb_p = (cb_p & 0xffff00ffu) | (unsigned)((rank >> 1) * 16 + 15) << 8;
    }

    int ready = (wid == 4 && !EXP_NOWAIT) ? 0 : 1 << 20, soft = ready, pend = -1;
    LAS char* const snap = lds + 2 * LBUF + 64;
    auto ensure_slow = [&](int need) {
        int vz = voff; asm volatile("" : "+v"(vz));
        bool use; int vf;
        auto aim = [&]() {
            const int round = ready >> 5, rb = (rb_p >> (8 * round)) & 255, cb = (cb_p >> (8 * round)) & 255;
            use = (vz & 511) >= (ready & 31) * 16;
            vf = use ? (vz & 511) + (vz < 512 ? 16 + cb : rb) * 512 : OOB;
        };
        auto take = [&](u32x4 f) {
            const unsigned long long bad = __ballot(use && min(min(f[0], f[1]), min(f[2], f[3])) < call);
            const unsigned bad32 = (unsigned)bad | (unsigned)(bad >> 32);
            ready = (ready & ~31) + (bad32 ? __builtin_ctz(bad32) : 32);
            ready = ready == TOT ? 1 << 20 : ready;
        };
        if (pend >= 0 && need >= pend) { aim(); take(*(volatile LAS u32x4*)(snap + vz)); pend = -1; }
        if (ready <= need) {
            const unsigned w0 = now_ticks();
            aim();
            for (unsigned spins = 1;; ++spins) {
                const int r0 = ready;
                take(__builtin_amdgcn_raw_buffer_load_b128(rW, vf, OFF_FLAG, AUX_SYS));
                if (ready > need) break;
                ready = r0;
                if ((spins & 63u) == 0u && wait_expired(rW, call, w0)) { lds_fail[0] = 1u; ready = 1 << 20; break; }
                __builtin_amdgcn_s_sleep(POLL_SLEEP);
            }
            pend = -1;
        } else if (pend < 0 && ready - need <= POLL_LOOK) {
            aim();
            __builtin_amdgcn_raw_ptr_buffer_load_lds(rW, snap, 16, vf, OFF_FLAG, 0, AUX_SYS);
            pend = need + ((need & 30) == 2 ? 2 : 1);
        }
        soft = pend >= 0 ? min(pend, ready) : max(ready - POLL_LOOK, need + 1);
    };
#define ENSURE(need) do { if (GRP == 1 && __builtin_expect(soft <= (need), 0)) ensure_slow(need); } while (0)

    const rsrc_t rS = mk_rsrc(P.ws0 + OFF_STAGE + par_off, 8u * REGION_BYTES - SHARD_BYTES);
    const rsrc_t rC = mk_rsrc(P.out0, OUT_BYTES);
    int src_tab;
    {
        const int s = lane & 31, u = s - 4, c = s < 4 ? s : (u * 37) >> 8;
        src_tab = (s < 4 ? 0 : (u - c * 7 + 1) * (int)REGION_BYTES) + (c << 19);
        if (lane >= 32) {
            const int j = lane & 7, rb = (rb_p >> (8 * (j & 3))) & 255, cb = (cb_p >> (8 * (j & 3))) & 255;
            src_tab = j >= 4 ? FAR : (lane & 8) ? (((cb >> 4) + 1) << 21) + (cb & 15) * 32768 + wid * 4096 : rb * 32768 + (GRP * 8 + wc) * 2048;
        }
    }
#define DMA(D, SO, IMM) __builtin_amdgcn_raw_ptr_buffer_load_lds(rS, D, 16, voff, SO, IMM, AUX_SYS)
#define ISSUE_B(so, buf) do { LAS char* d_ = lds + (buf) * LBUF + wid * 4096; DMA(d_, so, 0); DMA(d_, so, 1024); DMA(d_, so, 2048); DMA(d_, so, 3072); } while (0)
#define ISSUE_ALO(so, buf) do { LAS char* d_ = lds + (buf) * LBUF + 32768 + wid * 2048; DMA(d_, so, 0); DMA(d_, so, 1024); } while (0)
#define ISSUE_AHI(so, buf) do { LAS char* d_ = lds + (buf) * LBUF + 49152 + wid * 2048; DMA(d_, so, 0); DMA(d_, so, 1024); } while (0)

    i32x4 acc[8][4];
    i32x4 af[4][2], bf[4][2];
    const int rdA0 = 32768 + GRP * 8192 + (lane & 15) * 128 + (((lane >> 4) ^ ((lane >> 1) & 7)) << 4);
    const int rdA[2] = {rdA0, rdA0 ^ 64};
    const int rdB = wc * 4096 + lane * 16;

    ENSURE(1);
    BAR();
    int aj = __builtin_amdgcn_readlane(src_tab, 32), bj = __builtin_amdgcn_readlane(src_tab, 40);
    ISSUE_B(bj, 0); ISSUE_ALO(aj, 0); ISSUE_AHI(aj + 8192, 0);
    ISSUE_B(bj + (1 << 19), 1); ISSUE_ALO(aj + (1 << 19), 1);
    int oA1 = aj + (1 << 19) + 8192;
    const int vo = (voff >> 8) * 4 * N * 4 + (voff & 255);
    unsigned so0 = FAR;
    f32x2 sc2 = {0.f, 0.f};
#pragma unroll
    for (int m = 0; m < 8; ++m)
#pragma unroll
        for (int t = 0; t < 4; ++t) acc[m][t] = (i32x4){0, 0, 0, 0};
    {
        const f32x4 z[4] = {};
#pragma unroll
        for (int m = 0; m < EPI_A; ++m) store_out4<true>(z, rC, vo, so0);
    }
    WAIT_V(8 + 4 * EPI_A);
    BAR();
    if (GRP == 1) BAR();

#define MP(ROW, A_, t, k0, Z) do { \
        if (Z) asm volatile("v_mfma_i32_16x16x64_i8 %0, %1, %2, 0\n\tv_mfma_i32_16x16x64_i8 %0, %3, %4, %0" : "=&v"(acc[ROW][t]) : "v"(A_[k0]), "v"(bf[t][k0]), "v"(A_[1 - (k0)]), "v"(bf[t][1 - (k0)])); \
        else asm volatile("v_mfma_i32_16x16x64_i8 %0, %1, %2, %0\n\tv_mfma_i32_16x16x64_i8 %0, %3, %4, %0" : "+v"(acc[ROW][t]) : "v"(A_[k0]), "v"(bf[t][k0]), "v"(A_[1 - (k0)]), "v"(bf[t][1 - (k0)])); \
        SCHED(); } while (0)
#define MROW_UP(ROW, A_, Z) do { MP(ROW, A_, 0, 0, Z); MP(ROW, A_, 1, 1, Z); MP(ROW, A_, 2, 0, Z); MP(ROW, A_, 3, 1, Z); } while (0)
#define MROW_DN(ROW, A_, Z) do { MP(ROW, A_, 3, 0, Z); MP(ROW, A_, 2, 1, Z); MP(ROW, A_, 1, 0, Z); MP(ROW, A_, 0, 1, Z); } while (0)
#define PIN(x) do { asm volatile("" : "+s"(x)); SCHED(); } while (0)
#define SEG_WAIT(n) do { WAIT_V(n); WAIT_L0(); SCHED(); BAR(); SCHED(); } while (0)
#define EMIT(lo, hi) do { _Pragma("unroll") for (int m = (lo); m < (hi); ++m) { f32x4 o[4]; SCHED(); \
        _Pragma("unroll") for (int r = 0; r < 4; ++r) { \
            const f32x2 lo2 = (f32x2){(float)acc[m][0][r], (float)acc[m][1][r]} * sc2, hi2 = (f32x2){(float)acc[m][2][r], (float)acc[m][3][r]} * sc2; \
            o[r] = __builtin_elementwise_maximum((f32x4){lo2[0], lo2[1], hi2[0], hi2[1]}, (f32x4){0.f, 0.f, 0.f, 0.f}); } \
        store_out4<true>(o, rC, vo, so0 + (unsigned)(m * 16 * N * 4)); SCHED(); } } while (0)
    constexpr int EA = EPI_A, EB = EPI_B;
#define STAGE(buf, KIND) do { \
        LAS char* sb = lds + (buf) * LBUF; \
        ENSURE(g2); \
        _Pragma("unroll") for (int t = 0; t < 4; ++t) _Pragma("unroll") for (int kk = 0; kk < 2; ++kk) bf[t][kk] = *(LAS i32x4*)(sb + rdB + kk * 16384 + t * 1024); \
        _Pragma("unroll") for (int m = 0; m < 4; ++m) _Pragma("unroll") for (int kk = 0; kk < 2; ++kk) af[m][kk] = *(LAS i32x4*)(sb + rdA[kk] + m * 2048); \
        ISSUE_AHI(oA1, (buf) ^ 1); \
        if (KIND == 1) EMIT(EA, EB); \
        SEG_WAIT(KIND == 1 ? 8 + 4 * EB : KIND == 2 ? 8 + 4 * (8 - EA) : 8); \
        __builtin_amdgcn_s_setprio(1); \
        MP(0, af[0], 0, 0, KIND == 1); PIN(g2); int s2 = g2 & 31; PIN(s2); \
        MP(0, af[0], 1, 1, KIND == 1); int so2 = __builtin_amdgcn_readlane(src_tab, s2); PIN(so2); \
        MP(0, af[0], 2, 0, KIND == 1); int oA2 = aj + so2; PIN(oA2); int oB2 = bj + so2; PIN(oB2); \
        MP(0, af[0], 3, 1, KIND == 1); oA1 = oA2 + 8192; PIN(oA1); g2 += 1; PIN(g2); \
        MROW_DN(1, af[1], KIND == 1); MROW_UP(2, af[2], KIND == 1); MROW_DN(3, af[3], KIND == 1); \
        __builtin_amdgcn_s_setprio(0); \
        SCHED(); BAR(); SCHED(); \
        _Pragma("unroll") for (int m = 0; m < 4; ++m) _Pragma("unroll") for (int kk = 0; kk < 2; ++kk) af[m][kk] = *(LAS i32x4*)(sb + rdA[kk] + 16384 + m * 2048); \
        unsigned failed_ = 0; if (KIND == 3) failed_ = lds_fail[0]; \
        ISSUE_B(oB2, buf); ISSUE_ALO(oA2, buf); \
        if (KIND == 3) { \
            const int rb = (rb_p >> (8 * round)) & 255, cb = (cb_p >> (8 * round)) & 255; \
            so0 = (unsigned)((rb * 256 + GRP * 128) * N + cb * 256 + wc * 64) * 4u; \
            sc2[0] = sc2[1] = failed_ ? __builtin_nanf("") : scale; \
            EMIT(0, EA); \
        } \
        if (KIND == 1) EMIT(EB, 8); \
        SEG_WAIT(KIND == 3 ? 8 + 4 * EA : KIND == 1 ? 40 : KIND == 2 ? 8 + 4 * (8 - EB) : 8); \
        __builtin_amdgcn_s_setprio(1); \
        if ((buf) == 1) { \
            MP(4, af[0], 0, 0, false); PIN(g2); int j2 = (g2 >> 5) + 32; PIN(j2); \
            MP(4, af[0], 1, 1, false); aj = __builtin_amdgcn_readlane(src_tab, j2); PIN(aj); j2 += 8; PIN(j2); \
            MP(4, af[0], 2, 0, false); bj = __builtin_amdgcn_readlane(src_tab, j2); PIN(bj); \
            MP(4, af[0], 3, 1, false); \
        } else { MROW_UP(4, af[0], KIND == 1); } \
        MROW_DN(5, af[1], KIND == 1); MROW_UP(6, af[2], KIND == 1); MROW_DN(7, af[3], KIND == 1); \
        __builtin_amdgcn_s_setprio(0); \
        SCHED(); BAR(); SCHED(); \
    } while (0)

    int g2 = 2;
    const int flag_at = shared ? 64 + 6 : -1;
    for (int round = 0; round < 4; ++round) {
        STAGE(0, 1);
        STAGE(1, 2);
        STMP();
        for (int s = 2; s < 30; s += 2) {
            if (__builtin_expect(g2 == flag_at, 0)) __builtin_amdgcn_raw_buffer_store_b32(call, rW, voff == 0 ? 0 : OOB, OFF_TILE + (ri * 8 + wid) * 4, AUX_SYS);
            STAGE(0, 0);
            STAGE(1, 0);
        }
        STMP();
        STAGE(0, 0);
        STAGE(1, 3);
    }
    asm volatile("s_nop 7\n\ts_nop 7"); SCHED();
    EMIT(EA, 8);
    STMP();
    WAIT_V(0);
    if (GRP == 0) BAR();
    (void)t0;
}

__global__ __launch_bounds__(512, 2) void fused_kernel(Params P) {
    extern __shared__ __attribute__((aligned(16))) char smem_[];
    LAS char* lds = (LAS char*)smem_;
    const unsigned t0 = STAMP ? now_ticks() : 0u;
    const int tid = threadIdx.x, lane = tid & 63;
    const int wid = __builtin_amdgcn_readfirstlane(tid >> 6);
    const int b = blockIdx.x;
    const rsrc_t rW = mk_rsrc(P.ws0, WS_NEED);
    const unsigned call_v = __builtin_amdgcn_raw_buffer_load_b32(rW, 0, OFF_CNT + b * 128, AUX_SYS);

    if (b < 16) {
        comm_role(P, rW, lds, b, b, wid, lane, call_v, t0);
        return;
    }

    const int n = wid * 240 + (b - 16);
    unsigned call;
    if (n < 1024) {
        const int j = n >> 8, c = (n >> 6) & 3, cqp = (n >> 2) & 15, part = n & 3;
        for (int i = 0; i < PACK_DELAY + PACK_STEP * wid; ++i) __builtin_amdgcn_s_sleep(16);
        const rsrc_t rWm = mk_rsrc(P.w, KSH * N);
        const int vw = (lane >> 4) * 16 * N + (lane & 15) * 4;
        const unsigned sw0 = (unsigned)((c * 128 + (part >> 1) * 64) * N + (j * 16 + cqp) * 256 + (part & 1) * 128);
        unsigned d[2][16];
#pragma unroll
        for (int i = 0; i < 16; ++i) {
            d[0][i] = __builtin_amdgcn_raw_buffer_load_b32(rWm, vw, sw0 + i * N, 0);
            d[1][i] = __builtin_amdgcn_raw_buffer_load_b32(rWm, vw + 64, sw0 + i * N, 0);
        }
        call = __builtin_amdgcn_readfirstlane(call_v) + 1u;
        const unsigned so = OFF_STAGE + (call & 1u) * SHARD_BYTES + (2u << 20) + (unsigned)n * 8192u;
#pragma unroll
        for (int h = 0; h < 2; ++h) {
            i32x4 pb[4];
#pragma unroll
            for (int dd = 0; dd < 4; ++dd) {
                const unsigned d0 = d[h][4 * dd], d1 = d[h][4 * dd + 1], d2 = d[h][4 * dd + 2], d3 = d[h][4 * dd + 3];
                const unsigned lo01 = __builtin_amdgcn_perm(d1, d0, 0x05010400u), hi01 = __builtin_amdgcn_perm(d1, d0, 0x07030602u);
                const unsigned lo23 = __builtin_amdgcn_perm(d3, d2, 0x05010400u), hi23 = __builtin_amdgcn_perm(d3, d2, 0x07030602u);
                pb[0][dd] = (int)__builtin_amdgcn_perm(lo23, lo01, 0x05040100u);
                pb[1][dd] = (int)__builtin_amdgcn_perm(lo23, lo01, 0x07060302u);
                pb[2][dd] = (int)__builtin_amdgcn_perm(hi23, hi01, 0x05040100u);
                pb[3][dd] = (int)__builtin_amdgcn_perm(hi23, hi01, 0x07060302u);
            }
            store4k(pb[0], pb[1], pb[2], pb[3], rW, lane * 16, so + h * 4096u);
        }
        WAIT_V(0);
        __builtin_amdgcn_raw_buffer_store_b32(call, rW, lane == 0 ? 0 : OOB, OFF_FLAG + (16 + j * 16 + cqp) * 512 + c * 16 + part * 4, AUX_SYS);
    } else if (n >= 1200 && n < 1456) {
        const int c = (n - 1200) >> 6, un = (n - 1200) & 63;
        const rsrc_t rX = mk_rsrc(P.x, M * KSH);
        const int vx0 = (lane >> 3) * KSH + (((lane & 7) ^ (lane >> 4)) << 4), vx1 = (vx0 ^ 64) + 8 * KSH;
        u32x4 d[8];
#pragma unroll
        for (int i = 0; i < 8; ++i) d[i] = __builtin_amdgcn_raw_buffer_load_b128(rX, (i & 1) ? vx1 : vx0, (un * 64 + (i >> 1) * 16) * KSH + c * 128, 0);
        call = __builtin_amdgcn_readfirstlane(call_v) + 1u;
        const unsigned so = OFF_STAGE + (call & 1u) * SHARD_BYTES + ((unsigned)c << 19) + (unsigned)un * 8192u;
#pragma unroll
        for (int i = 0; i < 8; i += 4)
            store4k(__builtin_bit_cast(i32x4, d[i]), __builtin_bit_cast(i32x4, d[i + 1]), __builtin_bit_cast(i32x4, d[i + 2]), __builtin_bit_cast(i32x4, d[i + 3]), rW, lane * 16, so + i * 1024u);
        WAIT_V(0);
        __builtin_amdgcn_raw_buffer_store_b32(call, rW, lane == 0 ? 0 : OOB, OFF_FLAG + (un >> 2) * 512 + c * 16 + (un & 3) * 4, AUX_SYS);
    } else {
        call = __builtin_amdgcn_readfirstlane(call_v) + 1u;
    }
    if (wid == 4) *(volatile LAS unsigned*)(lds + 2 * LBUF) = 0u;

    const int xcd = b & 7, i30 = (b >> 3) - 2;
    const int kc = i30 < 28 ? (i30 >> 2) * 2 + (xcd & 1) : 14;
    const int ri = (xcd >> 1) * 4 + (i30 < 28 ? i30 & 3 : (xcd & 1) * 2 + (i30 - 28));
    if (wid < 4) gemm_role<0>(P, rW, lds, wid, lane, call, ri, kc, t0);
    else gemm_role<1>(P, rW, lds, wid, lane, call, ri, kc, t0);
    if (wid == 0) __builtin_amdgcn_raw_buffer_store_b32(call, rW, 0, OFF_CNT + b * 128, AUX_SYS);
}

}

extern "C" void kernel_launch(int rank, int nranks, ncclComm_t comm,
                              hipStream_t stream,
                              void* const* d_in, const int* in_sizes, int n_in,
                              void* d_out, int out_size,
                              const int* in_dtypes, int out_dtype,
                              void* const* d_out_all,
                              void* const* d_ws_all, size_t ws_bytes) {
    (void)comm; (void)in_dtypes; (void)out_dtype;
    if (nranks != 8 || (unsigned)rank >= 8u || n_in != 4 || ws_bytes < WS_NEED || out_size != M * N || in_sizes[0] != M * KSH || in_sizes[1] != KSH * N) {
        fprintf(stderr, "unexpected problem shape\n");
        return;
    }
    static bool attr_set = false;
    if (!attr_set) {
        HIP_CHECK(hipFuncSetAttribute((const void*)fused_kernel, hipFuncAttributeMaxDynamicSharedMemorySize, LDS_BYTES));
        attr_set = true;
    }
    Params p;
    p.x = (const signed char*)d_in[0];
    p.w = (const signed char*)d_in[1];
    p.sx = (const float*)d_in[2];
    p.sw = (const float*)d_in[3];
    p.ws0 = (char*)d_ws_all[rank];
    p.out0 = (char*)d_out;
    for (int i = 0; i < 8; ++i) {
        const int q = i < 7 ? i + 1 : 1, r = (rank + q) & 7;
        p.peer[i] = (char*)d_ws_all[r] + OFF_FLAG;
        p.peer[8 + i] = (char*)d_out_all[r];
        p.peer[16 + i] = (char*)d_ws_all[r] + OFF_STAGE + (size_t)(8 - q) * REGION_BYTES;
    }
    p.rank = rank;
    hipLaunchKernelGGL(fused_kernel, dim3(NWG), dim3(512), LDS_BYTES, stream, p);
    HIP_CHECK(hipGetLastError());
}
```

```cpp
#include <hip/hip_runtime.h>
#include <rccl/rccl.h>
#include <cstdio>

#define HIP_CHECK(expr) do { hipError_t _e = (expr); if (_e != hipSuccess) { \
    fprintf(stderr, "HIP error %s at %s:%d\n", hipGetErrorString(_e), __FILE__, __LINE__); \
    return; } } while (0)

namespace {

typedef int i32x4 __attribute__((ext_vector_type(4)));
typedef unsigned u32x4 __attribute__((ext_vector_type(4)));
typedef unsigned u32x2 __attribute__((ext_vector_type(2)));
typedef float f32x4 __attribute__((ext_vector_type(4)));
typedef float f32x2 __attribute__((ext_vector_type(2)));
typedef __amdgpu_buffer_rsrc_t rsrc_t;
#define LAS __attribute__((address_space(3)))
#define GAS __attribute__((address_space(1)))

constexpr int M = 4096, N = 16384, KSH = 512;
constexpr unsigned OUT_BYTES = (unsigned)((size_t)M * N * 4);
constexpr int NWG = 256, NSEND = 16;
constexpr int TOT = 128;
constexpr unsigned OFF_CNT = 0;
constexpr unsigned OFF_TMO = 32768;
constexpr unsigned OFF_TICKET = 32768 + 128;
constexpr unsigned OFF_TILE = 32768 + 1024;
constexpr unsigned OFF_FLAG = 65536;
constexpr unsigned FLAG_DONE = 80 * 512;
constexpr unsigned FLAG_BYTES = FLAG_DONE + 512;
constexpr unsigned OFF_DBG = 1u << 20;
constexpr unsigned OFF_STAGE = 4u << 20;
constexpr unsigned SHARD_BYTES = 10u << 20;
constexpr unsigned REGION_BYTES = 2u * SHARD_BYTES;
constexpr unsigned WS_NEED = OFF_STAGE + 8u * REGION_BYTES;
constexpr int LBUF = 65536;
constexpr int LDS_BYTES = 2 * LBUF + 64 + 1024;
constexpr unsigned DEADLINE_TICKS = 30000000u;
constexpr int FAR = 0x40000000;
constexpr int OOB = 0x7ffffff0;

#define STAMP 0
#define EXP_NOWAIT 0
#define EXP_NOPUSH 0
#define HEAD 5
#define EPI_A 2
#define EPI_B 5
#define PARK_LATE 1
#define OUT_POL "sc0 sc1"
#define PACK_DELAY 3
#define PACK_STEP 2
#define POLL_LOOK 3
#define POLL_SLEEP 4
#define AUX_SYS 17

constexpr unsigned long long SHARED_NEED = 0xffull | 0x03ull << 8 | 0x0cull << 16 | 0x30ull << 24 | 0xc0ull << 32;
constexpr int NIN = 1280, NOUT = 320, NUNITS = NIN + NOUT;
#define FIRST_ROWS {0, 0}, {0, 1}, {1, 0}, {1, 1}, {0, 2}, {1, 2}, {0, 3}, {1, 3}
constexpr int row_info(int row) {
    constexpr int first[8][2] = {FIRST_ROWS};
    int n = 0;
    for (int r = 0;; ++r) {
        const int g = r < 8 ? first[r][0] : r / 4, c = r < 8 ? first[r][1] : r % 4;
        if (r == row) return n | g << 16 | c << 20;
        n += g == 0 ? 64 : g == 2 ? 76 : 60;
    }
}
static_assert((row_info(20) & 0xffff) == NIN, "send rows");

struct Params {
    const signed char* x;
    const signed char* w;
    const float* sx;
    const float* sw;
    char* ws0;
    char* out0;
    char* peer[24];
    int rank;
};

#define WAIT_V(n) asm volatile("s_waitcnt vmcnt(%0)" ::"n"(n) : "memory")
#define WAIT_L0() __builtin_amdgcn_s_waitcnt(0xC07F)
#define BAR() __builtin_amdgcn_s_barrier()
#define SCHED() __builtin_amdgcn_sched_barrier(0)

__device__ __forceinline__ rsrc_t mk_rsrc(const void* p, unsigned bytes) { return __builtin_amdgcn_make_buffer_rsrc((void*)p, 0, bytes, 0x00020000); }
__device__ __forceinline__ unsigned now_ticks() { return (unsigned)__builtin_amdgcn_s_memrealtime(); }
__device__ __forceinline__ char* lane_ptr(char* p, int l) {
    const unsigned lo = __builtin_amdgcn_readlane((unsigned)(unsigned long long)p, l), hi = __builtin_amdgcn_readlane((unsigned)((unsigned long long)p >> 32), l);
    return (char*)(((unsigned long long)hi << 32) | lo);
}

__device__ __forceinline__ void store_out(f32x4 v, rsrc_t r, int vo, unsigned so) {
    asm volatile("buffer_store_dwordx4 %0, %1, %2, %3 offen " OUT_POL ::"v"(v), "v"(vo), "s"(r), "s"(so) : "memory");
}
__device__ __forceinline__ void store4k(i32x4 a, i32x4 b, i32x4 c, i32x4 d, rsrc_t r, int vo, unsigned so) {
    asm volatile("buffer_store_dwordx4 %0, %4, %5, %6 offen sc0 sc1\n\t"
                 "buffer_store_dwordx4 %1, %4, %5, %6 offen offset:1024 sc0 sc1\n\t"
                 "buffer_store_dwordx4 %2, %4, %5, %6 offen offset:2048 sc0 sc1\n\t"
                 "buffer_store_dwordx4 %3, %4, %5, %6 offen offset:3072 sc0 sc1\n\t"
                 "s_nop 1"
                 ::"v"(a), "v"(b), "v"(c), "v"(d), "v"(vo), "s"(r), "s"(so) : "memory");
}

__device__ __forceinline__ bool wait_expired(rsrc_t rW, unsigned call, unsigned w0) {
    const unsigned tv = __builtin_amdgcn_readfirstlane(__builtin_amdgcn_raw_buffer_load_b32(rW, 0, OFF_TMO, AUX_SYS));
    const bool dead = tv == call || now_ticks() - w0 > DEADLINE_TICKS;
    if (dead) __builtin_amdgcn_raw_buffer_store_b32(call, rW, 0, OFF_TMO, AUX_SYS);
    return dead;
}
template <int CNT>
__device__ __forceinline__ bool wait_words(rsrc_t rW, unsigned call, int vo, bool use, unsigned so, int step) {
    const unsigned w0 = now_ticks();
    for (unsigned spins = 1;; ++spins) {
        unsigned mn = 0xffffffffu;
#pragma unroll
        for (int i = 0; i < CNT; ++i) {
            const u32x4 f = __builtin_amdgcn_raw_buffer_load_b128(rW, use ? vo : OOB, so + i * step, AUX_SYS);
            mn = min(mn, min(min(f[0], f[1]), min(f[2], f[3])));
        }
        if (__ballot(use && mn < call) == 0ull) return true;
        if ((spins & 63u) == 0u && wait_expired(rW, call, w0)) return false;
        __builtin_amdgcn_s_sleep(8);
    }
}

__device__ __forceinline__ unsigned stage_off(int e) { return (unsigned)((e >> 8) & 7) << 21 | (unsigned)((e >> 2) & 3) << 19 | (unsigned)((e >> 4) & 15) << 15 | (unsigned)(e & 3) << 13; }

__device__ __forceinline__ void comm_role(const Params& P, rsrc_t rW, LAS char* lds, int ci, int b, int wid, int lane, unsigned call_v, unsigned t0) {
    volatile LAS unsigned* lds_w = (volatile LAS unsigned*)lds;
    const rsrc_t rX = mk_rsrc(P.x, M * KSH);
    const int voff = lane * 16 + wid * 1024, vout = lane * 16 + wid * (N * 4);
    const int vx = ((lane >> 3) + wid * 8) * KSH + (((lane & 7) ^ (lane >> 4) ^ ((wid & 1) << 2)) << 4);
    i32x4 X, Y;
    asm volatile("buffer_load_dwordx4 %0, %2, %3, %4 offen\n\tbuffer_load_dwordx4 %1, %2, %3, %5 offen"
                 : "=&v"(X), "=&v"(Y) : "v"(vx), "s"(rX), "s"(ci * 64 * KSH), "s"((NSEND + ci) * 64 * KSH) : "memory");
    char* const peer = P.peer[lane < 24 ? lane : 0];
    const int rank = P.rank & 7;
    const int ql = (lane & 7) + 1;
    if (wid == 2) lds_w[2] = 0u;
    unsigned long long kneed = 0;
    const int rot = (ci + wid) % 7;
#pragma unroll
    for (int k = 0; k < 7; ++k) {
        int q = k + rot; q = (q >= 7 ? q - 7 : q) + 1;
        kneed |= ((SHARED_NEED >> ((rank + q) & 7)) & 0x0101010101ull) << k;
    }
    const rsrc_t rO = mk_rsrc(P.out0, OUT_BYTES);
    const int vtick = lane == 0 ? 0 : OOB;

    int row_tab = 0xffff;
#pragma unroll
    for (int r = 0; r < 20; ++r) row_tab = lane == r ? row_info(r) : row_tab;
    auto code_of = [&](int n) {
        const int ri_ = __builtin_amdgcn_readlane(row_tab, __builtin_popcountll(__ballot((row_tab & 0xffff) <= n)) - 1);
        int i = n - (ri_ & 0xffff);
        const int g = (ri_ >> 16) & 7, c = ri_ >> 20;
        const int i5 = (i * 205) >> 10;
        const bool is_sh = g == 2 && (i - 5 * i5 == 4 || i == 75);
        const int sh = (i5 >> 2) | c << 2 | (31 + (i5 & 3) * 16) << 4 | ((i5 & 3) + 1) << 11;
        if (g == 2) i -= i5;
        return n >= NIN ? 0x4000 + n - NIN : is_sh ? sh : (i & 3) | c << 2 | (g * 16 + (i >> 2)) << 4;
    };
    auto out_dist = [&](int e) { const int i = e - 0x4000, j = (i * 205) >> 11; return i - 10 * j == 9 ? 7 - (j - ((j * 37) >> 8) * 7) : 0; };
    auto out_off = [&](int e) {
        const int i = e - 0x4000, j = (i * 205) >> 11, u = i - j, q = out_dist(e);
        const int row8 = q ? (8 + ((rank + q) & 7)) * 32 + j : u < 256 ? (rank & 1) * 256 + u : rank * 32 + u - 256;
        return (unsigned)(row8 * (8 * N * 4) + (q == 0 && u < 256 ? (rank >> 1) * 16 + 15 : 30) * 1024);
    };
    asm volatile("" : "+v"(row_tab), "+s"(kneed));
    SCHED();
    rsrc_t rD[7];
#pragma unroll
    for (int k = 0; k < 7; ++k) {
        int q = k + rot; q = q >= 7 ? q - 7 : q;
        rD[k] = mk_rsrc(lane_ptr(peer, 16 + q), EXP_NOPUSH ? 0u : REGION_BYTES);
    }
    const unsigned call = __builtin_amdgcn_readfirstlane(call_v) + 1u;
    const unsigned par_off = (call & 1u) * SHARD_BYTES;
    const unsigned tick_base = (call - 1u) * (unsigned)NUNITS - 3u * NSEND;
    auto flag_ptr = [&](int e) { return (GAS unsigned*)(peer + (((e >> 4) & 127) * 512 + (11 + 7 * ((e >> 2) & 3) - ql) * 16 + (e & 3) * 4)); };
    auto flag_on = [&](int e) { return lane < 7 && e < 0x4000 && ((SHARED_NEED >> (((e >> 11) & 7) * 8 + ((rank + ql) & 7))) & 1); };
    auto wait_packed = [&]() { if (!wait_words<4>(rW, call, lane * 512, true, OFF_FLAG + 16 * 512, 16)) lds_w[2] = 1u; };
    auto wait_tiles = [&]() { if (!wait_words<1>(rW, call, lane * 16, lane < 32, OFF_TILE, 0)) lds_w[2] = 1u; };
#define CST() do {} while (0)
#define CSHIFT(nn) do {} while (0)
#define ST1(D, k) asm volatile("buffer_store_dwordx4 %0, %1, %2, %3 offen sc0 sc1" ::"v"(D), "v"(vst), "s"(rD[k]), "s"(so[k]) : "memory")

    int e0 = (ci & 3) | (ci >> 2) << 4, e1 = e0 + 64;
    unsigned so[7];
#pragma unroll
    for (int k = 0; k < 7; ++k) so[k] = stage_off(e0) + par_off;
    GAS unsigned* fp = flag_ptr(e0);
    bool fon = false;
    asm volatile("s_waitcnt vmcnt(0)\n\ts_nop 4" : "+v"(X), "+v"(Y) :: "memory");
    int step = 0, vst = voff;
    unsigned tkv = 0;
    bool outs = false, tiles_ok = false;
    constexpr int CHECK_STEP = 5;
    const int vdraw = wid == 0 ? vtick : OOB;
#define STEP(D, O) do { \
        _Pragma("unroll") for (int k = 0; k < HEAD; ++k) ST1(D, k); \
        unsigned tks; \
        asm volatile("s_waitcnt vmcnt(%6)\n\tv_readfirstlane_b32 %0, %1\n\tv_mov_b32 %1, 1\n\tbuffer_atomic_add %1, %3, %4, %5 offen sc0 sc1" \
                     : "=&s"(tks), "+v"(tkv), "+v"(O) : "v"(vdraw), "s"(rW), "s"(OFF_TICKET), "n"(HEAD) : "memory"); \
        if (wid == 0) lds_w[step & 1] = tks; \
        if (wid == 2 && step == CHECK_STEP) wait_packed(); \
        WAIT_L0(); \
        BAR(); \
        if (fon) __hip_atomic_store(fp, call, __ATOMIC_RELAXED, __HIP_MEMORY_SCOPE_SYSTEM); \
        CST(); \
        _Pragma("unroll") for (int k = HEAD; k < 7; ++k) ST1(D, k); \
        asm volatile("s_nop 1" ::"v"(D)); \
        SCHED(); \
        const unsigned tk = __builtin_amdgcn_readfirstlane(lds_w[step & 1]); \
        const bool dead = __builtin_amdgcn_readfirstlane(lds_w[2]) != 0u; \
        const int nn = step > 0 ? (int)(tk - tick_base) : 2 * NSEND + ci; \
        const int e2 = code_of(nn); \
        { const bool isO = e2 >= 0x4000, isA = e2 < 256; \
          if (step < CHECK_STEP && !isA) wait_packed(); \
          if (isO && !tiles_ok) { wait_tiles(); tiles_ok = true; } \
          const rsrc_t rs = isO ? rO : isA ? rX : rW; const int vs = isO ? vout : isA ? vx : voff; \
          const unsigned src = e2 >= 0x4000 + NOUT ? (unsigned)FAR : isO ? out_off(e2) : isA ? (unsigned)(((e2 >> 4) * 4 + (e2 & 3)) * (64 * KSH) + ((e2 >> 2) & 3) * 128) : stage_off(e2) + par_off + OFF_STAGE; \
          asm volatile("buffer_load_dwordx4 %0, %1, %2, %3 offen sc1" : "=v"(D) : "v"(vs), "s"(rs), "s"(src) : "memory"); } \
        if (e1 >= 0x4000 && !outs) { \
            outs = true; vst = vout; \
            _Pragma("unroll") for (int k = 0; k < 7; ++k) { int q = k + rot; q = q >= 7 ? q - 7 : q; rD[k] = mk_rsrc(lane_ptr(peer, 8 + q), EXP_NOPUSH ? 0u : OUT_BYTES); } \
            asm volatile("s_nop 4"); \
        } \
        { const unsigned dst = outs ? out_off(e1) : stage_off(e1) + par_off; \
          int ko = out_dist(e1) - 1 - rot; ko = ko < 0 ? ko + 7 : ko; \
          const unsigned km = outs ? (out_dist(e1) ? 1u << ko : 127u) : (unsigned)(kneed >> ((e1 >> 11) * 8)); \
          _Pragma("unroll") for (int k = 0; k < 7; ++k) so[k] = (km >> k) & 1 ? dst : (unsigned)FAR; } \
        fp = flag_ptr(e0); \
        fon = wid == 1 && !dead && flag_on(e0); \
        CSHIFT(nn); \
        e0 = e1; e1 = e2; \
        ++step; \
    } while (0)
    for (;;) {
        STEP(X, Y);
        if (e0 >= 0x4000 + NOUT) break;
        STEP(Y, X);
        if (e0 >= 0x4000 + NOUT) break;
    }
    asm volatile("s_waitcnt vmcnt(0) lgkmcnt(0)" : "+v"(X), "+v"(Y), "+v"(tkv) :: "memory");
    BAR();
    CST();
    if (wid == 1 && __builtin_amdgcn_readfirstlane(lds_w[2]) == 0u) {
        if (fon) __hip_atomic_store(fp, call, __ATOMIC_RELAXED, __HIP_MEMORY_SCOPE_SYSTEM);
        if (lane < 7) __hip_atomic_store((GAS unsigned*)(peer + (FLAG_DONE + (8 - ql) * 64 + ci * 4)), call, __ATOMIC_RELAXED, __HIP_MEMORY_SCOPE_SYSTEM);
    }
    if (wid == 0) {
        if (ci == 0 && !EXP_NOWAIT && !wait_words<1>(rW, call, lane * 16, lane >= 4 && lane < 32, OFF_FLAG + FLAG_DONE, 0))
            __builtin_amdgcn_raw_buffer_store_b32(0x7fc00000u, rO, vtick, 0, AUX_SYS);
        __builtin_amdgcn_raw_buffer_store_b32(call, rW, 0, OFF_CNT + b * 128, AUX_SYS);
    }
    (void)t0;
}

template <int GRP>
__device__ __forceinline__ void gemm_role(const Params& P, rsrc_t rW, LAS char* lds, int wid, int lane, unsigned call,
                                          int ri, int kc, unsigned t0) {
    volatile LAS unsigned* lds_fail = (volatile LAS unsigned*)(lds + 2 * LBUF);
    const unsigned par_off = (call & 1u) * SHARD_BYTES;
    const int wc = wid & 3;
    const int voff = lane * 16;
#define STMP() do {} while (0)
    STMP();
    const float scale = P.sx[0] * P.sw[0];

    unsigned rb_p = (unsigned)ri * 0x01010101u, cb_p = (unsigned)kc * 0x01010101u + 0x30201000u;
    const bool shared = kc == 14;
    if (shared) {
        const int rank = P.rank & 7;
        rb_p = (rb_p & 0xffff00ffu) | (unsigned)(ri < 8 ? (rank & 1) * 8 + ri : ri == 8 + rank ? rank : ri) << 8;
        if (ri < 8) cb_p = (cb_p & 0xffff00ffu) | (unsigned)((rank >> 1) * 16 + 15) << 8;
    }

    int ready = (wid == 4 && !EXP_NOWAIT) ? 0 : 1 << 20, soft = ready, pend = -1;
    LAS char* const snap = lds + 2 * LBUF + 64;
    auto ensure_slow = [&](int need) {
        int vz = voff; asm volatile("" : "+v"(vz));
        bool use; int vf;
        auto aim = [&]() {
            const int round = ready >> 5, rb = (rb_p >> (8 * round)) & 255, cb = (cb_p >> (8 * round)) & 255;
            use = (vz & 511) >= (ready & 31) * 16;
            vf = use ? (vz & 511) + (vz < 512 ? 16 + cb : rb) * 512 : OOB;
        };
        auto take = [&](u32x4 f) {
            const unsigned long long bad = __ballot(use && min(min(f[0], f[1]), min(f[2], f[3])) < call);
            const unsigned bad32 = (unsigned)bad | (unsigned)(bad >> 32);
            ready = (ready & ~31) + (bad32 ? __builtin_ctz(bad32) : 32);
            ready = ready == TOT ? 1 << 20 : ready;
        };
        if (pend >= 0 && need >= pend) { aim(); take(*(volatile LAS u32x4*)(snap + vz)); pend = -1; }
        if (ready <= need) {
            const unsigned w0 = now_ticks();
            aim();
            for (unsigned spins = 1;; ++spins) {
                const int r0 = ready;
                take(__builtin_amdgcn_raw_buffer_load_b128(rW, vf, OFF_FLAG, AUX_SYS));
                if (ready > need) break;
                ready = r0;
                if ((spins & 63u) == 0u && wait_expired(rW, call, w0)) { lds_fail[0] = 1u; ready = 1 << 20; break; }
                __builtin_amdgcn_s_sleep(POLL_SLEEP);
            }
            pend = -1;
        } else if (pend < 0 && ready - need <= POLL_LOOK) {
            aim();
            __builtin_amdgcn_raw_ptr_buffer_load_lds(rW, snap, 16, vf, OFF_FLAG, 0, AUX_SYS);
            pend = need + (((need - 2) & 31) < 4 ? 2 : 1);
        }
        soft = pend >= 0 ? min(pend, ready) : max(ready - POLL_LOOK, need + 1);
    };
#define ENSURE(need) do { if (GRP == 1 && __builtin_expect(soft <= (need), 0)) ensure_slow(need); } while (0)

    const rsrc_t rS = mk_rsrc(P.ws0 + OFF_STAGE + par_off, 8u * REGION_BYTES - SHARD_BYTES);
    const rsrc_t rC = mk_rsrc(P.out0, OUT_BYTES);
    int src_tab;
    {
        const int s = lane & 31, u = s - 4, c = s < 4 ? s : (u * 37) >> 8;
        src_tab = (s < 4 ? 0 : (u - c * 7 + 1) * (int)REGION_BYTES) + (c << 19);
        if (lane >= 32) {
            const int j = lane & 7, rb = (rb_p >> (8 * (j & 3))) & 255, cb = (cb_p >> (8 * (j & 3))) & 255;
            src_tab = j >= 4 ? FAR : (lane & 8) ? (((cb >> 4) + 1) << 21) + (cb & 15) * 32768 + wid * 4096 : rb * 32768 + (GRP * 8 + wc) * 2048;
        }
    }
#define DMA(D, SO, IMM) __builtin_amdgcn_raw_ptr_buffer_load_lds(rS, D, 16, voff, SO, IMM, AUX_SYS)
#define ISSUE_B(so, buf) do { LAS char* d_ = lds + (buf) * LBUF + wid * 4096; DMA(d_, so, 0); DMA(d_, so, 1024); DMA(d_, so, 2048); DMA(d_, so, 3072); } while (0)
#define ISSUE_ALO(so, buf) do { LAS char* d_ = lds + (buf) * LBUF + 32768 + wid * 2048; DMA(d_, so, 0); DMA(d_, so, 1024); } while (0)
#define ISSUE_AHI(so, buf) do { LAS char* d_ = lds + (buf) * LBUF + 49152 + wid * 2048; DMA(d_, so, 0); DMA(d_, so, 1024); } while (0)

    i32x4 acc[8][4];
    i32x4 af[4][2], bf[4][2];
    const int rdA0 = 32768 + GRP * 8192 + (lane & 15) * 128 + (((lane >> 4) ^ ((lane >> 1) & 7)) << 4);
    const int rdA[2] = {rdA0, rdA0 ^ 64};
    const int rdB = wc * 4096 + lane * 16;

    ENSURE(1);
    BAR();
    int aj = __builtin_amdgcn_readlane(src_tab, 32), bj = __builtin_amdgcn_readlane(src_tab, 40);
    ISSUE_B(bj, 0); ISSUE_ALO(aj, 0); ISSUE_AHI(aj + 8192, 0);
    ISSUE_B(bj + (1 << 19), 1); ISSUE_ALO(aj + (1 << 19), 1);
    int oA1 = aj + (1 << 19) + 8192;
    const int vo = (voff >> 8) * 4 * N * 4 + (voff & 255);
    unsigned so0 = FAR;
    f32x2 sc2 = {0.f, 0.f};
#pragma unroll
    for (int m = 0; m < 8; ++m)
#pragma unroll
        for (int t = 0; t < 4; ++t) acc[m][t] = (i32x4){0, 0, 0, 0};
    {
        const f32x4 z = {};
#pragma unroll
        for (int m = 0; m < 4 * EPI_A; ++m) store_out(z, rC, vo, so0);
        asm volatile("s_nop 1" ::"v"(z));
    }
    WAIT_V(8 + 4 * EPI_A);
    BAR();
    if (GRP == 1) BAR();

#define MP(ROW, A_, t, k0, Z) do { \
        if (Z) asm volatile("v_mfma_i32_16x16x64_i8 %0, %1, %2, 0\n\tv_mfma_i32_16x16x64_i8 %0, %3, %4, %0" : "=&v"(acc[ROW][t]) : "v"(A_[k0]), "v"(bf[t][k0]), "v"(A_[1 - (k0)]), "v"(bf[t][1 - (k0)])); \
        else asm volatile("v_mfma_i32_16x16x64_i8 %0, %1, %2, %0\n\tv_mfma_i32_16x16x64_i8 %0, %3, %4, %0" : "+v"(acc[ROW][t]) : "v"(A_[k0]), "v"(bf[t][k0]), "v"(A_[1 - (k0)]), "v"(bf[t][1 - (k0)])); \
        SCHED(); } while (0)
#define MROW_UP(ROW, A_, Z) do { MP(ROW, A_, 0, 0, Z); MP(ROW, A_, 1, 1, Z); MP(ROW, A_, 2, 0, Z); MP(ROW, A_, 3, 1, Z); } while (0)
#define MROW_DN(ROW, A_, Z) do { MP(ROW, A_, 3, 0, Z); MP(ROW, A_, 2, 1, Z); MP(ROW, A_, 1, 0, Z); MP(ROW, A_, 0, 1, Z); } while (0)
#define PIN(x) do { asm volatile("" : "+s"(x)); SCHED(); } while (0)
#define SEG_WAIT(n) do { WAIT_V(n); WAIT_L0(); SCHED(); BAR(); SCHED(); } while (0)
    auto conv1 = [&](int q) {
        const int m = q >> 2, r = q & 3;
        const f32x2 lo2 = (f32x2){(float)acc[m][0][r], (float)acc[m][1][r]} * sc2, hi2 = (f32x2){(float)acc[m][2][r], (float)acc[m][3][r]} * sc2;
        return __builtin_elementwise_maximum((f32x4){lo2[0], lo2[1], hi2[0], hi2[1]}, (f32x4){0.f, 0.f, 0.f, 0.f});
    };
#define PUT1(V, q) store_out(V, rC, vo, so0 + (unsigned)(((q) >> 2) * 16 + ((q) & 3)) * (unsigned)(N * 4))
#define FIX(m) asm volatile("" : "+v"(acc[m][0]), "+v"(acc[m][1]), "+v"(acc[m][2]), "+v"(acc[m][3]))
#define CONV(O, m) do { SCHED(); FIX(m); _Pragma("unroll") for (int r = 0; r < 4; ++r) O[r] = conv1(4 * (m) + r); } while (0)
#define PUT(O, m) do { _Pragma("unroll") for (int r = 0; r < 4; ++r) PUT1(O[r], 4 * (m) + r); asm volatile("s_nop 1" ::"v"(O[2]), "v"(O[3])); SCHED(); } while (0)
#define EMIT(lo, hi) do { if ((lo) < (hi)) { SCHED(); _Pragma("unroll") for (int m = (lo); m < (hi); ++m) FIX(m); f32x4 o = conv1(4 * (lo)); \
        _Pragma("unroll") for (int q = 4 * (lo); q < 4 * (hi); ++q) { \
            SCHED(); PUT1(o, q); SCHED(); \
            if (q + 1 < 4 * (hi)) { f32x4 n_ = conv1(q + 1); asm volatile("" : "+v"(n_) : "v"(o)); o = n_; } \
            else asm volatile("s_nop 1" ::"v"(o)); } \
        SCHED(); } } while (0)
#define HOLD(O) asm volatile("" : "+v"(O[0]), "+v"(O[1]), "+v"(O[2]), "+v"(O[3]))
    constexpr int EA = EPI_A, EB = EPI_B;
    constexpr int SEG_ST[11] = {0, 0, 4 * EA, 4 * (EB - EA), 4 * (6 - EB), 4, PARK_LATE ? 0 : 4, PARK_LATE ? 4 : 0, 0, 0, 0};
#define CNT(i) (8 + SEG_ST[i] + SEG_ST[(i) - 1] + SEG_ST[(i) - 2])
    f32x4 p6[4], p7[4];
#define STAGE(buf, KIND) do { \
        LAS char* sb = lds + (buf) * LBUF; \
        ENSURE(g2); \
        _Pragma("unroll") for (int t = 0; t < 4; ++t) _Pragma("unroll") for (int kk = 0; kk < 2; ++kk) bf[t][kk] = *(LAS i32x4*)(sb + rdB + kk * 16384 + t * 1024); \
        _Pragma("unroll") for (int m = 0; m < 4; ++m) _Pragma("unroll") for (int kk = 0; kk < 2; ++kk) af[m][kk] = *(LAS i32x4*)(sb + rdA[kk] + m * 2048); \
        ISSUE_AHI(oA1, (buf) ^ 1); \
        if (KIND == 1) EMIT(EA, EB); \
        if (KIND == 2) PUT(p6, 6); \
        if (KIND == 3 && PARK_LATE) PUT(p7, 7); \
        SEG_WAIT(KIND >= 1 && KIND <= 4 ? CNT(2 * (KIND % 5) + 1 + (KIND == 0 || KIND == 5 ? 2 : 0)) : 8); \
        __builtin_amdgcn_s_setprio(1); \
        MP(0, af[0], 0, 0, KIND == 1); PIN(g2); int s2 = g2 & 31; PIN(s2); \
        MP(0, af[0], 1, 1, KIND == 1); int so2 = __builtin_amdgcn_readlane(src_tab, s2); PIN(so2); \
        MP(0, af[0], 2, 0, KIND == 1); int oA2 = aj + so2; PIN(oA2); int oB2 = bj + so2; PIN(oB2); \
        MP(0, af[0], 3, 1, KIND == 1); oA1 = oA2 + 8192; PIN(oA1); g2 += 1; PIN(g2); \
        MROW_DN(1, af[1], KIND == 1); MROW_UP(2, af[2], KIND == 1); MROW_DN(3, af[3], KIND == 1); \
        __builtin_amdgcn_s_setprio(0); \
        SCHED(); BAR(); SCHED(); \
        _Pragma("unroll") for (int m = 0; m < 4; ++m) _Pragma("unroll") for (int kk = 0; kk < 2; ++kk) af[m][kk] = *(LAS i32x4*)(sb + rdA[kk] + 16384 + m * 2048); \
        unsigned failed_ = 0; if (KIND == 5) failed_ = lds_fail[0]; \
        ISSUE_B(oB2, buf); ISSUE_ALO(oA2, buf); \
        if (KIND == 5) { \
            const int rb = (rb_p >> (8 * round)) & 255, cb = (cb_p >> (8 * round)) & 255; \
            so0 = (unsigned)((rb * 256 + GRP * 128) * N + cb * 256 + wc * 64) * 4u; \
            sc2[0] = sc2[1] = failed_ ? __builtin_nanf("") : scale; \
            EMIT(0, EA); \
        } \
        if (KIND == 1) { EMIT(EB, 6); CONV(p6, 6); CONV(p7, 7); HOLD(p6); HOLD(p7); SCHED(); } \
        if (KIND == 2 && !PARK_LATE) PUT(p7, 7); \
        SEG_WAIT(KIND != 0 ? CNT(2 * (KIND % 5) + 2) : 8); \
        __builtin_amdgcn_s_setprio(1); \
        if ((buf) == 1) { \
            MP(4, af[0], 0, 0, false); PIN(g2); int j2 = (g2 >> 5) + 32; PIN(j2); \
            MP(4, af[0], 1, 1, false); aj = __builtin_amdgcn_readlane(src_tab, j2); PIN(aj); j2 += 8; PIN(j2); \
            MP(4, af[0], 2, 0, false); bj = __builtin_amdgcn_readlane(src_tab, j2); PIN(bj); \
            MP(4, af[0], 3, 1, false); \
        } else { MROW_UP(4, af[0], KIND == 1); } \
        MROW_DN(5, af[1], KIND == 1); MROW_UP(6, af[2], KIND == 1); MROW_DN(7, af[3], KIND == 1); \
        __builtin_amdgcn_s_setprio(0); \
        SCHED(); BAR(); SCHED(); \
    } while (0)

    int g2 = 2;
    const int flag_at = shared ? 64 + 8 : -1;
    for (int round = 0; round < 4; ++round) {
        STAGE(0, 1);
        STAGE(1, 2);
        STAGE(0, 3);
        STAGE(1, 4);
        STMP();
        for (int s = 4; s < 30; s += 2) {
            if (__builtin_expect(g2 == flag_at, 0)) __builtin_amdgcn_raw_buffer_store_b32(call, rW, voff == 0 ? 0 : OOB, OFF_TILE + (ri * 8 + wid) * 4, AUX_SYS);
            STAGE(0, 0);
            STAGE(1, 0);
        }
        STMP();
        STAGE(0, 0);
        STAGE(1, 5);
    }
    asm volatile("s_nop 7\n\ts_nop 7"); SCHED();
    EMIT(EA, 8);
    STMP();
    WAIT_V(0);
    if (GRP == 0) BAR();
    (void)t0;
}

__global__ __launch_bounds__(512, 2) void fused_kernel(Params P) {
    extern __shared__ __attribute__((aligned(16))) char smem_[];
    LAS char* lds = (LAS char*)smem_;
    const unsigned t0 = STAMP ? now_ticks() : 0u;
    const int tid = threadIdx.x, lane = tid & 63;
    const int wid = __builtin_amdgcn_readfirstlane(tid >> 6);
    const int b = blockIdx.x;
    const rsrc_t rW = mk_rsrc(P.ws0, WS_NEED);
    const unsigned call_v = __builtin_amdgcn_raw_buffer_load_b32(rW, 0, OFF_CNT + b * 128, AUX_SYS);

    if (b < 16) {
        comm_role(P, rW, lds, b, b, wid, lane, call_v, t0);
        return;
    }

    const int n = wid * 240 + (b - 16);
    unsigned call;
    if (n < 1024) {
        const int j = n >> 8, c = (n >> 6) & 3, cqp = (n >> 2) & 15, part = n & 3;
        for (int i = 0; i < PACK_DELAY + PACK_STEP * wid; ++i) __builtin_amdgcn_s_sleep(16);
        const rsrc_t rWm = mk_rsrc(P.w, KSH * N);
        const int vw = (lane >> 4) * 16 * N + (lane & 15) * 4;
        const unsigned sw0 = (unsigned)((c * 128 + (part >> 1) * 64) * N + (j * 16 + cqp) * 256 + (part & 1) * 128);
        unsigned d[2][16];
#pragma unroll
        for (int i = 0; i < 16; ++i) {
            d[0][i] = __builtin_amdgcn_raw_buffer_load_b32(rWm, vw, sw0 + i * N, 0);
            d[1][i] = __builtin_amdgcn_raw_buffer_load_b32(rWm, vw + 64, sw0 + i * N, 0);
        }
        call = __builtin_amdgcn_readfirstlane(call_v) + 1u;
        const unsigned so = OFF_STAGE + (call & 1u) * SHARD_BYTES + (2u << 20) + (unsigned)n * 8192u;
#pragma unroll
        for (int h = 0; h < 2; ++h) {
            i32x4 pb[4];
#pragma unroll
            for (int dd = 0; dd < 4; ++dd) {
                const unsigned d0 = d[h][4 * dd], d1 = d[h][4 * dd + 1], d2 = d[h][4 * dd + 2], d3 = d[h][4 * dd + 3];
                const unsigned lo01 = __builtin_amdgcn_perm(d1, d0, 0x05010400u), hi01 = __builtin_amdgcn_perm(d1, d0, 0x07030602u);
                const unsigned lo23 = __builtin_amdgcn_perm(d3, d2, 0x05010400u), hi23 = __builtin_amdgcn_perm(d3, d2, 0x07030602u);
                pb[0][dd] = (int)__builtin_amdgcn_perm(lo23, lo01, 0x05040100u);
                pb[1][dd] = (int)__builtin_amdgcn_perm(lo23, lo01, 0x07060302u);
                pb[2][dd] = (int)__builtin_amdgcn_perm(hi23, hi01, 0x05040100u);
                pb[3][dd] = (int)__builtin_amdgcn_perm(hi23, hi01, 0x07060302u);
            }
            store4k(pb[0], pb[1], pb[2], pb[3], rW, lane * 16, so + h * 4096u);
        }
        WAIT_V(0);
        __builtin_amdgcn_raw_buffer_store_b32(call, rW, lane == 0 ? 0 : OOB, OFF_FLAG + (16 + j * 16 + cqp) * 512 + c * 16 + part * 4, AUX_SYS);
    } else if (n >= 1200 && n < 1456) {
        const int c = (n - 1200) >> 6, un = (n - 1200) & 63;
        const rsrc_t rX = mk_rsrc(P.x, M * KSH);
        const int vx0 = (lane >> 3) * KSH + (((lane & 7) ^ (lane >> 4)) << 4), vx1 = (vx0 ^ 64) + 8 * KSH;
        u32x4 d[8];
#pragma unroll
        for (int i = 0; i < 8; ++i) d[i] = __builtin_amdgcn_raw_buffer_load_b128(rX, (i & 1) ? vx1 : vx0, (un * 64 + (i >> 1) * 16) * KSH + c * 128, 0);
        call = __builtin_amdgcn_readfirstlane(call_v) + 1u;
        const unsigned so = OFF_STAGE + (call & 1u) * SHARD_BYTES + ((unsigned)c << 19) + (unsigned)un * 8192u;
#pragma unroll
        for (int i = 0; i < 8; i += 4)
            store4k(__builtin_bit_cast(i32x4, d[i]), __builtin_bit_cast(i32x4, d[i + 1]), __builtin_bit_cast(i32x4, d[i + 2]), __builtin_bit_cast(i32x4, d[i + 3]), rW, lane * 16, so + i * 1024u);
        WAIT_V(0);
        __builtin_amdgcn_raw_buffer_store_b32(call, rW, lane == 0 ? 0 : OOB, OFF_FLAG + (un >> 2) * 512 + c * 16 + (un & 3) * 4, AUX_SYS);
    } else {
        call = __builtin_amdgcn_readfirstlane(call_v) + 1u;
    }
    if (wid == 4) *(volatile LAS unsigned*)(lds + 2 * LBUF) = 0u;

    const int xcd = b & 7, i30 = (b >> 3) - 2;
    const int kc = i30 < 28 ? (i30 >> 2) * 2 + (xcd & 1) : 14;
    const int ri = (xcd >> 1) * 4 + (i30 < 28 ? i30 & 3 : (xcd & 1) * 2 + (i30 - 28));
    if (wid < 4) gemm_role<0>(P, rW, lds, wid, lane, call, ri, kc, t0);
    else gemm_role<1>(P, rW, lds, wid, lane, call, ri, kc, t0);
    if (wid == 0) __builtin_amdgcn_raw_buffer_store_b32(call, rW, 0, OFF_CNT + b * 128, AUX_SYS);
}

}

extern "C" void kernel_launch(int rank, int nranks, ncclComm_t comm,
                              hipStream_t stream,
                              void* const* d_in, const int* in_sizes, int n_in,
                              void* d_out, int out_size,
                              const int* in_dtypes, int out_dtype,
                              void* const* d_out_all,
                              void* const* d_ws_all, size_t ws_bytes) {
    (void)comm; (void)in_dtypes; (void)out_dtype;
    if (nranks != 8 || (unsigned)rank >= 8u || n_in != 4 || ws_bytes < WS_NEED || out_size != M * N || in_sizes[0] != M * KSH || in_sizes[1] != KSH * N) {
        fprintf(stderr, "unexpected problem shape\n");
        return;
    }
    static bool attr_set = false;
    if (!attr_set) {
        HIP_CHECK(hipFuncSetAttribute((const void*)fused_kernel, hipFuncAttributeMaxDynamicSharedMemorySize, LDS_BYTES));
        attr_set = true;
    }
    Params p;
    p.x = (const signed char*)d_in[0];
    p.w = (const signed char*)d_in[1];
    p.sx = (const float*)d_in[2];
    p.sw = (const float*)d_in[3];
    p.ws0 = (char*)d_ws_all[rank];
    p.out0 = (char*)d_out;
    for (int i = 0; i < 8; ++i) {
        const int q = i < 7 ? i + 1 : 1, r = (rank + q) & 7;
        p.peer[i] = (char*)d_ws_all[r] + OFF_FLAG;
        p.peer[8 + i] = (char*)d_out_all[r];
        p.peer[16 + i] = (char*)d_ws_all[r] + OFF_STAGE + (size_t)(8 - q) * REGION_BYTES;
    }
    p.rank = rank;
    hipLaunchKernelGGL(fused_kernel, dim3(NWG), dim3(512), LDS_BYTES, stream, p);
    HIP_CHECK(hipGetLastError());
}
```

```cpp
#include <hip/hip_runtime.h>
#include <rccl/rccl.h>
#include <cstdio>

#define HIP_CHECK(expr) do { hipError_t _e = (expr); if (_e != hipSuccess) { \
    fprintf(stderr, "HIP error %s at %s:%d\n", hipGetErrorString(_e), __FILE__, __LINE__); \
    return; } } while (0)

namespace {

typedef int i32x4 __attribute__((ext_vector_type(4)));
typedef unsigned u32x4 __attribute__((ext_vector_type(4)));
typedef unsigned u32x2 __attribute__((ext_vector_type(2)));
typedef float f32x4 __attribute__((ext_vector_type(4)));
typedef float f32x2 __attribute__((ext_vector_type(2)));
typedef __amdgpu_buffer_rsrc_t rsrc_t;
#define LAS __attribute__((address_space(3)))
#define GAS __attribute__((address_space(1)))

constexpr int M = 4096, N = 16384, KSH = 512;
constexpr unsigned OUT_BYTES = (unsigned)((size_t)M * N * 4);
constexpr int NWG = 256, NSEND = 16;
constexpr int TOT = 128;
constexpr unsigned OFF_CNT = 0;
constexpr unsigned OFF_TMO = 32768;
constexpr unsigned OFF_TICKET = 32768 + 128;
constexpr unsigned OFF_TILE = 32768 + 1024;
constexpr unsigned OFF_FLAG = 65536;
constexpr unsigned FLAG_DONE = 80 * 512;
constexpr unsigned FLAG_BYTES = FLAG_DONE + 512;
constexpr unsigned OFF_DBG = 1u << 20;
constexpr unsigned OFF_STAGE = 4u << 20;
constexpr unsigned SHARD_BYTES = 10u << 20;
constexpr unsigned REGION_BYTES = 2u * SHARD_BYTES;
constexpr unsigned WS_NEED = OFF_STAGE + 8u * REGION_BYTES;
constexpr int LBUF = 65536;
constexpr int LDS_BYTES = 2 * LBUF + 64 + 1024;
constexpr unsigned DEADLINE_TICKS = 30000000u;
constexpr int FAR = 0x40000000;
constexpr int OOB = 0x7ffffff0;

#define STAMP 0
#define EXP_NOWAIT 0
#define EXP_NOPUSH 0
#define HEAD 5
#define EPI_A 2
#define EPI_B 5
#define PARK_LATE 1
#define OUT_POL "sc0 sc1"
#define PACK_DELAY 3
#define PACK_STEP 2
#define PRIO 0
#define KC_MAP 0
#define POLL_LOOK 3
#define POLL_SLEEP 4
#define AUX_SYS 17

constexpr unsigned long long SHARED_NEED = 0xffull | 0x03ull << 8 | 0x0cull << 16 | 0x30ull << 24 | 0xc0ull << 32;
#define EXP_NOUT 320
#define EXP_SMASK 31
#define EXP_SKIPCOL 99
constexpr int NIN = 1280, NOUT = EXP_NOUT, NUNITS = NIN + NOUT;
#define FIRST_ROWS {0, 0}, {0, 1}, {1, 0}, {1, 1}, {0, 2}, {1, 2}, {0, 3}, {1, 3}
constexpr int row_info(int row) {
    constexpr int first[8][2] = {FIRST_ROWS};
    int n = 0;
    for (int r = 0;; ++r) {
        const int g = r < 8 ? first[r][0] : r / 4, c = r < 8 ? first[r][1] : r % 4;
        if (r == row) return n | g << 16 | c << 20;
        n += g == 0 ? 64 : g == 2 ? 76 : 60;
    }
}
static_assert((row_info(20) & 0xffff) == NIN, "send rows");

struct Params {
    const signed char* x;
    const signed char* w;
    const float* sx;
    const float* sw;
    char* ws0;
    char* out0;
    char* peer[24];
    int rank;
};

#define WAIT_V(n) asm volatile("s_waitcnt vmcnt(%0)" ::"n"(n) : "memory")
#define WAIT_L0() __builtin_amdgcn_s_waitcnt(0xC07F)
#define BAR() __builtin_amdgcn_s_barrier()
#define SCHED() __builtin_amdgcn_sched_barrier(0)

__device__ __forceinline__ rsrc_t mk_rsrc(const void* p, unsigned bytes) { return __builtin_amdgcn_make_buffer_rsrc((void*)p, 0, bytes, 0x00020000); }
__device__ __forceinline__ unsigned now_ticks() { return (unsigned)__builtin_amdgcn_s_memrealtime(); }
__device__ __forceinline__ char* lane_ptr(char* p, int l) {
    const unsigned lo = __builtin_amdgcn_readlane((unsigned)(unsigned long long)p, l), hi = __builtin_amdgcn_readlane((unsigned)((unsigned long long)p >> 32), l);
    return (char*)(((unsigned long long)hi << 32) | lo);
}

__device__ __forceinline__ void store_out(f32x4 v, rsrc_t r, int vo, unsigned so) {
    asm volatile("buffer_store_dwordx4 %0, %1, %2, %3 offen " OUT_POL ::"v"(v), "v"(vo), "s"(r), "s"(so) : "memory");
}
__device__ __forceinline__ void store4k(i32x4 a, i32x4 b, i32x4 c, i32x4 d, rsrc_t r, int vo, unsigned so) {
    asm volatile("buffer_store_dwordx4 %0, %4, %5, %6 offen sc0 sc1\n\t"
                 "buffer_store_dwordx4 %1, %4, %5, %6 offen offset:1024 sc0 sc1\n\t"
                 "buffer_store_dwordx4 %2, %4, %5, %6 offen offset:2048 sc0 sc1\n\t"
                 "buffer_store_dwordx4 %3, %4, %5, %6 offen offset:3072 sc0 sc1\n\t"
                 "s_nop 1"
                 ::"v"(a), "v"(b), "v"(c), "v"(d), "v"(vo), "s"(r), "s"(so) : "memory");
}

__device__ __forceinline__ bool wait_expired(rsrc_t rW, unsigned call, unsigned w0) {
    const unsigned tv = __builtin_amdgcn_readfirstlane(__builtin_amdgcn_raw_buffer_load_b32(rW, 0, OFF_TMO, AUX_SYS));
    const bool dead = tv == call || now_ticks() - w0 > DEADLINE_TICKS;
    if (dead) __builtin_amdgcn_raw_buffer_store_b32(call, rW, 0, OFF_TMO, AUX_SYS);
    return dead;
}
template <int CNT>
__device__ __forceinline__ bool wait_words(rsrc_t rW, unsigned call, int vo, bool use, unsigned so, int step) {
    const unsigned w0 = now_ticks();
    for (unsigned spins = 1;; ++spins) {
        unsigned mn = 0xffffffffu;
#pragma unroll
        for (int i = 0; i < CNT; ++i) {
            const u32x4 f = __builtin_amdgcn_raw_buffer_load_b128(rW, use ? vo : OOB, so + i * step, AUX_SYS);
            mn = min(mn, min(min(f[0], f[1]), min(f[2], f[3])));
        }
        if (__ballot(use && mn < call) == 0ull) return true;
        if ((spins & 63u) == 0u && wait_expired(rW, call, w0)) return false;
        __builtin_amdgcn_s_sleep(8);
    }
}

__device__ __forceinline__ unsigned stage_off(int e) { return (unsigned)((e >> 8) & 7) << 21 | (unsigned)((e >> 2) & 3) << 19 | (unsigned)((e >> 4) & 15) << 15 | (unsigned)(e & 3) << 13; }

__device__ __forceinline__ void comm_role(const Params& P, rsrc_t rW, LAS char* lds, int ci, int b, int wid, int lane, unsigned call_v, unsigned t0) {
    volatile LAS unsigned* lds_w = (volatile LAS unsigned*)lds;
    const rsrc_t rX = mk_rsrc(P.x, M * KSH);
    const int voff = lane * 16 + wid * 1024, vout = lane * 16 + wid * (N * 4);
    const int vx = ((lane >> 3) + wid * 8) * KSH + (((lane & 7) ^ (lane >> 4) ^ ((wid & 1) << 2)) << 4);
    i32x4 X, Y;
    asm volatile("buffer_load_dwordx4 %0, %2, %3, %4 offen\n\tbuffer_load_dwordx4 %1, %2, %3, %5 offen"
                 : "=&v"(X), "=&v"(Y) : "v"(vx), "s"(rX), "s"(ci * 64 * KSH), "s"((NSEND + ci) * 64 * KSH) : "memory");
    char* const peer = P.peer[lane < 24 ? lane : 0];
    const int rank = P.rank & 7;
    const int ql = (lane & 7) + 1;
    if (wid == 2) lds_w[2] = 0u;
    unsigned long long kneed = 0;
    const int rot = (ci + wid) % 7;
#pragma unroll
    for (int k = 0; k < 7; ++k) {
        int q = k + rot; q = (q >= 7 ? q - 7 : q) + 1;
        kneed |= ((SHARED_NEED >> ((rank + q) & 7)) & 0x0101010101ull) << k;
    }
    const rsrc_t rO = mk_rsrc(P.out0, OUT_BYTES);
    const int vtick = lane == 0 ? 0 : OOB;

    int row_tab = 0xffff;
#pragma unroll
    for (int r = 0; r < 20; ++r) row_tab = lane == r ? row_info(r) : row_tab;
    auto code_of = [&](int n) {
        const int ri_ = __builtin_amdgcn_readlane(row_tab, __builtin_popcountll(__ballot((row_tab & 0xffff) <= n)) - 1);
        int i = n - (ri_ & 0xffff);
        const int g = (ri_ >> 16) & 7, c = ri_ >> 20;
        const int i5 = (i * 205) >> 10;
        const bool is_sh = g == 2 && (i - 5 * i5 == 4 || i == 75);
        const int sh = (i5 >> 2) | c << 2 | (31 + (i5 & 3) * 16) << 4 | ((i5 & 3) + 1) << 11;
        if (g == 2) i -= i5;
        return n >= NIN ? 0x4000 + n - NIN : is_sh ? sh : (i & 3) | c << 2 | (g * 16 + (i >> 2)) << 4;
    };
    auto out_dist = [&](int e) { const int i = e - 0x4000, j = (i * 205) >> 11; return i - 10 * j == 9 ? 7 - (j - ((j * 37) >> 8) * 7) : 0; };
    auto out_off = [&](int e) {
        const int i = e - 0x4000, j = (i * 205) >> 11, u = i - j, q = out_dist(e);
        const int row8 = q ? (8 + ((rank + q) & 7)) * 32 + j : u < 256 ? (rank & 1) * 256 + u : rank * 32 + u - 256;
        return (unsigned)(row8 * (8 * N * 4) + (q == 0 && u < 256 ? (rank >> 1) * 16 + 15 : 30) * 1024);
    };
    asm volatile("" : "+v"(row_tab), "+s"(kneed));
    SCHED();
    rsrc_t rD[7];
#pragma unroll
    for (int k = 0; k < 7; ++k) {
        int q = k + rot; q = q >= 7 ? q - 7 : q;
        rD[k] = mk_rsrc(lane_ptr(peer, 16 + q), EXP_NOPUSH ? 0u : REGION_BYTES);
    }
    const unsigned call = __builtin_amdgcn_readfirstlane(call_v) + 1u;
    const unsigned par_off = (call & 1u) * SHARD_BYTES;
    const unsigned tick_base = (call - 1u) * (unsigned)NUNITS - 3u * NSEND;
    auto flag_ptr = [&](int e) { return (GAS unsigned*)(peer + (((e >> 4) & 127) * 512 + (11 + 7 * ((e >> 2) & 3) - ql) * 16 + (e & 3) * 4)); };
    auto flag_on = [&](int e) { return lane < 7 && e < 0x4000 && ((SHARED_NEED >> (((e >> 11) & 7) * 8 + ((rank + ql) & 7))) & 1); };
    auto wait_packed = [&]() { if (!wait_words<4>(rW, call, lane * 512, true, OFF_FLAG + 16 * 512, 16)) lds_w[2] = 1u; };
    auto wait_tiles = [&]() { if (!wait_words<1>(rW, call, lane * 16, lane < 32, OFF_TILE, 0)) lds_w[2] = 1u; };
#define CST() do {} while (0)
#define CSHIFT(nn) do {} while (0)
#define ST1(D, k) asm volatile("buffer_store_dwordx4 %0, %1, %2, %3 offen sc0 sc1" ::"v"(D), "v"(vst), "s"(rD[k]), "s"(so[k]) : "memory")

    int e0 = (ci & 3) | (ci >> 2) << 4, e1 = e0 + 64;
    unsigned so[7];
#pragma unroll
    for (int k = 0; k < 7; ++k) so[k] = stage_off(e0) + par_off;
    GAS unsigned* fp = flag_ptr(e0);
    bool fon = false;
    asm volatile("s_waitcnt vmcnt(0)\n\ts_nop 4" : "+v"(X), "+v"(Y) :: "memory");
    int step = 0, vst = voff;
    unsigned tkv = 0;
    bool outs = false, tiles_ok = false;
    constexpr int CHECK_STEP = 5;
    const int vdraw = wid == 0 ? vtick : OOB;
    int chk_step = CHECK_STEP;
    auto snap_ok = [&]() {
        unsigned mn = 0xffffffffu;
#pragma unroll
        for (int i = 0; i < 4; ++i) { const u32x4 f = *(volatile LAS u32x4*)(lds + 1024 + i * 1024 + lane * 16); mn = min(mn, min(min(f[0], f[1]), min(f[2], f[3]))); }
        return __ballot(mn < call) == 0ull;
    };
#define STEP(D, O) do { \
        _Pragma("unroll") for (int k = 0; k < HEAD; ++k) ST1(D, k); \
        unsigned tks; \
        asm volatile("s_waitcnt vmcnt(%6)\n\tv_readfirstlane_b32 %0, %1\n\tv_mov_b32 %1, 1\n\tbuffer_atomic_add %1, %3, %4, %5 offen sc0 sc1" \
                     : "=&s"(tks), "+v"(tkv), "+v"(O) : "v"(vdraw), "s"(rW), "s"(OFF_TICKET), "n"(HEAD) : "memory"); \
        if (wid == 0) lds_w[step & 1] = tks; \
        if (wid == 2 && step == chk_step && !snap_ok()) { if (step == CHECK_STEP) wait_packed(); else wait_tiles(); } \
        WAIT_L0(); \
        BAR(); \
        if (fon) __hip_atomic_store(fp, call, __ATOMIC_RELAXED, __HIP_MEMORY_SCOPE_SYSTEM); \
        CST(); \
        _Pragma("unroll") for (int k = HEAD; k < 7; ++k) ST1(D, k); \
        asm volatile("s_nop 1" ::"v"(D)); \
        SCHED(); \
        const unsigned tk = __builtin_amdgcn_readfirstlane(lds_w[step & 1]); \
        const bool dead = __builtin_amdgcn_readfirstlane(lds_w[2]) != 0u; \
        const int nn = step > 0 ? (int)(tk - tick_base) : 2 * NSEND + ci; \
        const int e2 = code_of(nn); \
        { const bool isO = e2 >= 0x4000, isA = e2 < 256; \
          if (step < CHECK_STEP && !isA) wait_packed(); \
          tiles_ok = tiles_ok || (step == chk_step && step > CHECK_STEP); \
          if (isO && !tiles_ok) { wait_tiles(); tiles_ok = true; } \
          if (step == CHECK_STEP - 1 || (nn >= NIN - 64 && chk_step == CHECK_STEP && step > CHECK_STEP)) { \
              chk_step = step + 1; \
              if (wid == 2) { const bool pk_ = step < CHECK_STEP; \
                  _Pragma("unroll") for (int i = 0; i < 4; ++i) \
                      __builtin_amdgcn_raw_ptr_buffer_load_lds(rW, lds + 1024 + i * 1024, 16, pk_ ? lane * 512 + i * 16 : (lane & 31) * 16, pk_ ? OFF_FLAG + 16 * 512 : OFF_TILE, 0, AUX_SYS); } } \
          const rsrc_t rs = isO ? rO : isA ? rX : rW; const int vs = isO ? vout : isA ? vx : voff; \
          const unsigned src = e2 >= 0x4000 + NOUT ? (unsigned)FAR : isO ? out_off(e2) : isA ? (unsigned)(((e2 >> 4) * 4 + (e2 & 3)) * (64 * KSH) + ((e2 >> 2) & 3) * 128) : stage_off(e2) + par_off + OFF_STAGE; \
          asm volatile("buffer_load_dwordx4 %0, %1, %2, %3 offen sc1" : "=v"(D) : "v"(vs), "s"(rs), "s"(src) : "memory"); } \
        if (e1 >= 0x4000 && !outs) { \
            outs = true; vst = vout; \
            _Pragma("unroll") for (int k = 0; k < 7; ++k) { int q = k + rot; q = q >= 7 ? q - 7 : q; rD[k] = mk_rsrc(lane_ptr(peer, 8 + q), EXP_NOPUSH ? 0u : OUT_BYTES); } \
            asm volatile("s_nop 4"); \
        } \
        { const unsigned dst = outs ? out_off(e1) : stage_off(e1) + par_off; \
          int ko = out_dist(e1) - 1 - rot; ko = ko < 0 ? ko + 7 : ko; \
          const unsigned km = outs ? (out_dist(e1) ? 1u << ko : 127u) : (unsigned)(kneed >> ((e1 >> 11) * 8)); \
          _Pragma("unroll") for (int k = 0; k < 7; ++k) so[k] = (km >> k) & 1 ? dst : (unsigned)FAR; } \
        fp = flag_ptr(e0); \
        fon = wid == 1 && !dead && flag_on(e0); \
        CSHIFT(nn); \
        e0 = e1; e1 = e2; \
        ++step; \
    } while (0)
    for (;;) {
        STEP(X, Y);
        if (e0 >= 0x4000 + NOUT) break;
        STEP(Y, X);
        if (e0 >= 0x4000 + NOUT) break;
    }
    asm volatile("s_waitcnt vmcnt(0) lgkmcnt(0)" : "+v"(X), "+v"(Y), "+v"(tkv) :: "memory");
    BAR();
    CST();
    if (wid == 1 && __builtin_amdgcn_readfirstlane(lds_w[2]) == 0u) {
        if (fon) __hip_atomic_store(fp, call, __ATOMIC_RELAXED, __HIP_MEMORY_SCOPE_SYSTEM);
        if (lane < 7) __hip_atomic_store((GAS unsigned*)(peer + (FLAG_DONE + (8 - ql) * 64 + ci * 4)), call, __ATOMIC_RELAXED, __HIP_MEMORY_SCOPE_SYSTEM);
    }
    if (wid == 0) {
        if (ci == 0 && !EXP_NOWAIT && !wait_words<1>(rW, call, lane * 16, lane >= 4 && lane < 32, OFF_FLAG + FLAG_DONE, 0))
            __builtin_amdgcn_raw_buffer_store_b32(0x7fc00000u, rO, vtick, 0, AUX_SYS);
        __builtin_amdgcn_raw_buffer_store_b32(call, rW, 0, OFF_CNT + b * 128, AUX_SYS);
    }
    (void)t0;
}

template <int GRP>
__device__ __forceinline__ void gemm_role(const Params& P, rsrc_t rW, LAS char* lds, int wid, int lane, unsigned call,
                                          int ri, int kc, unsigned t0) {
    volatile LAS unsigned* lds_fail = (volatile LAS unsigned*)(lds + 2 * LBUF);
    const unsigned par_off = (call & 1u) * SHARD_BYTES;
    const int wc = wid & 3;
    const int voff = lane * 16;
#define STMP() do {} while (0)
    STMP();
    const float scale = P.sx[0] * P.sw[0];

    unsigned rb_p = (unsigned)ri * 0x01010101u, cb_p = (unsigned)kc * 0x01010101u + 0x30201000u;
    const bool shared = kc == 14;
    if (shared) {
        const int rank = P.rank & 7;
        rb_p = (rb_p & 0xffff00ffu) | (unsigned)(ri < 8 ? (rank & 1) * 8 + ri : ri == 8 + rank ? rank : ri) << 8;
        if (ri < 8) cb_p = (cb_p & 0xffff00ffu) | (unsigned)((rank >> 1) * 16 + 15) << 8;
    }

    int ready = (wid == 4 && !EXP_NOWAIT) ? 0 : 1 << 20, soft = ready, pend = -1;
    LAS char* const snap = lds + 2 * LBUF + 64;
    auto ensure_slow = [&](int need) {
        int vz = voff; asm volatile("" : "+v"(vz));
        bool use; int vf;
        auto aim = [&]() {
            const int round = ready >> 5, rb = (rb_p >> (8 * round)) & 255, cb = (cb_p >> (8 * round)) & 255;
            use = (vz & 511) >= (ready & 31) * 16;
            vf = use ? (vz & 511) + (vz < 512 ? 16 + cb : rb) * 512 : OOB;
        };
        auto take = [&](u32x4 f) {
            const unsigned long long bad = __ballot(use && min(min(f[0], f[1]), min(f[2], f[3])) < call);
            const unsigned bad32 = (unsigned)bad | (unsigned)(bad >> 32);
            ready = (ready & ~31) + (bad32 ? __builtin_ctz(bad32) : 32);
            ready = ready == TOT ? 1 << 20 : ready;
        };
        if (pend >= 0 && need >= pend) { aim(); take(*(volatile LAS u32x4*)(snap + vz)); pend = -1; }
        if (ready <= need) {
            const unsigned w0 = now_ticks();
            aim();
            for (unsigned spins = 1;; ++spins) {
                const int r0 = ready;
                take(__builtin_amdgcn_raw_buffer_load_b128(rW, vf, OFF_FLAG, AUX_SYS));
                if (ready > need) break;
                ready = r0;
                if ((spins & 63u) == 0u && wait_expired(rW, call, w0)) { lds_fail[0] = 1u; ready = 1 << 20; break; }
                __builtin_amdgcn_s_sleep(POLL_SLEEP);
            }
            pend = -1;
        } else if (pend < 0 && ready - need <= POLL_LOOK) {
            aim();
            __builtin_amdgcn_raw_ptr_buffer_load_lds(rW, snap, 16, vf, OFF_FLAG, 0, AUX_SYS);
            pend = need + (((need - 2) & 31) < 4 ? 2 : 1);
        }
        soft = pend >= 0 ? min(pend, ready) : max(ready - POLL_LOOK, need + 1);
    };
#define ENSURE(need) do { if (GRP == 1 && __builtin_expect(soft <= (need), 0)) ensure_slow(need); } while (0)

    const rsrc_t rS = mk_rsrc(P.ws0 + OFF_STAGE + par_off, 8u * REGION_BYTES - SHARD_BYTES);
    const rsrc_t rC = mk_rsrc(P.out0, OUT_BYTES);
    int src_tab;
    {
        const int s = lane & EXP_SMASK, u = s - 4, c = s < 4 ? s : (u * 37) >> 8;
        src_tab = (s < 4 ? 0 : (u - c * 7 + 1) * (int)REGION_BYTES) + (c << 19);
        if (lane >= 32) {
            const int j = lane & 7, rb = (rb_p >> (8 * (j & 3))) & 255, cb = (cb_p >> (8 * (j & 3))) & 255;
            src_tab = (j >= 4 || (j == 3 && kc >= EXP_SKIPCOL)) ? FAR : (lane & 8) ? (((cb >> 4) + 1) << 21) + (cb & 15) * 32768 + wid * 4096 : rb * 32768 + (GRP * 8 + wc) * 2048;
        }
    }
#define DMA(D, SO, IMM) __builtin_amdgcn_raw_ptr_buffer_load_lds(rS, D, 16, voff, SO, IMM, AUX_SYS)
#define ISSUE_B(so, buf) do { LAS char* d_ = lds + (buf) * LBUF + wid * 4096; DMA(d_, so, 0); DMA(d_, so, 1024); DMA(d_, so, 2048); DMA(d_, so, 3072); } while (0)
#define ISSUE_ALO(so, buf) do { LAS char* d_ = lds + (buf) * LBUF + 32768 + wid * 2048; DMA(d_, so, 0); DMA(d_, so, 1024); } while (0)
#define ISSUE_AHI(so, buf) do { LAS char* d_ = lds + (buf) * LBUF + 49152 + wid * 2048; DMA(d_, so, 0); DMA(d_, so, 1024); } while (0)

    i32x4 acc[8][4];
    i32x4 af[4][2], bf[4][2];
    const int rdA0 = 32768 + GRP * 8192 + (lane & 15) * 128 + (((lane >> 4) ^ ((lane >> 1) & 7)) << 4);
    LAS char* rdA[2][2] = {{lds + rdA0, lds + (rdA0 ^ 64)}, {lds + LBUF + rdA0, lds + LBUF + (rdA0 ^ 64)}};
    LAS char* rdB[2] = {lds + wc * 4096 + lane * 16, lds + LBUF + wc * 4096 + lane * 16};
    asm volatile("" : "+v"(rdA[0][0]), "+v"(rdA[0][1]), "+v"(rdA[1][0]), "+v"(rdA[1][1]), "+v"(rdB[0]), "+v"(rdB[1]));

    ENSURE(1);
    BAR();
    int aj = __builtin_amdgcn_readlane(src_tab, 32), bj = __builtin_amdgcn_readlane(src_tab, 40);
    ISSUE_B(bj, 0); ISSUE_ALO(aj, 0); ISSUE_AHI(aj + 8192, 0);
    ISSUE_B(bj + (1 << 19), 1); ISSUE_ALO(aj + (1 << 19), 1);
    int oA1 = aj + (1 << 19) + 8192;
    const int vo = (voff >> 8) * 4 * N * 4 + (voff & 255);
    unsigned so0 = FAR;
    f32x2 sc2 = {0.f, 0.f};
#pragma unroll
    for (int m = 0; m < 8; ++m)
#pragma unroll
        for (int t = 0; t < 4; ++t) acc[m][t] = (i32x4){0, 0, 0, 0};
    {
        const f32x4 z = {};
#pragma unroll
        for (int m = 0; m < 4 * EPI_A; ++m) store_out(z, rC, vo, so0);
        asm volatile("s_nop 1" ::"v"(z));
    }
    WAIT_V(8 + 4 * EPI_A);
    BAR();
    if (GRP == 1) BAR();

#define MP(ROW, A_, t, k0, Z) do { \
        if (Z) asm volatile("v_mfma_i32_16x16x64_i8 %0, %1, %2, 0\n\tv_mfma_i32_16x16x64_i8 %0, %3, %4, %0" : "=&v"(acc[ROW][t]) : "v"(A_[k0]), "v"(bf[t][k0]), "v"(A_[1 - (k0)]), "v"(bf[t][1 - (k0)])); \
        else asm volatile("v_mfma_i32_16x16x64_i8 %0, %1, %2, %0\n\tv_mfma_i32_16x16x64_i8 %0, %3, %4, %0" : "+v"(acc[ROW][t]) : "v"(A_[k0]), "v"(bf[t][k0]), "v"(A_[1 - (k0)]), "v"(bf[t][1 - (k0)])); \
        SCHED(); } while (0)
#define MROW_UP(ROW, A_, Z) do { MP(ROW, A_, 0, 0, Z); MP(ROW, A_, 1, 1, Z); MP(ROW, A_, 2, 0, Z); MP(ROW, A_, 3, 1, Z); } while (0)
#define MROW_DN(ROW, A_, Z) do { MP(ROW, A_, 3, 0, Z); MP(ROW, A_, 2, 1, Z); MP(ROW, A_, 1, 0, Z); MP(ROW, A_, 0, 1, Z); } while (0)
#define PIN(x) do { asm volatile("" : "+s"(x)); SCHED(); } while (0)
#define SEG_WAIT(n) do { if (PRIO == 2) __builtin_amdgcn_s_setprio(1); WAIT_V(n); WAIT_L0(); SCHED(); BAR(); SCHED(); } while (0)
    auto conv1 = [&](int q) {
        const int m = q >> 2, r = q & 3;
        const f32x2 lo2 = (f32x2){(float)acc[m][0][r], (float)acc[m][1][r]} * sc2, hi2 = (f32x2){(float)acc[m][2][r], (float)acc[m][3][r]} * sc2;
        return __builtin_elementwise_maximum((f32x4){lo2[0], lo2[1], hi2[0], hi2[1]}, (f32x4){0.f, 0.f, 0.f, 0.f});
    };
#define PUT1(V, q) store_out(V, rC, vo, so0 + (unsigned)(((q) >> 2) * 16 + ((q) & 3)) * (unsigned)(N * 4))
#define FIX(m) asm volatile("" : "+v"(acc[m][0]), "+v"(acc[m][1]), "+v"(acc[m][2]), "+v"(acc[m][3]))
#define CONV(O, m) do { SCHED(); FIX(m); _Pragma("unroll") for (int r = 0; r < 4; ++r) O[r] = conv1(4 * (m) + r); } while (0)
#define PUT(O, m) do { _Pragma("unroll") for (int r = 0; r < 4; ++r) PUT1(O[r], 4 * (m) + r); asm volatile("s_nop 1" ::"v"(O[2]), "v"(O[3])); SCHED(); } while (0)
#define EMIT(lo, hi) do { if ((lo) < (hi)) { SCHED(); _Pragma("unroll") for (int m = (lo); m < (hi); ++m) FIX(m); f32x4 o = conv1(4 * (lo)); \
        _Pragma("unroll") for (int q = 4 * (lo); q < 4 * (hi); ++q) { \
            SCHED(); PUT1(o, q); SCHED(); \
            if (q + 1 < 4 * (hi)) { f32x4 n_ = conv1(q + 1); asm volatile("" : "+v"(n_) : "v"(o)); o = n_; } \
            else asm volatile("s_nop 1" ::"v"(o)); } \
        SCHED(); } } while (0)
#define HOLD(O) asm volatile("" : "+v"(O[0]), "+v"(O[1]), "+v"(O[2]), "+v"(O[3]))
    constexpr int EA = EPI_A, EB = EPI_B;
    constexpr int SEG_ST[11] = {0, 0, 4 * EA, 4 * (EB - EA), 4 * (6 - EB), 4, PARK_LATE ? 0 : 4, PARK_LATE ? 4 : 0, 0, 0, 0};
#define CNT(i) (8 + SEG_ST[i] + SEG_ST[(i) - 1] + SEG_ST[(i) - 2])
    f32x4 p6[4], p7[4];
#define STAGE(buf, KIND) do { \
        ENSURE(g2); \
        _Pragma("unroll") for (int t = 0; t < 4; ++t) _Pragma("unroll") for (int kk = 0; kk < 2; ++kk) bf[t][kk] = *(LAS i32x4*)(rdB[buf] + kk * 16384 + t * 1024); \
        _Pragma("unroll") for (int m = 0; m < 4; ++m) _Pragma("unroll") for (int kk = 0; kk < 2; ++kk) af[m][kk] = *(LAS i32x4*)(rdA[buf][kk] + m * 2048); \
        ISSUE_AHI(oA1, (buf) ^ 1); \
        PIN(g2); \
        const int so2 = __builtin_amdgcn_readlane(src_tab, g2 & 31); \
        int oA2 = aj + so2, oB2 = bj + so2; oA1 = oA2 + 8192; g2 += 1; \
        asm volatile("" : "+s"(oA2), "+s"(oB2), "+s"(oA1), "+s"(g2)); SCHED(); \
        if (KIND == 1) EMIT(EA, EB); \
        if (KIND == 2) PUT(p6, 6); \
        if (KIND == 3 && PARK_LATE) PUT(p7, 7); \
        SEG_WAIT(KIND >= 1 && KIND <= 4 ? CNT(2 * (KIND % 5) + 1 + (KIND == 0 || KIND == 5 ? 2 : 0)) : 8); \
        if (PRIO == 1) __builtin_amdgcn_s_setprio(1); \
        MROW_UP(0, af[0], KIND == 1); MROW_DN(1, af[1], KIND == 1); MROW_UP(2, af[2], KIND == 1); MROW_DN(3, af[3], KIND == 1); \
        if (PRIO == 1) __builtin_amdgcn_s_setprio(0); \
        SCHED(); BAR(); SCHED(); \
        if (PRIO == 2) __builtin_amdgcn_s_setprio(0); \
        _Pragma("unroll") for (int m = 0; m < 4; ++m) _Pragma("unroll") for (int kk = 0; kk < 2; ++kk) af[m][kk] = *(LAS i32x4*)(rdA[buf][kk] + 16384 + m * 2048); \
        unsigned failed_ = 0; if (KIND == 5) failed_ = lds_fail[0]; \
        ISSUE_B(oB2, buf); ISSUE_ALO(oA2, buf); \
        if (KIND == 5) { \
            const int rb = (rb_p >> (8 * round)) & 255, cb = (cb_p >> (8 * round)) & 255; \
            so0 = (unsigned)((rb * 256 + GRP * 128) * N + cb * 256 + wc * 64) * 4u; \
            sc2[0] = sc2[1] = failed_ ? __builtin_nanf("") : scale; \
            EMIT(0, EA); \
        } \
        if (KIND == 1) { EMIT(EB, 6); CONV(p6, 6); CONV(p7, 7); HOLD(p6); HOLD(p7); SCHED(); } \
        if (KIND == 2 && !PARK_LATE) PUT(p7, 7); \
        SEG_WAIT(KIND != 0 ? CNT(2 * (KIND % 5) + 2) : 8); \
        if (PRIO == 1) __builtin_amdgcn_s_setprio(1); \
        MROW_UP(4, af[0], KIND == 1); MROW_DN(5, af[1], KIND == 1); MROW_UP(6, af[2], KIND == 1); MROW_DN(7, af[3], KIND == 1); \
        if (PRIO == 1) __builtin_amdgcn_s_setprio(0); \
        SCHED(); BAR(); SCHED(); \
        if (PRIO == 2) __builtin_amdgcn_s_setprio(0); \
    } while (0)

    int g2 = 2;
    if (PRIO == 3 && GRP == 1) __builtin_amdgcn_s_setprio(1);
    for (int round = 0; round < 4; ++round) {
        STAGE(0, 1);
        STAGE(1, 2);
        STAGE(0, 3);
        STAGE(1, 4);
        if (shared && round == 2)
            __builtin_amdgcn_raw_buffer_store_b32(call, rW, voff == 0 ? 0 : OOB, OFF_TILE + (ri * 8 + wid) * 4, AUX_SYS);
        STMP();
        do {
            STAGE(0, 0);
            STAGE(1, 0);
        } while (g2 & 31);
        STMP();
        aj = __builtin_amdgcn_readlane(src_tab, 33 + round); bj = __builtin_amdgcn_readlane(src_tab, 41 + round);
        STAGE(0, 0);
        STAGE(1, 5);
    }
    asm volatile("s_nop 7\n\ts_nop 7"); SCHED();
    EMIT(EA, 8);
    STMP();
    WAIT_V(0);
    if (GRP == 0) BAR();
    (void)t0;
}

__global__ __launch_bounds__(512, 2) void fused_kernel(Params P) {
    extern __shared__ __attribute__((aligned(16))) char smem_[];
    LAS char* lds = (LAS char*)smem_;
    const unsigned t0 = STAMP ? now_ticks() : 0u;
    const int tid = threadIdx.x, lane = tid & 63;
    const int wid = __builtin_amdgcn_readfirstlane(tid >> 6);
    const int b = blockIdx.x;
    const rsrc_t rW = mk_rsrc(P.ws0, WS_NEED);
    const unsigned call_v = __builtin_amdgcn_raw_buffer_load_b32(rW, 0, OFF_CNT + b * 128, AUX_SYS);

    if (b < 16) {
        comm_role(P, rW, lds, b, b, wid, lane, call_v, t0);
        return;
    }

    const int n = wid * 240 + (b - 16);
    unsigned call;
    if (n < 1024) {
        const int j = n >> 8, c = (n >> 6) & 3, cqp = (n >> 2) & 15, part = n & 3;
        for (int i = 0; i < PACK_DELAY + PACK_STEP * wid; ++i) __builtin_amdgcn_s_sleep(16);
        const rsrc_t rWm = mk_rsrc(P.w, KSH * N);
        const int vw = (lane >> 4) * 16 * N + (lane & 15) * 4;
        const unsigned sw0 = (unsigned)((c * 128 + (part >> 1) * 64) * N + (j * 16 + cqp) * 256 + (part & 1) * 128);
        unsigned d[2][16];
#pragma unroll
        for (int i = 0; i < 16; ++i) {
            d[0][i] = __builtin_amdgcn_raw_buffer_load_b32(rWm, vw, sw0 + i * N, 0);
            d[1][i] = __builtin_amdgcn_raw_buffer_load_b32(rWm, vw + 64, sw0 + i * N, 0);
        }
        call = __builtin_amdgcn_readfirstlane(call_v) + 1u;
        const unsigned so = OFF_STAGE + (call & 1u) * SHARD_BYTES + (2u << 20) + (unsigned)n * 8192u;
#pragma unroll
        for (int h = 0; h < 2; ++h) {
            i32x4 pb[4];
#pragma unroll
            for (int dd = 0; dd < 4; ++dd) {
                const unsigned d0 = d[h][4 * dd], d1 = d[h][4 * dd + 1], d2 = d[h][4 * dd + 2], d3 = d[h][4 * dd + 3];
                const unsigned lo01 = __builtin_amdgcn_perm(d1, d0, 0x05010400u), hi01 = __builtin_amdgcn_perm(d1, d0, 0x07030602u);
                const unsigned lo23 = __builtin_amdgcn_perm(d3, d2, 0x05010400u), hi23 = __builtin_amdgcn_perm(d3, d2, 0x07030602u);
                pb[0][dd] = (int)__builtin_amdgcn_perm(lo23, lo01, 0x05040100u);
                pb[1][dd] = (int)__builtin_amdgcn_perm(lo23, lo01, 0x07060302u);
                pb[2][dd] = (int)__builtin_amdgcn_perm(hi23, hi01, 0x05040100u);
                pb[3][dd] = (int)__builtin_amdgcn_perm(hi23, hi01, 0x07060302u);
            }
            store4k(pb[0], pb[1], pb[2], pb[3], rW, lane * 16, so + h * 4096u);
        }
        WAIT_V(0);
        __builtin_amdgcn_raw_buffer_store_b32(call, rW, lane == 0 ? 0 : OOB, OFF_FLAG + (16 + j * 16 + cqp) * 512 + c * 16 + part * 4, AUX_SYS);
    } else if (n >= 1200 && n < 1456) {
        const int c = (n - 1200) >> 6, un = (n - 1200) & 63;
        const rsrc_t rX = mk_rsrc(P.x, M * KSH);
        const int vx0 = (lane >> 3) * KSH + (((lane & 7) ^ (lane >> 4)) << 4), vx1 = (vx0 ^ 64) + 8 * KSH;
        u32x4 d[8];
#pragma unroll
        for (int i = 0; i < 8; ++i) d[i] = __builtin_amdgcn_raw_buffer_load_b128(rX, (i & 1) ? vx1 : vx0, (un * 64 + (i >> 1) * 16) * KSH + c * 128, 0);
        call = __builtin_amdgcn_readfirstlane(call_v) + 1u;
        const unsigned so = OFF_STAGE + (call & 1u) * SHARD_BYTES + ((unsigned)c << 19) + (unsigned)un * 8192u;
#pragma unroll
        for (int i = 0; i < 8; i += 4)
            store4k(__builtin_bit_cast(i32x4, d[i]), __builtin_bit_cast(i32x4, d[i + 1]), __builtin_bit_cast(i32x4, d[i + 2]), __builtin_bit_cast(i32x4, d[i + 3]), rW, lane * 16, so + i * 1024u);
        WAIT_V(0);
        __builtin_amdgcn_raw_buffer_store_b32(call, rW, lane == 0 ? 0 : OOB, OFF_FLAG + (un >> 2) * 512 + c * 16 + (un & 3) * 4, AUX_SYS);
    } else {
        call = __builtin_amdgcn_readfirstlane(call_v) + 1u;
    }
    if (wid == 4) *(volatile LAS unsigned*)(lds + 2 * LBUF) = 0u;

    const int xcd = b & 7, i30 = (b >> 3) - 2;
    const int kc = i30 >= 28 ? 14 : KC_MAP == 0 ? (i30 >> 2) * 2 + (xcd & 1) : KC_MAP == 1 ? (i30 >> 2) * 2 + 1 - (xcd & 1)
                 : (int)(((xcd & 1) ? 0xa864210u : 0xdcb9753u) >> (i30 & ~3)) & 15;
    const int ri = (xcd >> 1) * 4 + (i30 < 28 ? i30 & 3 : (xcd & 1) * 2 + (i30 - 28));
    if (wid < 4) gemm_role<0>(P, rW, lds, wid, lane, call, ri, kc, t0);
    else gemm_role<1>(P, rW, lds, wid, lane, call, ri, kc, t0);
    if (wid == 0) __builtin_amdgcn_raw_buffer_store_b32(call, rW, 0, OFF_CNT + b * 128, AUX_SYS);
}

}

extern "C" void kernel_launch(int rank, int nranks, ncclComm_t comm,
                              hipStream_t stream,
                              void* const* d_in, const int* in_sizes, int n_in,
                              void* d_out, int out_size,
                              const int* in_dtypes, int out_dtype,
                              void* const* d_out_all,
                              void* const* d_ws_all, size_t ws_bytes) {
    (void)comm; (void)in_dtypes; (void)out_dtype;
    if (nranks != 8 || (unsigned)rank >= 8u || n_in != 4 || ws_bytes < WS_NEED || out_size != M * N || in_sizes[0] != M * KSH || in_sizes[1] != KSH * N) {
        fprintf(stderr, "unexpected problem shape\n");
        return;
    }
    static bool attr_set = false;
    if (!attr_set) {
        HIP_CHECK(hipFuncSetAttribute((const void*)fused_kernel, hipFuncAttributeMaxDynamicSharedMemorySize, LDS_BYTES));
        attr_set = true;
    }
    Params p;
    p.x = (const signed char*)d_in[0];
    p.w = (const signed char*)d_in[1];
    p.sx = (const float*)d_in[2];
    p.sw = (const float*)d_in[3];
    p.ws0 = (char*)d_ws_all[rank];
    p.out0 = (char*)d_out;
    for (int i = 0; i < 8; ++i) {
        const int q = i < 7 ? i + 1 : 1, r = (rank + q) & 7;
        p.peer[i] = (char*)d_ws_all[r] + OFF_FLAG;
        p.peer[8 + i] = (char*)d_out_all[r];
        p.peer[16 + i] = (char*)d_ws_all[r] + OFF_STAGE + (size_t)(8 - q) * REGION_BYTES;
    }
    p.rank = rank;
    hipLaunchKernelGGL(fused_kernel, dim3(NWG), dim3(512), LDS_BYTES, stream, p);
    HIP_CHECK(hipGetLastError());
}
```

```cpp
#include <hip/hip_runtime.h>
#include <rccl/rccl.h>
#include <cstdio>

#define HIP_CHECK(expr) do { hipError_t _e = (expr); if (_e != hipSuccess) { \
    fprintf(stderr, "HIP error %s at %s:%d\n", hipGetErrorString(_e), __FILE__, __LINE__); \
    return; } } while (0)

namespace {

typedef int i32x4 __attribute__((ext_vector_type(4)));
typedef unsigned u32x4 __attribute__((ext_vector_type(4)));
typedef float f32x4 __attribute__((ext_vector_type(4)));
typedef float f32x2 __attribute__((ext_vector_type(2)));
typedef __amdgpu_buffer_rsrc_t rsrc_t;
#define LAS __attribute__((address_space(3)))
#define GAS __attribute__((address_space(1)))

constexpr int M = 4096, N = 16384, KSH = 512;
constexpr unsigned OUT_BYTES = (unsigned)((size_t)M * N * 4);
constexpr int NWG = 256, NSEND = 16;
constexpr int TOT = 128;
constexpr unsigned OFF_CNT = 0;
constexpr unsigned OFF_TMO = 32768;
constexpr unsigned OFF_TICKET = 32768 + 128;
constexpr unsigned OFF_TILE = 32768 + 1024;
constexpr unsigned OFF_FLAG = 65536;
constexpr unsigned FLAG_DONE = 80 * 512;
constexpr unsigned OFF_STAGE = 4u << 20;
constexpr unsigned SHARD_BYTES = 10u << 20;
constexpr unsigned REGION_BYTES = 2u * SHARD_BYTES;
constexpr unsigned WS_NEED = OFF_STAGE + 8u * REGION_BYTES;
constexpr int LBUF = 65536;
constexpr int LDS_BYTES = 2 * LBUF + 64 + 1024;
constexpr unsigned DEADLINE_TICKS = 30000000u;
constexpr int FAR = 0x40000000;
constexpr int OOB = 0x7ffffff0;

constexpr int HEAD = 5;
constexpr int EA = 2, EB = 5;
constexpr int MERGE_FROM = 3;
constexpr int POLL_LOOK = 3;
#define AUX_SYS 17

constexpr unsigned long long SHARED_NEED = 0xffull | 0x03ull << 8 | 0x0cull << 16 | 0x30ull << 24 | 0xc0ull << 32;
constexpr int NIN = 1280, NOUT = 320, NUNITS = NIN + NOUT;
#define FIRST_ROWS {0, 0}, {0, 1}, {1, 0}, {1, 1}, {0, 2}, {1, 2}, {0, 3}, {1, 3}
constexpr int row_info(int row) {
    constexpr int first[8][2] = {FIRST_ROWS};
    int n = 0;
    for (int r = 0;; ++r) {
        const int g = r < 8 ? first[r][0] : r / 4, c = r < 8 ? first[r][1] : r % 4;
        if (r == row) return n | g << 16 | c << 20;
        n += g == 0 ? 64 : g == 2 ? 76 : 60;
    }
}
static_assert((row_info(20) & 0xffff) == NIN, "send rows");
#define SKEW 5
constexpr int skew_piece(int pos) {
    for (int c = 0; c < 4; ++c)
        for (int k = 0; k < 15; ++k) {
            int before = 0;
            for (int c2 = 0; c2 < 4; ++c2)
                for (int k2 = 0; k2 < 15; ++k2) before += (35 * c2 + SKEW * k2) * 4 + c2 < (35 * c + SKEW * k) * 4 + c ? 1 : 0;
            if (before == pos) return c << 4 | k;
        }
    return 0;
}
constexpr unsigned long long skew_word(int w) { unsigned long long r = 0; for (int i = 0; i < 10; ++i) r |= (unsigned long long)skew_piece(w * 10 + i) << (6 * i); return r; }

struct Params {
    const signed char* x;
    const signed char* w;
    const float* sx;
    const float* sw;
    char* ws0;
    char* out0;
    char* peer[24];
    int rank;
};

#define WAIT_V(n) asm volatile("s_waitcnt vmcnt(%0)" ::"n"(n) : "memory")
#define WAIT_L0() __builtin_amdgcn_s_waitcnt(0xC07F)
#define BAR() do { asm volatile("" ::: "memory"); __builtin_amdgcn_s_barrier(); asm volatile("" ::: "memory"); } while (0)
#define SCHED() __builtin_amdgcn_sched_barrier(0)

__device__ __forceinline__ rsrc_t mk_rsrc(const void* p, unsigned bytes) { return __builtin_amdgcn_make_buffer_rsrc((void*)p, 0, bytes, 0x00020000); }
__device__ __forceinline__ unsigned now_ticks() { return (unsigned)__builtin_amdgcn_s_memrealtime(); }
__device__ __forceinline__ char* lane_ptr(char* p, int l) {
    const unsigned lo = __builtin_amdgcn_readlane((unsigned)(unsigned long long)p, l), hi = __builtin_amdgcn_readlane((unsigned)((unsigned long long)p >> 32), l);
    return (char*)(((unsigned long long)hi << 32) | lo);
}

__device__ __forceinline__ void store_out(f32x4 v, rsrc_t r, int vo, unsigned so) {
    asm volatile("buffer_store_dwordx4 %0, %1, %2, %3 offen sc0 sc1" ::"v"(v), "v"(vo), "s"(r), "s"(so) : "memory");
}
__device__ __forceinline__ void store4k(i32x4 a, i32x4 b, i32x4 c, i32x4 d, rsrc_t r, int vo, unsigned so) {
    asm volatile("buffer_store_dwordx4 %0, %4, %5, %6 offen sc0 sc1\n\t"
                 "buffer_store_dwordx4 %1, %4, %5, %6 offen offset:1024 sc0 sc1\n\t"
                 "buffer_store_dwordx4 %2, %4, %5, %6 offen offset:2048 sc0 sc1\n\t"
                 "buffer_store_dwordx4 %3, %4, %5, %6 offen offset:3072 sc0 sc1\n\t"
                 "s_nop 1"
                 ::"v"(a), "v"(b), "v"(c), "v"(d), "v"(vo), "s"(r), "s"(so) : "memory");
}

__device__ __forceinline__ bool wait_expired(rsrc_t rW, unsigned call, unsigned w0) {
    const unsigned tv = __builtin_amdgcn_readfirstlane(__builtin_amdgcn_raw_buffer_load_b32(rW, 0, OFF_TMO, AUX_SYS));
    const bool dead = tv == call || now_ticks() - w0 > DEADLINE_TICKS;
    if (dead) __builtin_amdgcn_raw_buffer_store_b32(call, rW, 0, OFF_TMO, AUX_SYS);
    return dead;
}
template <int CNT>
__device__ __forceinline__ bool wait_words(rsrc_t rW, unsigned call, int vo, bool use, unsigned so, int step) {
    const unsigned w0 = now_ticks();
    for (unsigned spins = 1;; ++spins) {
        unsigned mn = 0xffffffffu;
#pragma unroll
        for (int i = 0; i < CNT; ++i) {
            const u32x4 f = __builtin_amdgcn_raw_buffer_load_b128(rW, use ? vo : OOB, so + i * step, AUX_SYS);
            mn = min(mn, min(min(f[0], f[1]), min(f[2], f[3])));
        }
        if (__ballot(use && mn < call) == 0ull) return true;
        if ((spins & 63u) == 0u && wait_expired(rW, call, w0)) return false;
        __builtin_amdgcn_s_sleep(8);
    }
}

__device__ __forceinline__ unsigned stage_off(int e) { return (unsigned)((e >> 8) & 7) << 21 | (unsigned)((e >> 2) & 3) << 19 | (unsigned)((e >> 4) & 15) << 15 | (unsigned)(e & 3) << 13; }

__device__ __forceinline__ void comm_role(const Params& P, rsrc_t rW, LAS char* lds, int ci, int b, int wid, int lane, unsigned call_v) {
    volatile LAS unsigned* lds_w = (volatile LAS unsigned*)lds;
    const rsrc_t rX = mk_rsrc(P.x, M * KSH);
    const int voff = lane * 16 + wid * 1024, vout = lane * 16 + wid * (N * 4);
    const int vx = ((lane >> 3) + wid * 8) * KSH + (((lane & 7) ^ (lane >> 4) ^ ((wid & 1) << 2)) << 4);
    i32x4 X, Y;
    asm volatile("buffer_load_dwordx4 %0, %2, %3, %4 offen\n\tbuffer_load_dwordx4 %1, %2, %3, %5 offen"
                 : "=&v"(X), "=&v"(Y) : "v"(vx), "s"(rX), "s"(ci * 64 * KSH), "s"((NSEND + ci) * 64 * KSH) : "memory");
    char* const peer = P.peer[lane < 24 ? lane : 0];
    const int rank = P.rank & 7;
    const int ql = (lane & 7) + 1;
    if (wid == 2) lds_w[2] = 0u;
    unsigned long long kneed = 0;
    const int rot = (ci + wid) % 7;
#pragma unroll
    for (int k = 0; k < 7; ++k) {
        int q = k + rot; q = (q >= 7 ? q - 7 : q) + 1;
        kneed |= ((SHARED_NEED >> ((rank + q) & 7)) & 0x0101010101ull) << k;
    }
    const rsrc_t rO = mk_rsrc(P.out0, OUT_BYTES);
    const int vtick = lane == 0 ? 0 : OOB;

    int row_tab = 0xffff;
#pragma unroll
    for (int r = 0; r < 20; ++r) row_tab = lane == r ? row_info(r) : row_tab;
    int skew_tab;
    {
        const int w = (lane * 26) >> 8;
        constexpr unsigned long long W0 = skew_word(0), W1 = skew_word(1), W2 = skew_word(2), W3 = skew_word(3), W4 = skew_word(4), W5 = skew_word(5);
        const unsigned long long word = w == 0 ? W0 : w == 1 ? W1 : w == 2 ? W2 : w == 3 ? W3 : w == 4 ? W4 : W5;
        skew_tab = (int)(word >> ((lane - 10 * w) * 6)) & 63;
    }
    auto code_of = [&](int n) {
        const int ri_ = __builtin_amdgcn_readlane(row_tab, __builtin_popcountll(__ballot((row_tab & 0xffff) <= n)) - 1);
        int i = n - (ri_ & 0xffff), g = (ri_ >> 16) & 7;
        int c = ri_ >> 20;
        if (c >= MERGE_FROM && g < 2) {
            const int p = (i >> 2) + g * 16, h = p >> 1;
            g = p & 1;
            i = (i & 3) | (g ? h : (h & 3) * 4 + (h >> 2)) << 2;
        }
        const int i5 = (i * 205) >> 10;
        const bool is_sh = g == 2 && (i - 5 * i5 == 4 || i == 75);
        const int sh = (i5 >> 2) | c << 2 | (31 + (i5 & 3) * 16) << 4 | ((i5 & 3) + 1) << 11;
        if (g == 2) {
            i -= i5;
            const int sp = __builtin_amdgcn_readlane(skew_tab, c * 15 + (i >> 2));
            c = sp >> 4; i = (i & 3) | (sp & 15) << 2;
        }
        return n >= NIN ? 0x4000 + n - NIN : is_sh ? sh : (i & 3) | c << 2 | (g * 16 + (i >> 2)) << 4;
    };
    auto out_dist = [&](int e) { const int i = e - 0x4000, j = (i * 205) >> 11; return i - 10 * j == 9 ? 7 - (j - ((j * 37) >> 8) * 7) : 0; };
    auto out_off = [&](int e) {
        const int i = e - 0x4000, j = (i * 205) >> 11, u = i - j, q = out_dist(e);
        const int row8 = q ? (8 + ((rank + q) & 7)) * 32 + j : u < 256 ? (rank & 1) * 256 + u : rank * 32 + u - 256;
        return (unsigned)(row8 * (8 * N * 4) + (q == 0 && u < 256 ? (rank >> 1) * 16 + 15 : 30) * 1024);
    };
    asm volatile("" : "+v"(row_tab), "+v"(skew_tab)); asm volatile("" : "+s"(kneed));
    SCHED();
    rsrc_t rD[7];
#pragma unroll
    for (int k = 0; k < 7; ++k) {
        int q = k + rot; q = q >= 7 ? q - 7 : q;
        rD[k] = mk_rsrc(lane_ptr(peer, 16 + q), REGION_BYTES);
    }
    const unsigned call = __builtin_amdgcn_readfirstlane(call_v) + 1u;
    const unsigned par_off = (call & 1u) * SHARD_BYTES;
    const unsigned tick_base = (call - 1u) * (unsigned)NUNITS - 3u * NSEND;
    auto flag_ptr = [&](int e) { return (GAS unsigned*)(peer + (((e >> 4) & 127) * 512 + (11 + 7 * ((e >> 2) & 3) - ql) * 16 + (e & 3) * 4)); };
    auto flag_on = [&](int e) { return lane < 7 && e < 0x4000 && ((SHARED_NEED >> (((e >> 11) & 7) * 8 + ((rank + ql) & 7))) & 1); };
    auto wait_packed = [&]() { if (!wait_words<4>(rW, call, lane * 512, true, OFF_FLAG + 16 * 512, 16)) lds_w[2] = 1u; };
    auto wait_tiles = [&]() { if (!wait_words<1>(rW, call, lane * 16, lane < 32, OFF_TILE, 0)) lds_w[2] = 1u; };
#define ST1(D, k) asm volatile("s_nop 4\n\tbuffer_store_dwordx4 %0, %1, %2, %3 offen sc0 sc1" ::"v"(D), "v"(vst), "s"(rD[k]), "s"(so[k]) : "memory")

    int e0 = (ci & 3) | (ci >> 2) << 4, e1 = e0 + 64;
    unsigned so[7];
#pragma unroll
    for (int k = 0; k < 7; ++k) so[k] = stage_off(e0) + par_off;
    GAS unsigned* fp = flag_ptr(e0);
    bool fon = false;
    asm volatile("s_waitcnt vmcnt(0)\n\ts_nop 4" : "+v"(X), "+v"(Y) :: "memory");
    int step = 0, vst = voff;
    unsigned tkv = 0;
    bool outs = false, tiles_ok = false;
    constexpr int CHECK_STEP = 5;
    const int vdraw = wid == 0 ? vtick : OOB;
    int chk_step = CHECK_STEP;
    auto snap_ok = [&]() {
        unsigned mn = 0xffffffffu;
#pragma unroll
        for (int i = 0; i < 4; ++i) { const u32x4 f = *(volatile LAS u32x4*)(lds + 1024 + i * 1024 + lane * 16); mn = min(mn, min(min(f[0], f[1]), min(f[2], f[3]))); }
        return __ballot(mn < call) == 0ull;
    };
#define STEP(D, O) do { \
        _Pragma("unroll") for (int k = 0; k < HEAD; ++k) ST1(D, k); \
        unsigned tks; \
        asm volatile("s_waitcnt vmcnt(%6)\n\tv_readfirstlane_b32 %0, %1\n\tv_mov_b32 %1, 1\n\ts_nop 1\n\tbuffer_atomic_add %1, %3, %4, %5 offen sc0 sc1" \
                     : "=&s"(tks), "+v"(tkv), "+v"(O) : "v"(vdraw), "s"(rW), "s"(OFF_TICKET), "n"(HEAD) : "memory"); \
        if (wid == 0) lds_w[step & 1] = tks; \
        if (wid == 2 && step == chk_step && !snap_ok()) { if (step == CHECK_STEP) wait_packed(); else wait_tiles(); } \
        WAIT_L0(); \
        BAR(); \
        if (fon) __hip_atomic_store(fp, call, __ATOMIC_RELAXED, __HIP_MEMORY_SCOPE_SYSTEM); \
        _Pragma("unroll") for (int k = HEAD; k < 7; ++k) ST1(D, k); \
        asm volatile("s_nop 1" ::"v"(D)); \
        SCHED(); \
        const unsigned tk = __builtin_amdgcn_readfirstlane(lds_w[step & 1]); \
        const bool dead = __builtin_amdgcn_readfirstlane(lds_w[2]) != 0u; \
        const int nn = step > 0 ? (int)(tk - tick_base) : 2 * NSEND + ci; \
        const int e2 = code_of(nn); \
        { const bool isO = e2 >= 0x4000, isA = e2 < 256; \
          if (step < CHECK_STEP && !isA) wait_packed(); \
          tiles_ok = tiles_ok || (step == chk_step && step > CHECK_STEP); \
          if (isO && !tiles_ok) { wait_tiles(); tiles_ok = true; } \
          if (step == CHECK_STEP - 1 || (nn >= NIN - 64 && chk_step == CHECK_STEP && step > CHECK_STEP)) { \
              chk_step = step + 1; \
              if (wid == 2) { const bool pk_ = step < CHECK_STEP; \
                  _Pragma("unroll") for (int i = 0; i < 4; ++i) \
                      __builtin_amdgcn_raw_ptr_buffer_load_lds(rW, lds + 1024 + i * 1024, 16, pk_ ? lane * 512 + i * 16 : (lane & 31) * 16, pk_ ? OFF_FLAG + 16 * 512 : OFF_TILE, 0, AUX_SYS); } } \
          const rsrc_t rs = isO ? rO : isA ? rX : rW; const int vs = isO ? vout : isA ? vx : voff; \
          const unsigned src = e2 >= 0x4000 + NOUT ? (unsigned)FAR : isO ? out_off(e2) : isA ? (unsigned)(((e2 >> 4) * 4 + (e2 & 3)) * (64 * KSH) + ((e2 >> 2) & 3) * 128) : stage_off(e2) + par_off + OFF_STAGE; \
          asm volatile("s_nop 4\n\tbuffer_load_dwordx4 %0, %1, %2, %3 offen sc1" : "=v"(D) : "v"(vs), "s"(rs), "s"(src) : "memory"); } \
        if (e1 >= 0x4000 && !outs) { \
            outs = true; vst = vout; \
            _Pragma("unroll") for (int k = 0; k < 7; ++k) { int q = k + rot; q = q >= 7 ? q - 7 : q; rD[k] = mk_rsrc(lane_ptr(peer, 8 + q), OUT_BYTES); } \
        } \
        { const unsigned dst = outs ? out_off(e1) : stage_off(e1) + par_off; \
          int ko = out_dist(e1) - 1 - rot; ko = ko < 0 ? ko + 7 : ko; \
          const unsigned km = outs ? (out_dist(e1) ? 1u << ko : 127u) : (unsigned)(kneed >> ((e1 >> 11) * 8)); \
          _Pragma("unroll") for (int k = 0; k < 7; ++k) so[k] = (km >> k) & 1 ? dst : (unsigned)FAR; } \
        fp = flag_ptr(e0); \
        fon = wid == 1 && !dead && flag_on(e0); \
        e0 = e1; e1 = e2; \
        ++step; \
    } while (0)
    for (;;) {
        STEP(X, Y);
        if (e0 >= 0x4000 + NOUT) break;
        STEP(Y, X);
        if (e0 >= 0x4000 + NOUT) break;
    }
    asm volatile("s_waitcnt vmcnt(0) lgkmcnt(0)" : "+v"(X), "+v"(Y), "+v"(tkv) :: "memory");
    BAR();
    if (wid == 1 && __builtin_amdgcn_readfirstlane(lds_w[2]) == 0u) {
        if (fon) __hip_atomic_store(fp, call, __ATOMIC_RELAXED, __HIP_MEMORY_SCOPE_SYSTEM);
        if (lane < 7) __hip_atomic_store((GAS unsigned*)(peer + (FLAG_DONE + (8 - ql) * 64 + ci * 4)), call, __ATOMIC_RELAXED, __HIP_MEMORY_SCOPE_SYSTEM);
    }
    if (wid == 0) {
        if (ci == 0 && !wait_words<1>(rW, call, lane * 16, lane >= 4 && lane < 32, OFF_FLAG + FLAG_DONE, 0))
            __builtin_amdgcn_raw_buffer_store_b32(0x7fc00000u, rO, vtick, 0, AUX_SYS);
        __builtin_amdgcn_raw_buffer_store_b32(call, rW, 0, OFF_CNT + b * 128, AUX_SYS);
    }
}

template <int GRP>
__device__ __forceinline__ void gemm_role(const Params& P, rsrc_t rW, LAS char* lds, int wid, int lane, unsigned call,
                                          int ri, int kc) {
    volatile LAS unsigned* lds_fail = (volatile LAS unsigned*)(lds + 2 * LBUF);
    const unsigned par_off = (call & 1u) * SHARD_BYTES;
    const int wc = wid & 3;
    const int voff = lane * 16;
    const float scale = P.sx[0] * P.sw[0];

    unsigned rb_p = (unsigned)ri * 0x01010101u, cb_p = (unsigned)kc * 0x01010101u + 0x30201000u;
    const bool shared = kc == 14;
    if (shared) {
        const int rank = P.rank & 7;
        rb_p = (rb_p & 0xffff00ffu) | (unsigned)(ri < 8 ? (rank & 1) * 8 + ri : ri == 8 + rank ? rank : ri) << 8;
        if (ri < 8) cb_p = (cb_p & 0xffff00ffu) | (unsigned)((rank >> 1) * 16 + 15) << 8;
    }

    int ready = wid == 4 ? 0 : 1 << 20, soft = ready, pend = -1;
    LAS char* const snap = lds + 2 * LBUF + 64;
    auto ensure_slow = [&](int need) {
        int vz = voff; asm volatile("" : "+v"(vz));
        bool use; int vf;
        auto aim = [&]() {
            const int round = ready >> 5, rb = (rb_p >> (8 * round)) & 255, cb = (cb_p >> (8 * round)) & 255;
            use = (vz & 511) >= (ready & 31) * 16;
            vf = use ? (vz & 511) + (vz < 512 ? 16 + cb : rb) * 512 : OOB;
        };
        auto take = [&](u32x4 f) {
            const unsigned long long bad = __ballot(use && min(min(f[0], f[1]), min(f[2], f[3])) < call);
            const unsigned bad32 = (unsigned)bad | (unsigned)(bad >> 32);
            ready = (ready & ~31) + (bad32 ? __builtin_ctz(bad32) : 32);
            ready = ready == TOT ? 1 << 20 : ready;
        };
        if (pend >= 0 && need >= pend) { aim(); take(*(volatile LAS u32x4*)(snap + vz)); pend = -1; }
        if (ready <= need) {
            const unsigned w0 = now_ticks();
            aim();
            for (unsigned spins = 1;; ++spins) {
                const int r0 = ready;
                take(__builtin_amdgcn_raw_buffer_load_b128(rW, vf, OFF_FLAG, AUX_SYS));
                if (ready > need) break;
                ready = r0;
                if ((spins & 63u) == 0u && wait_expired(rW, call, w0)) { lds_fail[0] = 1u; ready = 1 << 20; break; }
                __builtin_amdgcn_s_sleep(4);
            }
            pend = -1;
        } else if (pend < 0 && ready - need <= POLL_LOOK) {
            aim();
            __builtin_amdgcn_raw_ptr_buffer_load_lds(rW, snap, 16, vf, OFF_FLAG, 0, AUX_SYS);
            pend = need + (((need - 2) & 31) < 4 ? 2 : 1);
        }
        soft = pend >= 0 ? min(pend, ready) : max(ready - POLL_LOOK, need + 1);
    };
#define ENSURE(need) do { if (GRP == 1 && __builtin_expect(soft <= (need), 0)) ensure_slow(need); } while (0)

    const rsrc_t rS = mk_rsrc(P.ws0 + OFF_STAGE + par_off, 8u * REGION_BYTES - SHARD_BYTES);
    const rsrc_t rC = mk_rsrc(P.out0, OUT_BYTES);
    int src_tab;
    {
        const int s = lane & 31, u = s - 4, c = s < 4 ? s : (u * 37) >> 8;
        src_tab = (s < 4 ? 0 : (u - c * 7 + 1) * (int)REGION_BYTES) + (c << 19);
        if (lane >= 32) {
            const int j = lane & 7, rb = (rb_p >> (8 * (j & 3))) & 255, cb = (cb_p >> (8 * (j & 3))) & 255;
            src_tab = j >= 4 ? FAR : (lane & 8) ? (((cb >> 4) + 1) << 21) + (cb & 15) * 32768 + wid * 4096 : rb * 32768 + (GRP * 8 + wc) * 2048;
        }
    }
#define DMA(D, SO, IMM) __builtin_amdgcn_raw_ptr_buffer_load_lds(rS, D, 16, voff, SO, IMM, AUX_SYS)
#define ISSUE_B(so, buf) do { LAS char* d_ = lds + (buf) * LBUF + wid * 4096; DMA(d_, so, 0); DMA(d_, so, 1024); DMA(d_, so, 2048); DMA(d_, so, 3072); } while (0)
#define ISSUE_ALO(so, buf) do { LAS char* d_ = lds + (buf) * LBUF + 32768 + wid * 2048; DMA(d_, so, 0); DMA(d_, so, 1024); } while (0)
#define ISSUE_AHI(so, buf) do { LAS char* d_ = lds + (buf) * LBUF + 49152 + wid * 2048; DMA(d_, so, 0); DMA(d_, so, 1024); } while (0)

    i32x4 acc[8][4];
    i32x4 af[4][2], bf[4][2];
    const int rdA0 = 32768 + GRP * 8192 + (lane & 15) * 128 + (((lane >> 4) ^ ((lane >> 1) & 7)) << 4);
    LAS char* rdA[2][2] = {{lds + rdA0, lds + (rdA0 ^ 64)}, {lds + LBUF + rdA0, lds + LBUF + (rdA0 ^ 64)}};
    LAS char* rdB[2] = {lds + wc * 4096 + lane * 16, lds + LBUF + wc * 4096 + lane * 16};
    asm volatile("" : "+v"(rdA[0][0]), "+v"(rdA[0][1]), "+v"(rdA[1][0]), "+v"(rdA[1][1]), "+v"(rdB[0]), "+v"(rdB[1]));

    ENSURE(1);
    BAR();
    int aj = __builtin_amdgcn_readlane(src_tab, 32), bj = __builtin_amdgcn_readlane(src_tab, 40);
    ISSUE_B(bj, 0); ISSUE_ALO(aj, 0); ISSUE_AHI(aj + 8192, 0);
    ISSUE_B(bj + (1 << 19), 1); ISSUE_ALO(aj + (1 << 19), 1);
    int oA1 = aj + (1 << 19) + 8192;
    const int vo = (voff >> 8) * 4 * N * 4 + (voff & 255);
    unsigned so0 = FAR;
    f32x2 sc2 = {0.f, 0.f};
#pragma unroll
    for (int m = 0; m < 8; ++m)
#pragma unroll
        for (int t = 0; t < 4; ++t) acc[m][t] = (i32x4){0, 0, 0, 0};
    {
        const f32x4 z = {};
#pragma unroll
        for (int m = 0; m < 4 * EA; ++m) store_out(z, rC, vo, so0);
        asm volatile("s_nop 1" ::"v"(z));
    }
    WAIT_V(8 + 4 * EA);
    BAR();
    if (GRP == 1) BAR();

#define MP(ROW, A_, t, k0, Z) do { \
        if (Z) asm volatile("v_mfma_i32_16x16x64_i8 %0, %1, %2, 0\n\tv_mfma_i32_16x16x64_i8 %0, %3, %4, %0" : "=&v"(acc[ROW][t]) : "v"(A_[k0]), "v"(bf[t][k0]), "v"(A_[1 - (k0)]), "v"(bf[t][1 - (k0)])); \
        else asm volatile("v_mfma_i32_16x16x64_i8 %0, %1, %2, %0\n\tv_mfma_i32_16x16x64_i8 %0, %3, %4, %0" : "+v"(acc[ROW][t]) : "v"(A_[k0]), "v"(bf[t][k0]), "v"(A_[1 - (k0)]), "v"(bf[t][1 - (k0)])); \
        SCHED(); } while (0)
#define MROW_UP(ROW, A_, Z) do { MP(ROW, A_, 0, 0, Z); MP(ROW, A_, 1, 1, Z); MP(ROW, A_, 2, 0, Z); MP(ROW, A_, 3, 1, Z); } while (0)
#define MROW_DN(ROW, A_, Z) do { MP(ROW, A_, 3, 0, Z); MP(ROW, A_, 2, 1, Z); MP(ROW, A_, 1, 0, Z); MP(ROW, A_, 0, 1, Z); } while (0)
#define PIN(x) do { asm volatile("" : "+s"(x)); SCHED(); } while (0)
#define SEG_WAIT(n) do { WAIT_V(n); WAIT_L0(); SCHED(); BAR(); SCHED(); } while (0)
    auto conv1 = [&](int q) {
        const int m = q >> 2, r = q & 3;
        const f32x2 lo2 = (f32x2){(float)acc[m][0][r], (float)acc[m][1][r]} * sc2, hi2 = (f32x2){(float)acc[m][2][r], (float)acc[m][3][r]} * sc2;
        return __builtin_elementwise_maximum((f32x4){lo2[0], lo2[1], hi2[0], hi2[1]}, (f32x4){0.f, 0.f, 0.f, 0.f});
    };
#define PUT1(V, q) store_out(V, rC, vo, so0 + (unsigned)(((q) >> 2) * 16 + ((q) & 3)) * (unsigned)(N * 4))
#define FIX(m) asm volatile("" : "+v"(acc[m][0]), "+v"(acc[m][1]), "+v"(acc[m][2]), "+v"(acc[m][3]))
#define CONV(O, m) do { SCHED(); FIX(m); _Pragma("unroll") for (int r = 0; r < 4; ++r) O[r] = conv1(4 * (m) + r); } while (0)
#define PUT(O, m) do { _Pragma("unroll") for (int r = 0; r < 4; ++r) PUT1(O[r], 4 * (m) + r); asm volatile("s_nop 1" ::"v"(O[2]), "v"(O[3])); SCHED(); } while (0)
#define EMIT(lo, hi) do { if ((lo) < (hi)) { SCHED(); _Pragma("unroll") for (int m = (lo); m < (hi); ++m) FIX(m); f32x4 o = conv1(4 * (lo)); \
        _Pragma("unroll") for (int q = 4 * (lo); q < 4 * (hi); ++q) { \
            SCHED(); PUT1(o, q); SCHED(); \
            if (q + 1 < 4 * (hi)) { f32x4 n_ = conv1(q + 1); asm volatile("" : "+v"(n_) : "v"(o)); o = n_; } \
            else asm volatile("s_nop 1" ::"v"(o)); } \
        SCHED(); } } while (0)
#define HOLD(O) asm volatile("" : "+v"(O[0]), "+v"(O[1]), "+v"(O[2]), "+v"(O[3]))
    constexpr int SEG_ST[11] = {0, 0, 4 * EA, 4 * (EB - EA), 4 * (6 - EB), 4, 0, 4, 0, 0, 0};
#define CNT(i) (8 + SEG_ST[i] + SEG_ST[(i) - 1] + SEG_ST[(i) - 2])
    f32x4 p6[4], p7[4];
#define STAGE(buf, KIND) do { \
        ENSURE(g2); \
        _Pragma("unroll") for (int t = 0; t < 4; ++t) _Pragma("unroll") for (int kk = 0; kk < 2; ++kk) bf[t][kk] = *(LAS i32x4*)(rdB[buf] + kk * 16384 + t * 1024); \
        _Pragma("unroll") for (int m = 0; m < 4; ++m) _Pragma("unroll") for (int kk = 0; kk < 2; ++kk) af[m][kk] = *(LAS i32x4*)(rdA[buf][kk] + m * 2048); \
        ISSUE_AHI(oA1, (buf) ^ 1); \
        PIN(g2); \
        const int so2 = __builtin_amdgcn_readlane(src_tab, g2 & 31); \
        int oA2 = aj + so2, oB2 = bj + so2; oA1 = oA2 + 8192; g2 += 1; \
        asm volatile("" : "+s"(oA2), "+s"(oB2), "+s"(oA1), "+s"(g2)); SCHED(); \
        if (KIND == 1) EMIT(EA, EB); \
        if (KIND == 2) PUT(p6, 6); \
        if (KIND == 3) PUT(p7, 7); \
        SEG_WAIT(KIND >= 1 && KIND <= 4 ? CNT(2 * (KIND % 5) + 1 + (KIND == 0 || KIND == 5 ? 2 : 0)) : 8); \
        MROW_UP(0, af[0], KIND == 1); MROW_DN(1, af[1], KIND == 1); MROW_UP(2, af[2], KIND == 1); MROW_DN(3, af[3], KIND == 1); \
        SCHED(); BAR(); SCHED(); \
        _Pragma("unroll") for (int m = 0; m < 4; ++m) _Pragma("unroll") for (int kk = 0; kk < 2; ++kk) af[m][kk] = *(LAS i32x4*)(rdA[buf][kk] + 16384 + m * 2048); \
        unsigned failed_ = 0; if (KIND == 5) failed_ = lds_fail[0]; \
        ISSUE_B(oB2, buf); ISSUE_ALO(oA2, buf); \
        if (KIND == 5) { \
            const int rb = (rb_p >> (8 * round)) & 255, cb = (cb_p >> (8 * round)) & 255; \
            so0 = (unsigned)((rb * 256 + GRP * 128) * N + cb * 256 + wc * 64) * 4u; \
            sc2[0] = sc2[1] = failed_ ? __builtin_nanf("") : scale; \
            EMIT(0, EA); \
        } \
        if (KIND == 1) { EMIT(EB, 6); CONV(p6, 6); CONV(p7, 7); HOLD(p6); HOLD(p7); SCHED(); } \
        SEG_WAIT(KIND != 0 ? CNT(2 * (KIND % 5) + 2) : 8); \
        MROW_UP(4, af[0], KIND == 1); MROW_DN(5, af[1], KIND == 1); MROW_UP(6, af[2], KIND == 1); MROW_DN(7, af[3], KIND == 1); \
        SCHED(); BAR(); SCHED(); \
    } while (0)

    int g2 = 2;
    for (int round = 0; round < 4; ++round) {
        STAGE(0, 1);
        STAGE(1, 2);
        STAGE(0, 3);
        STAGE(1, 4);
        if (shared && round == 2)
            __builtin_amdgcn_raw_buffer_store_b32(call, rW, voff == 0 ? 0 : OOB, OFF_TILE + (ri * 8 + wid) * 4, AUX_SYS);
        do {
            STAGE(0, 0);
            STAGE(1, 0);
        } while (g2 & 31);
        aj = __builtin_amdgcn_readlane(src_tab, 33 + round); bj = __builtin_amdgcn_readlane(src_tab, 41 + round);
        STAGE(0, 0);
        STAGE(1, 5);
    }
    asm volatile("s_nop 7\n\ts_nop 7"); SCHED();
    EMIT(EA, 8);
    WAIT_V(0);
    if (GRP == 0) BAR();
}

__global__ __launch_bounds__(512, 2) void fused_kernel(Params P) {
    extern __shared__ __attribute__((aligned(16))) char smem_[];
    LAS char* lds = (LAS char*)smem_;
    const int tid = threadIdx.x, lane = tid & 63;
    const int wid = __builtin_amdgcn_readfirstlane(tid >> 6);
    const int b = blockIdx.x;
    const rsrc_t rW = mk_rsrc(P.ws0, WS_NEED);
    const unsigned call_v = __builtin_amdgcn_raw_buffer_load_b32(rW, 0, OFF_CNT + b * 128, AUX_SYS);

    if (b < 16) {
        comm_role(P, rW, lds, b, b, wid, lane, call_v);
        return;
    }

    const int n = wid * 240 + (b - 16);
    unsigned call;
    if (n < 1024) {
        const int j = n >> 8, c = (n >> 6) & 3, cqp = (n >> 2) & 15, part = n & 3;
        for (int i = 0; i < 3 + 2 * wid; ++i) __builtin_amdgcn_s_sleep(16);
        const rsrc_t rWm = mk_rsrc(P.w, KSH * N);
        const int vw = (lane >> 4) * 16 * N + (lane & 15) * 4;
        const unsigned sw0 = (unsigned)((c * 128 + (part >> 1) * 64) * N + (j * 16 + cqp) * 256 + (part & 1) * 128);
        unsigned d[2][16];
#pragma unroll
        for (int i = 0; i < 16; ++i) {
            d[0][i] = __builtin_amdgcn_raw_buffer_load_b32(rWm, vw, sw0 + i * N, 0);
            d[1][i] = __builtin_amdgcn_raw_buffer_load_b32(rWm, vw + 64, sw0 + i * N, 0);
        }
        call = __builtin_amdgcn_readfirstlane(call_v) + 1u;
        const unsigned so = OFF_STAGE + (call & 1u) * SHARD_BYTES + (2u << 20) + (unsigned)n * 8192u;
#pragma unroll
        for (int h = 0; h < 2; ++h) {
            i32x4 pb[4];
#pragma unroll
            for (int dd = 0; dd < 4; ++dd) {
                const unsigned d0 = d[h][4 * dd], d1 = d[h][4 * dd + 1], d2 = d[h][4 * dd + 2], d3 = d[h][4 * dd + 3];
                const unsigned lo01 = __builtin_amdgcn_perm(d1, d0, 0x05010400u), hi01 = __builtin_amdgcn_perm(d1, d0, 0x07030602u);
                const unsigned lo23 = __builtin_amdgcn_perm(d3, d2, 0x05010400u), hi23 = __builtin_amdgcn_perm(d3, d2, 0x07030602u);
                pb[0][dd] = (int)__builtin_amdgcn_perm(lo23, lo01, 0x05040100u);
                pb[1][dd] = (int)__builtin_amdgcn_perm(lo23, lo01, 0x07060302u);
                pb[2][dd] = (int)__builtin_amdgcn_perm(hi23, hi01, 0x05040100u);
                pb[3][dd] = (int)__builtin_amdgcn_perm(hi23, hi01, 0x07060302u);
            }
            store4k(pb[0], pb[1], pb[2], pb[3], rW, lane * 16, so + h * 4096u);
        }
        WAIT_V(0);
        __builtin_amdgcn_raw_buffer_store_b32(call, rW, lane == 0 ? 0 : OOB, OFF_FLAG + (16 + j * 16 + cqp) * 512 + c * 16 + part * 4, AUX_SYS);
    } else if (n >= 1200 && n < 1456) {
        const int c = (n - 1200) >> 6, un = (n - 1200) & 63;
        const rsrc_t rX = mk_rsrc(P.x, M * KSH);
        const int vx0 = (lane >> 3) * KSH + (((lane & 7) ^ (lane >> 4)) << 4), vx1 = (vx0 ^ 64) + 8 * KSH;
        u32x4 d[8];
#pragma unroll
        for (int i = 0; i < 8; ++i) d[i] = __builtin_amdgcn_raw_buffer_load_b128(rX, (i & 1) ? vx1 : vx0, (un * 64 + (i >> 1) * 16) * KSH + c * 128, 0);
        call = __builtin_amdgcn_readfirstlane(call_v) + 1u;
        const unsigned so = OFF_STAGE + (call & 1u) * SHARD_BYTES + ((unsigned)c << 19) + (unsigned)un * 8192u;
#pragma unroll
        for (int i = 0; i < 8; i += 4)
            store4k(__builtin_bit_cast(i32x4, d[i]), __builtin_bit_cast(i32x4, d[i + 1]), __builtin_bit_cast(i32x4, d[i + 2]), __builtin_bit_cast(i32x4, d[i + 3]), rW, lane * 16, so + i * 1024u);
        WAIT_V(0);
        __builtin_amdgcn_raw_buffer_store_b32(call, rW, lane == 0 ? 0 : OOB, OFF_FLAG + (un >> 2) * 512 + c * 16 + (un & 3) * 4, AUX_SYS);
    } else {
        call = __builtin_amdgcn_readfirstlane(call_v) + 1u;
    }
    if (wid == 4) *(volatile LAS unsigned*)(lds + 2 * LBUF) = 0u;

    const int xcd = b & 7, i30 = (b >> 3) - 2;
    const int kc = i30 < 28 ? (i30 >> 2) * 2 + (xcd & 1) : 14;
    const int ri = (xcd >> 1) * 4 + (i30 < 28 ? i30 & 3 : (xcd & 1) * 2 + (i30 - 28));
    if (wid < 4) gemm_role<0>(P, rW, lds, wid, lane, call, ri, kc);
    else gemm_role<1>(P, rW, lds, wid, lane, call, ri, kc);
    if (wid == 0) __builtin_amdgcn_raw_buffer_store_b32(call, rW, 0, OFF_CNT + b * 128, AUX_SYS);
}

}

extern "C" void kernel_launch(int rank, int nranks, ncclComm_t comm,
                              hipStream_t stream,
                              void* const* d_in, const int* in_sizes, int n_in,
                              void* d_out, int out_size,
                              const int* in_dtypes, int out_dtype,
                              void* const* d_out_all,
                              void* const* d_ws_all, size_t ws_bytes) {
    (void)comm; (void)in_dtypes; (void)out_dtype;
    if (nranks != 8 || (unsigned)rank >= 8u || n_in != 4 || ws_bytes < WS_NEED || out_size != M * N || in_sizes[0] != M * KSH || in_sizes[1] != KSH * N) {
        fprintf(stderr, "unexpected problem shape\n");
        return;
    }
    static bool attr_set = false;
    if (!attr_set) {
        HIP_CHECK(hipFuncSetAttribute((const void*)fused_kernel, hipFuncAttributeMaxDynamicSharedMemorySize, LDS_BYTES));
        attr_set = true;
    }
    Params p;
    p.x = (const signed char*)d_in[0];
    p.w = (const signed char*)d_in[1];
    p.sx = (const float*)d_in[2];
    p.sw = (const float*)d_in[3];
    p.ws0 = (char*)d_ws_all[rank];
    p.out0 = (char*)d_out;
    for (int i = 0; i < 8; ++i) {
        const int q = i < 7 ? i + 1 : 1, r = (rank + q) & 7;
        p.peer[i] = (char*)d_ws_all[r] + OFF_FLAG;
        p.peer[8 + i] = (char*)d_out_all[r];
        p.peer[16 + i] = (char*)d_ws_all[r] + OFF_STAGE + (size_t)(8 - q) * REGION_BYTES;
    }
    p.rank = rank;
    hipLaunchKernelGGL(fused_kernel, dim3(NWG), dim3(512), LDS_BYTES, stream, p);
    HIP_CHECK(hipGetLastError());
}
```

```cpp
#include <hip/hip_runtime.h>
#include <rccl/rccl.h>
#include <cstdio>

#define HIP_CHECK(expr) do { hipError_t _e = (expr); if (_e != hipSuccess) { \
    fprintf(stderr, "HIP error %s at %s:%d\n", hipGetErrorString(_e), __FILE__, __LINE__); \
    return; } } while (0)

namespace {

typedef int i32x4 __attribute__((ext_vector_type(4)));
typedef unsigned u32x4 __attribute__((ext_vector_type(4)));
typedef float f32x4 __attribute__((ext_vector_type(4)));
typedef float f32x2 __attribute__((ext_vector_type(2)));
typedef __amdgpu_buffer_rsrc_t rsrc_t;
#define LAS __attribute__((address_space(3)))
#define GAS __attribute__((address_space(1)))

constexpr int M = 4096, N = 16384, KSH = 512;
constexpr unsigned OUT_BYTES = (unsigned)((size_t)M * N * 4);
constexpr int NWG = 256, NSEND = 16;
constexpr int TOT = 128;
constexpr unsigned OFF_CNT = 0;
constexpr unsigned OFF_TMO = 32768;
constexpr unsigned OFF_TICKET = 32768 + 128;
constexpr unsigned OFF_TILE = 32768 + 1024;
constexpr unsigned OFF_FLAG = 65536;
constexpr unsigned FLAG_DONE = 80 * 512;
constexpr unsigned OFF_STAGE = 4u << 20;
constexpr unsigned SHARD_BYTES = 10u << 20;
constexpr unsigned REGION_BYTES = 2u * SHARD_BYTES;
constexpr unsigned WS_NEED = OFF_STAGE + 8u * REGION_BYTES;
constexpr int LBUF = 65536;
constexpr int LDS_BYTES = 2 * LBUF + 64 + 1024;
constexpr unsigned DEADLINE_TICKS = 30000000u;
constexpr int FAR = 0x40000000;
constexpr int OOB = 0x7ffffff0;

constexpr int HEAD = 5;
constexpr int EA = 2, EB = 5;
constexpr int MERGE_FROM = 3;
constexpr int POLL_LOOK = 3;
#define AUX_SYS 17

constexpr unsigned long long SHARED_NEED = 0xffull | 0x03ull << 8 | 0x0cull << 16 | 0x30ull << 24 | 0xc0ull << 32;
constexpr int NIN = 1280, NOUT = 320, NUNITS = NIN + NOUT;
#define FIRST_ROWS {0, 0}, {0, 1}, {1, 0}, {1, 1}, {0, 2}, {1, 2}, {0, 3}, {1, 3}
constexpr int row_info(int row) {
    constexpr int first[8][2] = {FIRST_ROWS};
    int n = 0;
    for (int r = 0;; ++r) {
        const int g = r < 8 ? first[r][0] : r / 4, c = r < 8 ? first[r][1] : r % 4;
        if (r == row) return n | g << 16 | c << 20;
        n += g == 0 ? 64 : g == 2 ? 76 : 60;
    }
}
static_assert((row_info(20) & 0xffff) == NIN, "send rows");
#define SKEW 5
constexpr int skew_piece(int pos) {
    for (int c = 0; c < 4; ++c)
        for (int k = 0; k < 15; ++k) {
            int before = 0;
            for (int c2 = 0; c2 < 4; ++c2)
                for (int k2 = 0; k2 < 15; ++k2) before += (35 * c2 + SKEW * k2) * 4 + c2 < (35 * c + SKEW * k) * 4 + c ? 1 : 0;
            if (before == pos) return c << 4 | k;
        }
    return 0;
}
constexpr unsigned long long skew_word(int w) { unsigned long long r = 0; for (int i = 0; i < 10; ++i) r |= (unsigned long long)skew_piece(w * 10 + i) << (6 * i); return r; }

struct Params {
    const signed char* x;
    const signed char* w;
    const float* sx;
    const float* sw;
    char* ws0;
    char* out0;
    char* peer[24];
    int rank;
};

#define WAIT_V(n) asm volatile("s_waitcnt vmcnt(%0)" ::"n"(n) : "memory")
#define WAIT_L0() __builtin_amdgcn_s_waitcnt(0xC07F)
#define BAR() do { asm volatile("" ::: "memory"); __builtin_amdgcn_s_barrier(); asm volatile("" ::: "memory"); } while (0)
#define SCHED() __builtin_amdgcn_sched_barrier(0)

__device__ __forceinline__ rsrc_t mk_rsrc(const void* p, unsigned bytes) { return __builtin_amdgcn_make_buffer_rsrc((void*)p, 0, bytes, 0x00020000); }
__device__ __forceinline__ unsigned now_ticks() { return (unsigned)__builtin_amdgcn_s_memrealtime(); }
__device__ __forceinline__ char* lane_ptr(char* p, int l) {
    const unsigned lo = __builtin_amdgcn_readlane((unsigned)(unsigned long long)p, l), hi = __builtin_amdgcn_readlane((unsigned)((unsigned long long)p >> 32), l);
    return (char*)(((unsigned long long)hi << 32) | lo);
}

__device__ __forceinline__ void store_out(f32x4 v, rsrc_t r, int vo, unsigned so) {
    asm volatile("buffer_store_dwordx4 %0, %1, %2, %3 offen sc0 sc1" ::"v"(v), "v"(vo), "s"(r), "s"(so) : "memory");
}
__device__ __forceinline__ void store4k(i32x4 a, i32x4 b, i32x4 c, i32x4 d, rsrc_t r, int vo, unsigned so) {
    asm volatile("buffer_store_dwordx4 %0, %4, %5, %6 offen sc0 sc1\n\t"
                 "buffer_store_dwordx4 %1, %4, %5, %6 offen offset:1024 sc0 sc1\n\t"
                 "buffer_store_dwordx4 %2, %4, %5, %6 offen offset:2048 sc0 sc1\n\t"
                 "buffer_store_dwordx4 %3, %4, %5, %6 offen offset:3072 sc0 sc1\n\t"
                 "s_nop 1"
                 ::"v"(a), "v"(b), "v"(c), "v"(d), "v"(vo), "s"(r), "s"(so) : "memory");
}

__device__ __forceinline__ bool wait_expired(rsrc_t rW, unsigned call, unsigned w0) {
    const unsigned tv = __builtin_amdgcn_readfirstlane(__builtin_amdgcn_raw_buffer_load_b32(rW, 0, OFF_TMO, AUX_SYS));
    const bool dead = tv == call || now_ticks() - w0 > DEADLINE_TICKS;
    if (dead) __builtin_amdgcn_raw_buffer_store_b32(call, rW, 0, OFF_TMO, AUX_SYS);
    return dead;
}
template <int CNT>
__device__ __forceinline__ bool wait_words(rsrc_t rW, unsigned call, int vo, bool use, unsigned so, int step) {
    const unsigned w0 = now_ticks();
    for (unsigned spins = 1;; ++spins) {
        unsigned mn = 0xffffffffu;
#pragma unroll
        for (int i = 0; i < CNT; ++i) {
            const u32x4 f = __builtin_amdgcn_raw_buffer_load_b128(rW, use ? vo : OOB, so + i * step, AUX_SYS);
            mn = min(mn, min(min(f[0], f[1]), min(f[2], f[3])));
        }
        if (__ballot(use && mn < call) == 0ull) return true;
        if ((spins & 63u) == 0u && wait_expired(rW, call, w0)) return false;
        __builtin_amdgcn_s_sleep(8);
    }
}

__device__ __forceinline__ unsigned stage_off(int e) { return (unsigned)((e >> 8) & 7) << 21 | (unsigned)((e >> 2) & 3) << 19 | (unsigned)((e >> 4) & 15) << 15 | (unsigned)(e & 3) << 13; }

__device__ __forceinline__ void comm_role(const Params& P, rsrc_t rW, LAS char* lds, int ci, int b, int wid, int lane, unsigned call_v) {
    volatile LAS unsigned* lds_w = (volatile LAS unsigned*)lds;
    const rsrc_t rX = mk_rsrc(P.x, M * KSH);
    const int voff = lane * 16 + wid * 1024, vout = lane * 16 + wid * (N * 4);
    const int vx = ((lane >> 3) + wid * 8) * KSH + (((lane & 7) ^ (lane >> 4) ^ ((wid & 1) << 2)) << 4);
    i32x4 X, Y;
    asm volatile("buffer_load_dwordx4 %0, %2, %3, %4 offen\n\tbuffer_load_dwordx4 %1, %2, %3, %5 offen"
                 : "=&v"(X), "=&v"(Y) : "v"(vx), "s"(rX), "s"(ci * 64 * KSH), "s"((NSEND + ci) * 64 * KSH) : "memory");
    char* const peer = P.peer[lane < 24 ? lane : 0];
    const int rank = P.rank & 7;
    const int ql = (lane & 7) + 1;
    if (wid == 2) lds_w[2] = 0u;
    unsigned long long kneed = 0;
    const int rot = (ci + wid) % 7;
#pragma unroll
    for (int k = 0; k < 7; ++k) {
        int q = k + rot; q = (q >= 7 ? q - 7 : q) + 1;
        kneed |= ((SHARED_NEED >> ((rank + q) & 7)) & 0x0101010101ull) << k;
    }
    const rsrc_t rO = mk_rsrc(P.out0, OUT_BYTES);
    const int vtick = lane == 0 ? 0 : OOB;

    int row_tab = 0xffff;
#pragma unroll
    for (int r = 0; r < 20; ++r) row_tab = lane == r ? row_info(r) : row_tab;
    int skew_tab;
    {
        const int w = (lane * 26) >> 8;
        constexpr unsigned long long W0 = skew_word(0), W1 = skew_word(1), W2 = skew_word(2), W3 = skew_word(3), W4 = skew_word(4), W5 = skew_word(5);
        const unsigned long long word = w == 0 ? W0 : w == 1 ? W1 : w == 2 ? W2 : w == 3 ? W3 : w == 4 ? W4 : W5;
        skew_tab = (int)(word >> ((lane - 10 * w) * 6)) & 63;
    }
    auto code_of = [&](int n) {
        const int ri_ = __builtin_amdgcn_readlane(row_tab, __builtin_popcountll(__ballot((row_tab & 0xffff) <= n)) - 1);
        int i = n - (ri_ & 0xffff), g = (ri_ >> 16) & 7;
        int c = ri_ >> 20;
        if (c >= MERGE_FROM && g < 2) {
            const int p = (i >> 2) + g * 16, h = p >> 1;
            g = p & 1;
            i = (i & 3) | (g ? h : (h & 3) * 4 + (h >> 2)) << 2;
        }
        const int i5 = (i * 205) >> 10;
        const bool is_sh = g == 2 && (i - 5 * i5 == 4 || i == 75);
        const int sh = (i5 >> 2) | c << 2 | (31 + (i5 & 3) * 16) << 4 | ((i5 & 3) + 1) << 11;
        if (g == 2) {
            i -= i5;
            const int sp = __builtin_amdgcn_readlane(skew_tab, c * 15 + (i >> 2));
            c = sp >> 4; i = (i & 3) | (sp & 15) << 2;
        }
        return n >= NIN ? 0x4000 + n - NIN : is_sh ? sh : (i & 3) | c << 2 | (g * 16 + (i >> 2)) << 4;
    };
    auto out_dist = [&](int e) { const int i = e - 0x4000, j = (i * 205) >> 11; return i - 10 * j == 9 ? 7 - (j - ((j * 37) >> 8) * 7) : 0; };
    auto out_off = [&](int e) {
        const int i = e - 0x4000, j = (i * 205) >> 11, u = i - j, q = out_dist(e);
        const int row8 = q ? (8 + ((rank + q) & 7)) * 32 + j : u < 256 ? (rank & 1) * 256 + u : rank * 32 + u - 256;
        return (unsigned)(row8 * (8 * N * 4) + (q == 0 && u < 256 ? (rank >> 1) * 16 + 15 : 30) * 1024);
    };
    asm volatile("" : "+v"(row_tab), "+v"(skew_tab)); asm volatile("" : "+s"(kneed));
    SCHED();
    rsrc_t rD[7];
#pragma unroll
    for (int k = 0; k < 7; ++k) {
        int q = k + rot; q = q >= 7 ? q - 7 : q;
        rD[k] = mk_rsrc(lane_ptr(peer, 16 + q), REGION_BYTES);
    }
    const unsigned call = __builtin_amdgcn_readfirstlane(call_v) + 1u;
    const unsigned par_off = (call & 1u) * SHARD_BYTES;
    const unsigned tick_base = (call - 1u) * (unsigned)NUNITS - 3u * NSEND;
    auto flag_ptr = [&](int e) { return (GAS unsigned*)(peer + (((e >> 4) & 127) * 512 + (11 + 7 * ((e >> 2) & 3) - ql) * 16 + (e & 3) * 4)); };
    auto flag_on = [&](int e) { return lane < 7 && e < 0x4000 && ((SHARED_NEED >> (((e >> 11) & 7) * 8 + ((rank + ql) & 7))) & 1); };
    auto wait_packed = [&]() { if (!wait_words<4>(rW, call, lane * 512, true, OFF_FLAG + 16 * 512, 16)) lds_w[2] = 1u; };
    auto wait_tiles = [&]() { if (!wait_words<1>(rW, call, lane * 16, lane < 32, OFF_TILE, 0)) lds_w[2] = 1u; };
#define ST1(D, k) asm volatile("s_nop 4\n\tbuffer_store_dwordx4 %0, %1, %2, %3 offen sc0 sc1" ::"v"(D), "v"(vst), "s"(rD[k]), "s"(so[k]) : "memory")

    int e0 = (ci & 3) | (ci >> 2) << 4, e1 = e0 + 64;
    unsigned so[7];
#pragma unroll
    for (int k = 0; k < 7; ++k) so[k] = stage_off(e0) + par_off;
    GAS unsigned* fp = flag_ptr(e0);
    bool fon = false;
    asm volatile("s_waitcnt vmcnt(0)\n\ts_nop 4" : "+v"(X), "+v"(Y) :: "memory");
    int step = 0, vst = voff;
    unsigned tkv = 0;
    bool outs = false, tiles_ok = false;
    constexpr int CHECK_STEP = 5;
    const int vdraw = wid == 0 ? vtick : OOB;
    int chk_step = CHECK_STEP;
    auto snap_ok = [&]() {
        unsigned mn = 0xffffffffu;
#pragma unroll
        for (int i = 0; i < 4; ++i) { const u32x4 f = *(volatile LAS u32x4*)(lds + 1024 + i * 1024 + lane * 16); mn = min(mn, min(min(f[0], f[1]), min(f[2], f[3]))); }
        return __ballot(mn < call) == 0ull;
    };
#define STEP(D, O) do { \
        _Pragma("unroll") for (int k = 0; k < HEAD; ++k) ST1(D, k); \
        unsigned tks; \
        asm volatile("s_waitcnt vmcnt(%6)\n\tv_readfirstlane_b32 %0, %1\n\tv_mov_b32 %1, 1\n\ts_nop 1\n\tbuffer_atomic_add %1, %3, %4, %5 offen sc0 sc1" \
                     : "=&s"(tks), "+v"(tkv), "+v"(O) : "v"(vdraw), "s"(rW), "s"(OFF_TICKET), "n"(HEAD) : "memory"); \
        if (wid == 0) lds_w[step & 1] = tks; \
        if (wid == 2 && step == chk_step && !snap_ok()) { if (step == CHECK_STEP) wait_packed(); else wait_tiles(); } \
        WAIT_L0(); \
        BAR(); \
        if (fon) __hip_atomic_store(fp, call, __ATOMIC_RELAXED, __HIP_MEMORY_SCOPE_SYSTEM); \
        _Pragma("unroll") for (int k = HEAD; k < 7; ++k) ST1(D, k); \
        asm volatile("s_nop 1" ::"v"(D)); \
        SCHED(); \
        const unsigned tk = __builtin_amdgcn_readfirstlane(lds_w[step & 1]); \
        const bool dead = __builtin_amdgcn_readfirstlane(lds_w[2]) != 0u; \
        const int nn = step > 0 ? (int)(tk - tick_base) : 2 * NSEND + ci; \
        const int e2 = code_of(nn); \
        { const bool isO = e2 >= 0x4000, isA = e2 < 256; \
          if (step < CHECK_STEP && !isA) wait_packed(); \
          tiles_ok = tiles_ok || (step == chk_step && step > CHECK_STEP); \
          if (isO && !tiles_ok) { wait_tiles(); tiles_ok = true; } \
          if (step == CHECK_STEP - 1 || (nn >= NIN - 64 && chk_step == CHECK_STEP && step > CHECK_STEP)) { \
              chk_step = step + 1; \
              if (wid == 2) { const bool pk_ = step < CHECK_STEP; \
                  _Pragma("unroll") for (int i = 0; i < 4; ++i) \
                      __builtin_amdgcn_raw_ptr_buffer_load_lds(rW, lds + 1024 + i * 1024, 16, pk_ ? lane * 512 + i * 16 : (lane & 31) * 16, pk_ ? OFF_FLAG + 16 * 512 : OFF_TILE, 0, AUX_SYS); } } \
          const rsrc_t rs = isO ? rO : isA ? rX : rW; const int vs = isO ? vout : isA ? vx : voff; \
          const unsigned src = e2 >= 0x4000 + NOUT ? (unsigned)FAR : isO ? out_off(e2) : isA ? (unsigned)(((e2 >> 4) * 4 + (e2 & 3)) * (64 * KSH) + ((e2 >> 2) & 3) * 128) : stage_off(e2) + par_off + OFF_STAGE; \
          asm volatile("s_nop 4\n\tbuffer_load_dwordx4 %0, %1, %2, %3 offen sc1" : "=v"(D) : "v"(vs), "s"(rs), "s"(src) : "memory"); } \
        if (e1 >= 0x4000 && !outs) { \
            outs = true; vst = vout; \
            _Pragma("unroll") for (int k = 0; k < 7; ++k) { int q = k + rot; q = q >= 7 ? q - 7 : q; rD[k] = mk_rsrc(lane_ptr(peer, 8 + q), OUT_BYTES); } \
        } \
        { const unsigned dst = outs ? out_off(e1) : stage_off(e1) + par_off; \
          int ko = out_dist(e1) - 1 - rot; ko = ko < 0 ? ko + 7 : ko; \
          const unsigned km = outs ? (out_dist(e1) ? 1u << ko : 127u) : (unsigned)(kneed >> ((e1 >> 11) * 8)); \
          _Pragma("unroll") for (int k = 0; k < 7; ++k) so[k] = (km >> k) & 1 ? dst : (unsigned)FAR; } \
        fp = flag_ptr(e0); \
        fon = wid == 1 && !dead && flag_on(e0); \
        e0 = e1; e1 = e2; \
        ++step; \
    } while (0)
    for (;;) {
        STEP(X, Y);
        if (e0 >= 0x4000 + NOUT) break;
        STEP(Y, X);
        if (e0 >= 0x4000 + NOUT) break;
    }
    asm volatile("s_waitcnt vmcnt(0) lgkmcnt(0)" : "+v"(X), "+v"(Y), "+v"(tkv) :: "memory");
    BAR();
    if (wid == 1 && __builtin_amdgcn_readfirstlane(lds_w[2]) == 0u) {
        if (fon) __hip_atomic_store(fp, call, __ATOMIC_RELAXED, __HIP_MEMORY_SCOPE_SYSTEM);
        if (lane < 7) __hip_atomic_store((GAS unsigned*)(peer + (FLAG_DONE + (8 - ql) * 64 + ci * 4)), call, __ATOMIC_RELAXED, __HIP_MEMORY_SCOPE_SYSTEM);
    }
    if (wid == 0) {
        if (ci == 0 && !wait_words<1>(rW, call, lane * 16, lane >= 4 && lane < 32, OFF_FLAG + FLAG_DONE, 0))
            __builtin_amdgcn_raw_buffer_store_b32(0x7fc00000u, rO, vtick, 0, AUX_SYS);
        __builtin_amdgcn_raw_buffer_store_b32(call, rW, 0, OFF_CNT + b * 128, AUX_SYS);
    }
}

template <int GRP>
__device__ __forceinline__ void gemm_role(const Params& P, rsrc_t rW, LAS char* lds, int wid, int lane, unsigned call,
                                          int ri, int kc) {
    volatile LAS unsigned* lds_fail = (volatile LAS unsigned*)(lds + 2 * LBUF);
    const unsigned par_off = (call & 1u) * SHARD_BYTES;
    const int wc = wid & 3;
    const int voff = lane * 16;
    const float scale = P.sx[0] * P.sw[0];

    unsigned rb_p = (unsigned)ri * 0x01010101u, cb_p = (unsigned)kc * 0x01010101u + 0x30201000u;
    const bool shared = kc == 14;
    if (shared) {
        const int rank = P.rank & 7;
        rb_p = (rb_p & 0xffff00ffu) | (unsigned)(ri < 8 ? (rank & 1) * 8 + ri : ri == 8 + rank ? rank : ri) << 8;
        if (ri < 8) cb_p = (cb_p & 0xffff00ffu) | (unsigned)((rank >> 1) * 16 + 15) << 8;
    }

    int ready = wid == 4 ? 0 : 1 << 20, soft = ready, pend = -1;
    LAS char* const snap = lds + 2 * LBUF + 64;
    auto ensure_slow = [&](int need) {
        int vz = voff; asm volatile("" : "+v"(vz));
        bool use; int vf;
        auto aim = [&]() {
            const int round = ready >> 5, rb = (rb_p >> (8 * round)) & 255, cb = (cb_p >> (8 * round)) & 255;
            use = (vz & 511) >= (ready & 31) * 16;
            vf = use ? (vz & 511) + (vz < 512 ? 16 + cb : rb) * 512 : OOB;
        };
        auto take = [&](u32x4 f) {
            const unsigned long long bad = __ballot(use && min(min(f[0], f[1]), min(f[2], f[3])) < call);
            const unsigned bad32 = (unsigned)bad | (unsigned)(bad >> 32);
            ready = (ready & ~31) + (bad32 ? __builtin_ctz(bad32) : 32);
            ready = ready == TOT ? 1 << 20 : ready;
        };
        if (pend >= 0 && need >= pend) { aim(); take(*(volatile LAS u32x4*)(snap + vz)); pend = -1; }
        if (ready <= need) {
            const unsigned w0 = now_ticks();
            aim();
            for (unsigned spins = 1;; ++spins) {
                const int r0 = ready;
                take(__builtin_amdgcn_raw_buffer_load_b128(rW, vf, OFF_FLAG, AUX_SYS));
                if (ready > need) break;
                ready = r0;
                if ((spins & 63u) == 0u && wait_expired(rW, call, w0)) { lds_fail[0] = 1u; ready = 1 << 20; break; }
                __builtin_amdgcn_s_sleep(4);
            }
            pend = -1;
        } else if (pend < 0 && ready - need <= POLL_LOOK) {
            aim();
            __builtin_amdgcn_raw_ptr_buffer_load_lds(rW, snap, 16, vf, OFF_FLAG, 0, AUX_SYS);
            pend = need + (((need - 2) & 31) < 4 ? 2 : 1);
        }
        soft = pend >= 0 ? min(pend, ready) : max(ready - POLL_LOOK, need + 1);
    };
#define ENSURE(need) do { if (GRP == 1 && __builtin_expect(soft <= (need), 0)) ensure_slow(need); } while (0)

    const rsrc_t rS = mk_rsrc(P.ws0 + OFF_STAGE + par_off, 8u * REGION_BYTES - SHARD_BYTES);
    const rsrc_t rC = mk_rsrc(P.out0, OUT_BYTES);
    int src_tab;
    {
        const int s = lane & 31, u = s - 4, c = s < 4 ? s : (u * 37) >> 8;
        src_tab = (s < 4 ? 0 : (u - c * 7 + 1) * (int)REGION_BYTES) + (c << 19);
        if (lane >= 32) {
            const int j = lane & 7, rb = (rb_p >> (8 * (j & 3))) & 255, cb = (cb_p >> (8 * (j & 3))) & 255;
            src_tab = j >= 4 ? FAR : (lane & 8) ? (((cb >> 4) + 1) << 21) + (cb & 15) * 32768 + wid * 4096 : rb * 32768 + (GRP * 8 + wc) * 2048;
        }
    }
#define DMA(D, SO, IMM) __builtin_amdgcn_raw_ptr_buffer_load_lds(rS, D, 16, voff, SO, IMM, AUX_SYS)
#define ISSUE_B(so, buf) do { LAS char* d_ = lds + (buf) * LBUF + wid * 4096; DMA(d_, so, 0); DMA(d_, so, 1024); DMA(d_, so, 2048); DMA(d_, so, 3072); } while (0)
#define ISSUE_ALO(so, buf) do { LAS char* d_ = lds + (buf) * LBUF + 32768 + wid * 2048; DMA(d_, so, 0); DMA(d_, so, 1024); } while (0)
#define ISSUE_AHI(so, buf) do { LAS char* d_ = lds + (buf) * LBUF + 49152 + wid * 2048; DMA(d_, so, 0); DMA(d_, so, 1024); } while (0)

    i32x4 acc[8][4];
    i32x4 af[4][2], bf[4][2];
    const int rdA0 = 32768 + GRP * 8192 + (lane & 15) * 128 + (((lane >> 4) ^ ((lane >> 1) & 7)) << 4);
    LAS char* rdA[2][2] = {{lds + rdA0, lds + (rdA0 ^ 64)}, {lds + LBUF + rdA0, lds + LBUF + (rdA0 ^ 64)}};
    LAS char* rdB[2] = {lds + wc * 4096 + lane * 16, lds + LBUF + wc * 4096 + lane * 16};
    asm volatile("" : "+v"(rdA[0][0]), "+v"(rdA[0][1]), "+v"(rdA[1][0]), "+v"(rdA[1][1]), "+v"(rdB[0]), "+v"(rdB[1]));

    ENSURE(1);
    BAR();
    int aj = __builtin_amdgcn_readlane(src_tab, 32), bj = __builtin_amdgcn_readlane(src_tab, 40);
    ISSUE_B(bj, 0); ISSUE_ALO(aj, 0); ISSUE_AHI(aj + 8192, 0);
    ISSUE_B(bj + (1 << 19), 1); ISSUE_ALO(aj + (1 << 19), 1);
    int oA1 = aj + (1 << 19) + 8192;
    const int vo = (voff >> 8) * 4 * N * 4 + (voff & 255);
    unsigned so0 = FAR;
    f32x2 sc2 = {0.f, 0.f};
#pragma unroll
    for (int m = 0; m < 8; ++m)
#pragma unroll
        for (int t = 0; t < 4; ++t) acc[m][t] = (i32x4){0, 0, 0, 0};
    {
        const f32x4 z = {};
#pragma unroll
        for (int m = 0; m < 4 * EA; ++m) store_out(z, rC, vo, so0);
        asm volatile("s_nop 1" ::"v"(z));
    }
    WAIT_V(8 + 4 * EA);
    BAR();
    if (GRP == 1) BAR();

#define MP(ROW, A_, t, k0, Z) do { \
        if (Z) asm volatile("v_mfma_i32_16x16x64_i8 %0, %1, %2, 0\n\tv_mfma_i32_16x16x64_i8 %0, %3, %4, %0" : "=&v"(acc[ROW][t]) : "v"(A_[k0]), "v"(bf[t][k0]), "v"(A_[1 - (k0)]), "v"(bf[t][1 - (k0)])); \
        else asm volatile("v_mfma_i32_16x16x64_i8 %0, %1, %2, %0\n\tv_mfma_i32_16x16x64_i8 %0, %3, %4, %0" : "+v"(acc[ROW][t]) : "v"(A_[k0]), "v"(bf[t][k0]), "v"(A_[1 - (k0)]), "v"(bf[t][1 - (k0)])); \
        SCHED(); } while (0)
#define MROW_UP(ROW, A_, Z) do { MP(ROW, A_, 0, 0, Z); MP(ROW, A_, 1, 1, Z); MP(ROW, A_, 2, 0, Z); MP(ROW, A_, 3, 1, Z); } while (0)
#define MROW_DN(ROW, A_, Z) do { MP(ROW, A_, 3, 0, Z); MP(ROW, A_, 2, 1, Z); MP(ROW, A_, 1, 0, Z); MP(ROW, A_, 0, 1, Z); } while (0)
#define PIN(x) do { asm volatile("" : "+s"(x)); SCHED(); } while (0)
#define SEG_WAIT(n) do { WAIT_V(n); WAIT_L0(); SCHED(); BAR(); SCHED(); } while (0)
    auto conv1 = [&](int q) {
        const int m = q >> 2, r = q & 3;
        const f32x2 lo2 = (f32x2){(float)acc[m][0][r], (float)acc[m][1][r]} * sc2, hi2 = (f32x2){(float)acc[m][2][r], (float)acc[m][3][r]} * sc2;
        return __builtin_elementwise_maximum((f32x4){lo2[0], lo2[1], hi2[0], hi2[1]}, (f32x4){0.f, 0.f, 0.f, 0.f});
    };
#define PUT1(V, q) store_out(V, rC, vo, so0 + (unsigned)(((q) >> 2) * 16 + ((q) & 3)) * (unsigned)(N * 4))
#define FIX(m) asm volatile("" : "+v"(acc[m][0]), "+v"(acc[m][1]), "+v"(acc[m][2]), "+v"(acc[m][3]))
#define CONV(O, m) do { SCHED(); FIX(m); _Pragma("unroll") for (int r = 0; r < 4; ++r) O[r] = conv1(4 * (m) + r); } while (0)
#define PUT(O, m) do { _Pragma("unroll") for (int r = 0; r < 4; ++r) PUT1(O[r], 4 * (m) + r); asm volatile("s_nop 1" ::"v"(O[2]), "v"(O[3])); SCHED(); } while (0)
#define EMIT(lo, hi) do { if ((lo) < (hi)) { SCHED(); _Pragma("unroll") for (int m = (lo); m < (hi); ++m) FIX(m); f32x4 o = conv1(4 * (lo)); \
        _Pragma("unroll") for (int q = 4 * (lo); q < 4 * (hi); ++q) { \
            SCHED(); PUT1(o, q); SCHED(); \
            if (q + 1 < 4 * (hi)) { f32x4 n_ = conv1(q + 1); asm volatile("" : "+v"(n_) : "v"(o)); o = n_; } \
            else asm volatile("s_nop 1" ::"v"(o)); } \
        SCHED(); } } while (0)
#define HOLD(O) asm volatile("" : "+v"(O[0]), "+v"(O[1]), "+v"(O[2]), "+v"(O[3]))
    constexpr int SEG_ST[11] = {0, 0, 4 * EA, 4 * (EB - EA), 4 * (6 - EB), 4, 0, 4, 0, 0, 0};
#define CNT(i) (8 + SEG_ST[i] + SEG_ST[(i) - 1] + SEG_ST[(i) - 2])
    f32x4 p6[4], p7[4];
#define STAGE(buf, KIND) do { \
        ENSURE(g2); \
        _Pragma("unroll") for (int t = 0; t < 4; ++t) _Pragma("unroll") for (int kk = 0; kk < 2; ++kk) bf[t][kk] = *(LAS i32x4*)(rdB[buf] + kk * 16384 + t * 1024); \
        _Pragma("unroll") for (int m = 0; m < 4; ++m) _Pragma("unroll") for (int kk = 0; kk < 2; ++kk) af[m][kk] = *(LAS i32x4*)(rdA[buf][kk] + m * 2048); \
        ISSUE_AHI(oA1, (buf) ^ 1); \
        PIN(g2); \
        const int so2 = __builtin_amdgcn_readlane(src_tab, g2 & 31); \
        int oA2 = aj + so2, oB2 = bj + so2; oA1 = oA2 + 8192; g2 += 1; \
        asm volatile("" : "+s"(oA2), "+s"(oB2), "+s"(oA1), "+s"(g2)); SCHED(); \
        if (KIND == 1) EMIT(EA, EB); \
        if (KIND == 2) PUT(p6, 6); \
        if (KIND == 3) PUT(p7, 7); \
        SEG_WAIT(KIND >= 1 && KIND <= 4 ? CNT(2 * (KIND % 5) + 1 + (KIND == 0 || KIND == 5 ? 2 : 0)) : 8); \
        MROW_UP(0, af[0], KIND == 1); MROW_DN(1, af[1], KIND == 1); MROW_UP(2, af[2], KIND == 1); MROW_DN(3, af[3], KIND == 1); \
        SCHED(); BAR(); SCHED(); \
        _Pragma("unroll") for (int m = 0; m < 4; ++m) _Pragma("unroll") for (int kk = 0; kk < 2; ++kk) af[m][kk] = *(LAS i32x4*)(rdA[buf][kk] + 16384 + m * 2048); \
        unsigned failed_ = 0; if (KIND == 5) failed_ = lds_fail[0]; \
        ISSUE_B(oB2, buf); ISSUE_ALO(oA2, buf); \
        if (KIND == 5) { \
            const int rb = (rb_p >> (8 * round)) & 255, cb = (cb_p >> (8 * round)) & 255; \
            so0 = (unsigned)((rb * 256 + GRP * 128) * N + cb * 256 + wc * 64) * 4u; \
            sc2[0] = sc2[1] = failed_ ? __builtin_nanf("") : scale; \
            EMIT(0, EA); \
        } \
        if (KIND == 1) { EMIT(EB, 6); CONV(p6, 6); CONV(p7, 7); HOLD(p6); HOLD(p7); SCHED(); } \
        SEG_WAIT(KIND != 0 ? CNT(2 * (KIND % 5) + 2) : 8); \
        MROW_UP(4, af[0], KIND == 1); MROW_DN(5, af[1], KIND == 1); MROW_UP(6, af[2], KIND == 1); MROW_DN(7, af[3], KIND == 1); \
        SCHED(); BAR(); SCHED(); \
    } while (0)

#define EMPTY_STAGE(buf) do { \
        ISSUE_AHI(oA1, (buf) ^ 1); \
        if (GRP == 1 && ready <= g2) ensure_slow(g2); \
        const int so2 = __builtin_amdgcn_readlane(src_tab, g2 & 31), oA2 = aj + so2, oB2 = bj + so2; \
        oA1 = oA2 + 8192; g2 += 1; \
        SEG_WAIT(8); BAR(); \
        ISSUE_B(oB2, buf); ISSUE_ALO(oA2, buf); \
        SEG_WAIT(8); BAR(); \
    } while (0)

    int g2 = 2;
    for (int round = 0; round < 4; ++round) {
        STAGE(0, 1);
        STAGE(1, 2);
        STAGE(0, 3);
        STAGE(1, 4);
        if (shared && round == 2)
            __builtin_amdgcn_raw_buffer_store_b32(call, rW, voff == 0 ? 0 : OOB, OFF_TILE + (ri * 8 + wid) * 4, AUX_SYS);
        for (int part = round == 0 ? 0 : 2;; ++part) {
            const int stop = part < 2 ? 24 : round * 32 + 32;
            do {
                STAGE(0, 0);
                STAGE(1, 0);
            } while (g2 != stop);
            if (part == 2) break;
            if (part == 0) { aj = FAR; bj = FAR; g2 = 22; }
            else {
                aj = __builtin_amdgcn_readlane(src_tab, 32); bj = __builtin_amdgcn_readlane(src_tab, 40);
                EMPTY_STAGE(0);
                EMPTY_STAGE(1);
            }
        }
        aj = __builtin_amdgcn_readlane(src_tab, 33 + round); bj = __builtin_amdgcn_readlane(src_tab, 41 + round);
        STAGE(0, 0);
        STAGE(1, 5);
    }
    asm volatile("s_nop 7\n\ts_nop 7"); SCHED();
    EMIT(EA, 8);
    WAIT_V(0);
    if (GRP == 0) BAR();
}

__global__ __launch_bounds__(512, 2) void fused_kernel(Params P) {
    extern __shared__ __attribute__((aligned(16))) char smem_[];
    LAS char* lds = (LAS char*)smem_;
    const int tid = threadIdx.x, lane = tid & 63;
    const int wid = __builtin_amdgcn_readfirstlane(tid >> 6);
    const int b = blockIdx.x;
    const rsrc_t rW = mk_rsrc(P.ws0, WS_NEED);
    const unsigned call_v = __builtin_amdgcn_raw_buffer_load_b32(rW, 0, OFF_CNT + b * 128, AUX_SYS);

    if (b < 16) {
        comm_role(P, rW, lds, b, b, wid, lane, call_v);
        return;
    }

    const int n = wid * 240 + (b - 16);
    unsigned call;
    if (n < 1024) {
        const int j = n >> 8, c = (n >> 6) & 3, cqp = (n >> 2) & 15, part = n & 3;
        for (int i = 0; i < 3 + 2 * wid; ++i) __builtin_amdgcn_s_sleep(16);
        const rsrc_t rWm = mk_rsrc(P.w, KSH * N);
        const int vw = (lane >> 4) * 16 * N + (lane & 15) * 4;
        const unsigned sw0 = (unsigned)((c * 128 + (part >> 1) * 64) * N + (j * 16 + cqp) * 256 + (part & 1) * 128);
        unsigned d[2][16];
#pragma unroll
        for (int i = 0; i < 16; ++i) {
            d[0][i] = __builtin_amdgcn_raw_buffer_load_b32(rWm, vw, sw0 + i * N, 0);
            d[1][i] = __builtin_amdgcn_raw_buffer_load_b32(rWm, vw + 64, sw0 + i * N, 0);
        }
        call = __builtin_amdgcn_readfirstlane(call_v) + 1u;
        const unsigned so = OFF_STAGE + (call & 1u) * SHARD_BYTES + (2u << 20) + (unsigned)n * 8192u;
#pragma unroll
        for (int h = 0; h < 2; ++h) {
            i32x4 pb[4];
#pragma unroll
            for (int dd = 0; dd < 4; ++dd) {
                const unsigned d0 = d[h][4 * dd], d1 = d[h][4 * dd + 1], d2 = d[h][4 * dd + 2], d3 = d[h][4 * dd + 3];
                const unsigned lo01 = __builtin_amdgcn_perm(d1, d0, 0x05010400u), hi01 = __builtin_amdgcn_perm(d1, d0, 0x07030602u);
                const unsigned lo23 = __builtin_amdgcn_perm(d3, d2, 0x05010400u), hi23 = __builtin_amdgcn_perm(d3, d2, 0x07030602u);
                pb[0][dd] = (int)__builtin_amdgcn_perm(lo23, lo01, 0x05040100u);
                pb[1][dd] = (int)__builtin_amdgcn_perm(lo23, lo01, 0x07060302u);
                pb[2][dd] = (int)__builtin_amdgcn_perm(hi23, hi01, 0x05040100u);
                pb[3][dd] = (int)__builtin_amdgcn_perm(hi23, hi01, 0x07060302u);
            }
            store4k(pb[0], pb[1], pb[2], pb[3], rW, lane * 16, so + h * 4096u);
        }
        WAIT_V(0);
        __builtin_amdgcn_raw_buffer_store_b32(call, rW, lane == 0 ? 0 : OOB, OFF_FLAG + (16 + j * 16 + cqp) * 512 + c * 16 + part * 4, AUX_SYS);
    } else if (n >= 1200 && n < 1456) {
        const int c = (n - 1200) >> 6, un = (n - 1200) & 63;
        const rsrc_t rX = mk_rsrc(P.x, M * KSH);
        const int vx0 = (lane >> 3) * KSH + (((lane & 7) ^ (lane >> 4)) << 4), vx1 = (vx0 ^ 64) + 8 * KSH;
        u32x4 d[8];
#pragma unroll
        for (int i = 0; i < 8; ++i) d[i] = __builtin_amdgcn_raw_buffer_load_b128(rX, (i & 1) ? vx1 : vx0, (un * 64 + (i >> 1) * 16) * KSH + c * 128, 0);
        call = __builtin_amdgcn_readfirstlane(call_v) + 1u;
        const unsigned so = OFF_STAGE + (call & 1u) * SHARD_BYTES + ((unsigned)c << 19) + (unsigned)un * 8192u;
#pragma unroll
        for (int i = 0; i < 8; i += 4)
            store4k(__builtin_bit_cast(i32x4, d[i]), __builtin_bit_cast(i32x4, d[i + 1]), __builtin_bit_cast(i32x4, d[i + 2]), __builtin_bit_cast(i32x4, d[i + 3]), rW, lane * 16, so + i * 1024u);
        WAIT_V(0);
        __builtin_amdgcn_raw_buffer_store_b32(call, rW, lane == 0 ? 0 : OOB, OFF_FLAG + (un >> 2) * 512 + c * 16 + (un & 3) * 4, AUX_SYS);
    } else {
        call = __builtin_amdgcn_readfirstlane(call_v) + 1u;
    }
    if (wid == 4) *(volatile LAS unsigned*)(lds + 2 * LBUF) = 0u;

    const int xcd = b & 7, i30 = (b >> 3) - 2;
    const int kc = i30 < 28 ? (i30 >> 2) * 2 + (xcd & 1) : 14;
    const int ri = (xcd >> 1) * 4 + (i30 < 28 ? i30 & 3 : (xcd & 1) * 2 + (i30 - 28));
    if (wid < 4) gemm_role<0>(P, rW, lds, wid, lane, call, ri, kc);
    else gemm_role<1>(P, rW, lds, wid, lane, call, ri, kc);
    if (wid == 0) __builtin_amdgcn_raw_buffer_store_b32(call, rW, 0, OFF_CNT + b * 128, AUX_SYS);
}

}

extern "C" void kernel_launch(int rank, int nranks, ncclComm_t comm,
                              hipStream_t stream,
                              void* const* d_in, const int* in_sizes, int n_in,
                              void* d_out, int out_size,
                              const int* in_dtypes, int out_dtype,
                              void* const* d_out_all,
                              void* const* d_ws_all, size_t ws_bytes) {
    (void)comm; (void)in_dtypes; (void)out_dtype;
    if (nranks != 8 || (unsigned)rank >= 8u || n_in != 4 || ws_bytes < WS_NEED || out_size != M * N || in_sizes[0] != M * KSH || in_sizes[1] != KSH * N) {
        fprintf(stderr, "unexpected problem shape\n");
        return;
    }
    static bool attr_set = false;
    if (!attr_set) {
        HIP_CHECK(hipFuncSetAttribute((const void*)fused_kernel, hipFuncAttributeMaxDynamicSharedMemorySize, LDS_BYTES));
        attr_set = true;
    }
    Params p;
    p.x = (const signed char*)d_in[0];
    p.w = (const signed char*)d_in[1];
    p.sx = (const float*)d_in[2];
    p.sw = (const float*)d_in[3];
    p.ws0 = (char*)d_ws_all[rank];
    p.out0 = (char*)d_out;
    for (int i = 0; i < 8; ++i) {
        const int q = i < 7 ? i + 1 : 1, r = (rank + q) & 7;
        p.peer[i] = (char*)d_ws_all[r] + OFF_FLAG;
        p.peer[8 + i] = (char*)d_out_all[r];
        p.peer[16 + i] = (char*)d_ws_all[r] + OFF_STAGE + (size_t)(8 - q) * REGION_BYTES;
    }
    p.rank = rank;
    hipLaunchKernelGGL(fused_kernel, dim3(NWG), dim3(512), LDS_BYTES, stream, p);
    HIP_CHECK(hipGetLastError());
}
```
